```python
import numpy as np

kernel_name = "gpu_gemm_a2a_m4096_k4096_n16384_f32_reluq8"


ABSMAX_RTOL = 0.02

COMPUTE_DTYPE = "float32"
EPILOGUE = "relu_quant_int8"
IN_QUANT = None
OUT_QUANT = "int8"
IN_DTYPES = {"x": "float32", "w_mat": "float32"}
WORLD_SIZE = 8

def setup_inputs():

    x = np.random.randn(4096, 4096).astype(np.float32)
    w_mat = (np.random.randn(4096, 16384) / np.sqrt(4096)).astype(np.float32)
    return {"x": x, "w_mat": w_mat}

def _quant_dequant(y, amax):
    scale = amax / 127.0
    q = np.clip(np.round(y / scale), -127.0, 127.0)
    return (q * scale).astype(np.float32)
def _epilogue(y):
    y = np.maximum(y, 0.0)
    return _quant_dequant(y, np.abs(y).max())

def reference(x, w_mat):
    y = (x @ w_mat).astype(np.float32)
    return _epilogue(y)
```

```cpp
#include <hip/hip_runtime.h>
#include <rccl/rccl.h>
#include <cstdio>
#include <cstdint>

#define HIP_CHECK(expr) do { hipError_t _e = (expr); if (_e != hipSuccess) { \
    fprintf(stderr, "HIP error %s at %s:%d\n", hipGetErrorString(_e), __FILE__, __LINE__); \
    return; } } while (0)

namespace {

typedef unsigned u32x4 __attribute__((ext_vector_type(4)));
typedef float f32x4 __attribute__((ext_vector_type(4)));
typedef float f32x2 __attribute__((ext_vector_type(2)));
typedef __bf16 bf16x8 __attribute__((ext_vector_type(8)));
typedef __bf16 bf16x2 __attribute__((ext_vector_type(2)));
#define LAS __attribute__((address_space(3)))

#define FORCE_RAW 0
#define NWP 8

constexpr int NR = 8;
constexpr int KD = 4096, ND = 16384, NLOC = 2048;
constexpr int NSTEP = 128;
constexpr int NPULL = 28, NCOMM = 32, NGEMM = 224, NWG = 256;
constexpr int XSTG = 16384, WSTG = 8192;
constexpr int PKU = 12416;
constexpr size_t XB_OFF = 0;
constexpr size_t WB_OFF = 32u << 20;
constexpr size_t PK_OFF = 48u << 20;
constexpr size_t FL_OFF = 64u << 20;
constexpr size_t CNT_OFF = FL_OFF;
constexpr size_t TMO_OFF = FL_OFF + 0x4000;
constexpr size_t SF_OFF = FL_OFF + 0x5000;
constexpr size_t XF_OFF = FL_OFF + 0x8000;
constexpr size_t WF_OFF = FL_OFF + 0xA000;
constexpr size_t GR_OFF = FL_OFF + 0x10000;
constexpr size_t ST_OFF = FL_OFF + 0x20000;
constexpr size_t WS_NEED = FL_OFF + 0x40000;
constexpr int NS = 5;
constexpr int STG = 28672;
constexpr int SY = NS * STG;
constexpr int SY_FULL = SY, SY_READY = SY + 8, SY_DONE = SY + 16, SY_ARR = SY + 32, SY_WMAX = SY + 48,
              SY_GMAX = SY + 64, SY_BAD = SY + 80;
constexpr int LDS_BYTES = SY + 128;
constexpr unsigned long long DEADLINE = 50000000ull;
constexpr unsigned ABORT = 0x40000000u;

struct Params { const float* x; const float* w; char* out; char* ws[NR]; int rank; };

__device__ __forceinline__ unsigned long long rt() { return __builtin_amdgcn_s_memrealtime(); }
template <int N_> __device__ __forceinline__ void waitv() { asm volatile("s_waitcnt vmcnt(%0)" :: "i"(N_) : "memory"); }
__device__ __forceinline__ void cfence() { asm volatile("" ::: "memory"); }
__device__ __forceinline__ void st16(char* p, u32x4 v) {
  asm volatile("global_store_dwordx4 %0, %1, off sc0 sc1\n\ts_nop 1" :: "v"(p), "v"(v) : "memory");
}
__device__ __forceinline__ void st4(char* p, unsigned v) {
  asm volatile("global_store_dword %0, %1, off sc0 sc1" :: "v"(p), "v"(v) : "memory");
}
__device__ __forceinline__ void flag_st(char* p, unsigned v) {
  __hip_atomic_store((unsigned*)p, v, __ATOMIC_RELAXED, __HIP_MEMORY_SCOPE_SYSTEM);
}
__device__ __forceinline__ void poll16(u32x4& a, unsigned& t, const char* p, const char* tm) {
  asm volatile("global_load_dwordx4 %0, %2, off sc0 sc1\n\tglobal_load_dword %1, %3, off sc0 sc1\n\ts_waitcnt vmcnt(0)"
               : "=&v"(a), "=&v"(t) : "v"(p), "v"(tm) : "memory");
}
__device__ __forceinline__ void lds_st(int addr, unsigned v) {
  asm volatile("ds_write_b32 %0, %1" :: "v"(addr), "v"(v) : "memory");
}
__device__ __forceinline__ unsigned lds_ld(int addr) {
  unsigned v; asm volatile("ds_read_b32 %0, %1\n\ts_waitcnt lgkmcnt(0)" : "=&v"(v) : "v"(addr) : "memory"); return v;
}
__device__ __forceinline__ unsigned lds_add_rtn(int addr, unsigned a) {
  unsigned v; asm volatile("ds_add_rtn_u32 %0, %1, %2\n\ts_waitcnt lgkmcnt(0)" : "=&v"(v) : "v"(addr), "v"(a) : "memory"); return v;
}
__device__ __forceinline__ unsigned rfl(unsigned v) { return __builtin_amdgcn_readfirstlane(v); }
__device__ __forceinline__ unsigned umax(unsigned a, unsigned b) { return a > b ? a : b; }
__device__ __forceinline__ unsigned umin(unsigned a, unsigned b) { return a < b ? a : b; }
__device__ __forceinline__ unsigned wave_umax(unsigned v) {
  v = umax(v, (unsigned)__builtin_amdgcn_update_dpp(0, (int)v, 0x111, 0xf, 0xf, false));
  v = umax(v, (unsigned)__builtin_amdgcn_update_dpp(0, (int)v, 0x112, 0xf, 0xf, false));
  v = umax(v, (unsigned)__builtin_amdgcn_update_dpp(0, (int)v, 0x114, 0xf, 0xf, false));
  v = umax(v, (unsigned)__builtin_amdgcn_update_dpp(0, (int)v, 0x118, 0xf, 0xf, false));
  unsigned a = __builtin_amdgcn_readlane(v, 15), b = __builtin_amdgcn_readlane(v, 31);
  unsigned c = __builtin_amdgcn_readlane(v, 47), d = __builtin_amdgcn_readlane(v, 63);
  return umax(umax(a, b), umax(c, d));
}
__device__ __forceinline__ unsigned pk_bf16(float lo, float hi) {
  f32x2 f = {lo, hi};
  return __builtin_bit_cast(unsigned, __builtin_convertvector(f, bf16x2));
}
__device__ __forceinline__ unsigned perm(unsigned hi, unsigned lo, unsigned sel) { return __builtin_amdgcn_perm(hi, lo, sel); }
__device__ __forceinline__ unsigned rotr(unsigned v, unsigned n) { return __builtin_amdgcn_alignbit(v, v, n); }
__device__ __forceinline__ void glds4(unsigned voff, const char* sbase, unsigned dst) {
  asm volatile("s_mov_b32 m0, %2\n\ts_nop 4\n\t"
               "global_load_lds_dwordx4 %0, %1 sc0 sc1\n\t"
               "global_load_lds_dwordx4 %0, %1 offset:1024 sc0 sc1\n\t"
               "global_load_lds_dwordx4 %0, %1 offset:2048 sc0 sc1\n\t"
               "global_load_lds_dwordx4 %0, %1 offset:3072 sc0 sc1"
               :: "v"(voff), "s"(sbase), "s"(dst) : "memory");
}
#define STAMP(i) do {} while (0)

__device__ __forceinline__ bool expired(unsigned tmo_seen, unsigned call, unsigned long long t0, char* me) {
  if (rfl(tmo_seen) == call) return true;
  if (rt() - t0 > DEADLINE) {
    __hip_atomic_store((unsigned*)(me + TMO_OFF), call, __ATOMIC_RELAXED, __HIP_MEMORY_SCOPE_AGENT);
    return true;
  }
  return false;
}

__device__ __forceinline__ void convert_w(const Params& p, char* me, int panel, int s, int lane) {
  const float* wp = p.w + (size_t)(32 * s + 8 * (lane >> 4)) * ND + p.rank * NLOC + panel * 128 + 4 * (lane & 15);
  f32x4 v[2][8];
#pragma unroll
  for (int hf = 0; hf < 2; ++hf)
#pragma unroll
    for (int j = 0; j < 8; ++j) v[hf][j] = __builtin_nontemporal_load((const f32x4*)(wp + (size_t)j * ND + hf * 64));
  char* dst = me + WB_OFF + (size_t)(panel * NSTEP + s) * WSTG + lane * 16;
#pragma unroll
  for (int hf = 0; hf < 2; ++hf)
#pragma unroll
    for (int c = 0; c < 4; ++c) {
      u32x4 o = {pk_bf16(v[hf][0][c], v[hf][1][c]), pk_bf16(v[hf][2][c], v[hf][3][c]),
                 pk_bf16(v[hf][4][c], v[hf][5][c]), pk_bf16(v[hf][6][c], v[hf][7][c])};
      st16(dst + (4 * hf + c) * 1024, o);
    }
}

template <int NM, bool PF>
__device__ __forceinline__ void mstep(f32x4 (&acc)[NM][8], bf16x8 (&Wf)[8], bf16x8 (&Xc)[NM], bf16x8 (&Xn)[NM],
                                      LAS char* lds, int lane, int wave, unsigned& slot, unsigned& fpre, int s) {
  LAS char* xs = nullptr; LAS char* wsrc = nullptr;
  if (PF) {
    if (__builtin_expect(rfl(fpre) <= (unsigned)(s + 1), 0)) {
      do __builtin_amdgcn_s_sleep(1); while (rfl(lds_ld(SY_FULL + 4 * ((s + 1) & 1))) <= (unsigned)(s + 1));
    }
    cfence();
    slot = slot == (NS - 1) * STG ? 0 : slot + STG;
    xs = lds + slot + lane * 16 + wave * 4096;
    wsrc = lds + slot + lane * 16 + 20480;
  }
  __builtin_amdgcn_sched_barrier(0);
#pragma unroll
  for (int t = 0; t < 8; ++t) {
#pragma unroll
    for (int m = 0; m < NM; ++m) acc[m][t] = __builtin_amdgcn_mfma_f32_16x16x32_bf16(Wf[t], Xc[m], acc[m][t], 0, 0, 0);
#pragma unroll
    for (int m = 0; m < NM; ++m) asm volatile("" : "+v"(acc[m][t]));
    if (PF) {
      Wf[t] = *(LAS bf16x8*)(wsrc + t * 1024);
      if (t < 4) Xn[t] = *(LAS bf16x8*)(xs + t * 1024);
      if (NM == 5 && t == 4) Xn[NM - 1] = *(LAS bf16x8*)(lds + slot + lane * 16 + 16384 + wave * 1024);
      if (t == 5) fpre = *(volatile LAS unsigned*)(lds + SY_FULL + 4 * (s & 1));
      if (t == 7) { cfence(); *(volatile LAS unsigned*)(lds + SY_DONE + 4 * wave) = (unsigned)(s + 2); cfence(); }
    }
    __builtin_amdgcn_sched_barrier(0);
  }
}

template <int NM>
__device__ __forceinline__ void consumer(const Params& p, char* me, LAS char* lds, int lane, int wave, unsigned call,
                                         int gidx, int src, int h, int pn, int xj) {
  f32x4 acc[NM][8];
#pragma unroll
  for (int m = 0; m < NM; ++m)
#pragma unroll
    for (int t = 0; t < 8; ++t) acc[m][t] = f32x4{0.f, 0.f, 0.f, 0.f};
  bf16x8 Wf[8], Xa[NM], Xb[NM];
  unsigned slot = 0, fpre;
  {
    volatile LAS unsigned* fw = (volatile LAS unsigned*)(lds + SY_FULL);
    while (rfl(*fw) == 0) __builtin_amdgcn_s_sleep(2);
    cfence();
    STAMP(8 + wave);
#pragma unroll
    for (int t = 0; t < 8; ++t) Wf[t] = *(LAS bf16x8*)(lds + lane * 16 + 20480 + t * 1024);
#pragma unroll
    for (int t = 0; t < 4; ++t) Xa[t] = *(LAS bf16x8*)(lds + lane * 16 + wave * 4096 + t * 1024);
    if (NM == 5) Xa[NM - 1] = *(LAS bf16x8*)(lds + lane * 16 + 16384 + wave * 1024);
    fpre = fw[1];
    cfence(); *(volatile LAS unsigned*)(lds + SY_DONE + 4 * wave) = 1u; cfence();
  }
  for (int s = 0; s < NSTEP - 2; s += 2) {
    mstep<NM, true>(acc, Wf, Xa, Xb, lds, lane, wave, slot, fpre, s);
    mstep<NM, true>(acc, Wf, Xb, Xa, lds, lane, wave, slot, fpre, s + 1);
  }
  mstep<NM, true>(acc, Wf, Xa, Xb, lds, lane, wave, slot, fpre, NSTEP - 2);
  mstep<NM, false>(acc, Wf, Xb, Xa, lds, lane, wave, slot, fpre, NSTEP - 1);
  STAMP(12 + wave);
  float mx = 0.f;
#pragma unroll
  for (int m = 0; m < NM; ++m)
#pragma unroll
    for (int t = 0; t < 8; ++t) mx = fmaxf(fmaxf(mx, fmaxf(acc[m][t][0], acc[m][t][1])), fmaxf(acc[m][t][2], acc[m][t][3]));
  unsigned wm = wave_umax(__builtin_bit_cast(unsigned, mx));
  asm volatile("ds_max_u32 %0, %1" :: "v"(SY_WMAX), "v"(wm) : "memory");
  unsigned old = 0;
  if (lane == 0) old = lds_add_rtn(SY_ARR + 8, 1u);
  if (rfl(old) == 3) {
    unsigned m4 = lds_ld(SY_WMAX);
    if (lane < NR) {
      char* g = p.ws[lane] + GR_OFF + (size_t)(((call & 1) * NR + p.rank) * 256 + gidx) * 8;
      __hip_atomic_store((unsigned long long*)g, ((unsigned long long)call << 32) | m4, __ATOMIC_RELAXED, __HIP_MEMORY_SCOPE_SYSTEM);
    }
  }
#pragma unroll
  for (int m = 0; m < NM; ++m)
#pragma unroll
    for (int t = 0; t < 8; ++t)
#pragma unroll
      for (int j = 0; j < 4; ++j) { acc[m][t][j] = fmaxf(acc[m][t][j], 0.f); asm volatile("" : "+v"(acc[m][t][j])); }
  asm volatile("s_waitcnt lgkmcnt(0)" ::: "memory");
  __builtin_amdgcn_s_barrier();
  cfence();
  u32x4 g4 = *(volatile LAS u32x4*)(lds + SY_GMAX);
  unsigned bad = *(volatile LAS unsigned*)(lds + SY_BAD);
  float gmax = __builtin_bit_cast(float, umax(umax(g4[0], g4[1]), umax(g4[2], g4[3])));
  float inv = gmax > 0.f ? 127.0f / gmax : 0.f;
  STAMP(16 + wave);
  char* ob = p.out + pn * 128 + 16 * (lane >> 4);
#pragma unroll
  for (int m = 0; m < NM; ++m) {
    int row = (m < 4 ? src * 512 + 256 * h + 64 * wave + 16 * m : p.rank * 512 + 256 * h + 64 * xj + 16 * wave) + (lane & 15);
#pragma unroll
    for (int hf = 0; hf < 2; ++hf) {
      u32x4 o;
#pragma unroll
      for (int j = 0; j < 4; ++j) {
        unsigned q[4];
#pragma unroll
        for (int c = 0; c < 4; ++c) q[c] = __builtin_bit_cast(unsigned, __builtin_fmaf(acc[m][4 * hf + c][j], inv, 8388608.0f));
        o[j] = perm((q[3] << 8) | q[2], (q[1] << 8) | q[0], 0x05040100u);
      }
      if (bad) o = u32x4{0x80808080u, 0x80808080u, 0x80808080u, 0x80808080u};
      *(u32x4*)(ob + (size_t)row * NLOC + 64 * hf) = o;
    }
  }
  STAMP(20 + wave);
}

__global__ __launch_bounds__(512, 2) void fused(Params p) {
  extern __shared__ char smem[];
  LAS char* lds = (LAS char*)smem;
  const int lane = threadIdx.x & 63;
  const int wave = __builtin_amdgcn_readfirstlane(threadIdx.x >> 6);
  const int b = blockIdx.x;
  const int rank = p.rank;
  char* me = p.ws[rank];
  const char* tmo = me + TMO_OFF;
  if (wave == 0) STAMP(0);

  f32x4 xv[4][2];
  const int us = b >> 1, uh = b & 1;
  if (wave < 4) {
    const float* xp = p.x + (size_t)(256 * uh + 64 * wave + (lane & 15)) * KD + 32 * us + 8 * (lane >> 4);
#pragma unroll
    for (int f = 0; f < 4; ++f) {
      xv[f][0] = __builtin_nontemporal_load((const f32x4*)(xp + (size_t)f * 16 * KD));
      xv[f][1] = __builtin_nontemporal_load((const f32x4*)(xp + (size_t)f * 16 * KD + 4));
    }
  } else if (wave == 4) {
    if (lane < 32) lds_st(SY + lane * 4, 0u);
  }
  unsigned* cntp = (unsigned*)(me + CNT_OFF + b * 64);
  const unsigned call = rfl(__hip_atomic_load(cntp, __ATOMIC_RELAXED, __HIP_MEMORY_SCOPE_SYSTEM)) + 1u;
  if (wave < 4) {
    float mx = 0.f;
    u32x4 D[4];
#pragma unroll
    for (int f = 0; f < 4; ++f) {
#pragma unroll
      for (int i = 0; i < 2; ++i) {
        mx = fmaxf(fmaxf(mx, fmaxf(fabsf(xv[f][i][0]), fabsf(xv[f][i][1]))), fmaxf(fabsf(xv[f][i][2]), fabsf(xv[f][i][3])));
        D[f][2 * i] = pk_bf16(xv[f][i][0], xv[f][i][1]);
        D[f][2 * i + 1] = pk_bf16(xv[f][i][2], xv[f][i][3]);
      }
    }
    unsigned hi7 = wave_umax((pk_bf16(mx, 0.f) >> 8) & 0x7fu);
    unsigned base = hi7 > 7u ? hi7 - 7u : 0u;
    unsigned B4 = base * 0x01010101u;
    unsigned L[8], T[8], Hr[8], any = 0;
#pragma unroll
    for (int i = 0; i < 8; ++i) {
      unsigned d0 = D[i >> 1][2 * (i & 1)], d1 = D[i >> 1][2 * (i & 1) + 1];
      L[i] = perm(d1, d0, 0x06040200u);
      Hr[i] = perm(d1, d0, 0x07050301u);
      T[i] = (Hr[i] | 0x80808080u) - B4;
      any |= T[i] ^ 0x80808080u;
    }
    u32x4 N;
#pragma unroll
    for (int f = 0; f < 4; ++f) {
      unsigned c1 = (T[2 * f] & 0x07070707u) | (Hr[2 * f] & 0x80808080u);
      unsigned c2 = (T[2 * f + 1] & 0x07070707u) | (Hr[2 * f + 1] & 0x80808080u);
      N[f] = c1 | rotr(c2, 28);
    }
    char* pk = me + PK_OFF + (size_t)b * PKU;
    st16(pk + wave * 3072 + lane * 16, u32x4{L[0], L[1], L[2], L[3]});
    st16(pk + wave * 3072 + 1024 + lane * 16, u32x4{L[4], L[5], L[6], L[7]});
    st16(pk + wave * 3072 + 2048 + lane * 16, N);
    char* hdr = pk + 12288 + wave * 32;
    unsigned cnt = 0;
    if (__builtin_amdgcn_ballot_w64((any & 0xF8F8F8F8u) != 0)) {
#pragma unroll
      for (int i = 0; i < 8; ++i) {
        unsigned E = (T[i] ^ 0x80808080u) & 0xF8F8F8F8u;
        if (__builtin_amdgcn_ballot_w64(E != 0)) {
#pragma unroll
          for (int by = 0; by < 4; ++by) {
            bool fl = ((E >> (8 * by)) & 0xffu) != 0;
            unsigned long long mk = __builtin_amdgcn_ballot_w64(fl);
            unsigned idx = cnt + __builtin_amdgcn_mbcnt_hi((unsigned)(mk >> 32), __builtin_amdgcn_mbcnt_lo((unsigned)mk, 0u));
            if (fl && idx < 7u) st4(hdr + 4 + 4 * idx, ((Hr[i] >> (8 * by)) & 0xffu) | (lane << 8) | (by << 14) | (i << 16));
            cnt += (unsigned)__builtin_popcountll(mk);
          }
        }
      }
    }
    if (cnt > 7u || FORCE_RAW) cnt = 255u;
    if (lane == 0) st4(hdr, base | (cnt << 8));
    char* xl = me + XB_OFF + (size_t)((rank * 2 + uh) * NSTEP + us) * XSTG + wave * 4096 + lane * 16;
#pragma unroll
    for (int f = 0; f < 4; ++f) st16(xl + f * 1024, D[f]);
  }
  asm volatile("s_waitcnt lgkmcnt(0)" ::: "memory");
  __builtin_amdgcn_s_barrier();
  cfence();
  if (wave == 7 && lane == 0) __hip_atomic_store(cntp, call, __ATOMIC_RELAXED, __HIP_MEMORY_SCOPE_SYSTEM);
  if (wave < 4) {
    waitv<0>();
    unsigned old = 0;
    if (lane == 0) old = lds_add_rtn(SY_ARR, 1u);
    if (rfl(old) == 3) {
      if (lane < NR) {
        char* f = lane == rank ? me + XF_OFF + (size_t)((rank * 2 + uh) * NSTEP + us) * 4
                               : p.ws[lane] + SF_OFF + (size_t)(rank * 256 + b) * 4;
        flag_st(f, call);
      }
      STAMP(1);
    }
  }
  const unsigned long long t0 = rt();

  if (b < NPULL) {
    if (wave >= NWP) return;
    const int src = (rank + 1 + b % 7) & 7, pp = b / 7;
    const char* sws = p.ws[src];
    const char* sf = me + SF_OFF + src * 1024 + lane * 16;
    bool allset = false;
    const unsigned v0 = lane * 16, v1 = v0 + 4096, v2 = v0 + 8192, vh = 12288 + (lane & 31) * 4;
    for (int n = wave; n < 64; n += NWP) {
      const int u = n * 4 + pp, s = u >> 1, h = u & 1;
      if (!allset) {
        bool dead = false;
        for (;;) {
          u32x4 f; unsigned t;
          poll16(f, t, sf, tmo);
          allset = __builtin_amdgcn_ballot_w64(umin(umin(f[0], f[1]), umin(f[2], f[3])) >= call) == ~0ull;
          unsigned mine = pp == 0 ? f[0] : pp == 1 ? f[1] : pp == 2 ? f[2] : f[3];
          if ((unsigned)__builtin_amdgcn_readlane(mine, n) >= call) break;
          if (expired(t, call, t0, me)) { dead = true; break; }
          __builtin_amdgcn_s_sleep(2);
        }
        if (dead) return;
        if (n == wave) STAMP(8 + wave);
      }
      u32x4 A[12]; unsigned hd;
      const char* sb = sws + PK_OFF + (size_t)u * PKU;
      asm volatile("s_nop 4\n\t"
                   "global_load_dword %12, %16, %17 sc0 sc1\n\t"
                   "global_load_dwordx4 %0, %13, %17 sc0 sc1\n\t"
                   "global_load_dwordx4 %1, %13, %17 offset:1024 sc0 sc1\n\t"
                   "global_load_dwordx4 %2, %13, %17 offset:2048 sc0 sc1\n\t"
                   "global_load_dwordx4 %3, %13, %17 offset:3072 sc0 sc1\n\t"
                   "global_load_dwordx4 %4, %14, %17 sc0 sc1\n\t"
                   "global_load_dwordx4 %5, %14, %17 offset:1024 sc0 sc1\n\t"
                   "global_load_dwordx4 %6, %14, %17 offset:2048 sc0 sc1\n\t"
                   "global_load_dwordx4 %7, %14, %17 offset:3072 sc0 sc1\n\t"
                   "global_load_dwordx4 %8, %15, %17 sc0 sc1\n\t"
                   "global_load_dwordx4 %9, %15, %17 offset:1024 sc0 sc1\n\t"
                   "global_load_dwordx4 %10, %15, %17 offset:2048 sc0 sc1\n\t"
                   "global_load_dwordx4 %11, %15, %17 offset:3072 sc0 sc1\n\t"
                   "s_waitcnt vmcnt(0)"
                   : "=&v"(A[0]), "=&v"(A[1]), "=&v"(A[2]), "=&v"(A[3]), "=&v"(A[4]), "=&v"(A[5]), "=&v"(A[6]), "=&v"(A[7]),
                     "=&v"(A[8]), "=&v"(A[9]), "=&v"(A[10]), "=&v"(A[11]), "=&v"(hd)
                   : "v"(v0), "v"(v1), "v"(v2), "v"(vh), "s"(sb) : "memory");
      const size_t xo = XB_OFF + (size_t)((src * 2 + h) * NSTEP + s) * XSTG;
      char* xd = me + xo + lane * 16;
#pragma unroll
      for (int q = 0; q < 4; ++q) {
        const unsigned h0 = __builtin_amdgcn_readlane(hd, 8 * q);
        const unsigned cnt = (h0 >> 8) & 0xffu;
        if (cnt > 7u) {
          u32x4 R0, R1, R2, R3;
          const char* rb = sws + xo + q * 4096;
          asm volatile("s_nop 4\n\t"
                       "global_load_dwordx4 %0, %4, %5 sc0 sc1\n\t"
                       "global_load_dwordx4 %1, %4, %5 offset:1024 sc0 sc1\n\t"
                       "global_load_dwordx4 %2, %4, %5 offset:2048 sc0 sc1\n\t"
                       "global_load_dwordx4 %3, %4, %5 offset:3072 sc0 sc1\n\t"
                       "s_waitcnt vmcnt(0)"
                       : "=&v"(R0), "=&v"(R1), "=&v"(R2), "=&v"(R3) : "v"(v0), "s"(rb) : "memory");
          st16(xd + q * 4096, R0); st16(xd + q * 4096 + 1024, R1); st16(xd + q * 4096 + 2048, R2); st16(xd + q * 4096 + 3072, R3);
        } else {
          const unsigned B4 = (h0 & 0x7fu) * 0x01010101u;
          unsigned H[8];
#pragma unroll
          for (int f = 0; f < 4; ++f) {
            H[2 * f] = (A[3 * q + 2][f] & 0x87878787u) + B4;
            H[2 * f + 1] = (rotr(A[3 * q + 2][f], 4) & 0x87878787u) + B4;
          }
          for (unsigned e = 1; e <= cnt; ++e) {
            const unsigned ent = __builtin_amdgcn_readlane(hd, 8 * q + e);
            const unsigned sh = ((ent >> 14) & 3u) * 8u, wi = (ent >> 16) & 7u;
            const unsigned msk = (unsigned)lane == ((ent >> 8) & 63u) ? 0xffu << sh : 0u, val = (ent & 0xffu) << sh;
#pragma unroll
            for (int i = 0; i < 8; ++i) { unsigned mi = wi == (unsigned)i ? msk : 0u; H[i] = (H[i] & ~mi) | (val & mi); }
          }
#pragma unroll
          for (int f = 0; f < 4; ++f) {
            const unsigned l0 = A[3 * q + (f >> 1)][2 * (f & 1)], l1 = A[3 * q + (f >> 1)][2 * (f & 1) + 1];
            st16(xd + q * 4096 + f * 1024, u32x4{perm(H[2 * f], l0, 0x05010400u), perm(H[2 * f], l0, 0x07030602u),
                                                 perm(H[2 * f + 1], l1, 0x05010400u), perm(H[2 * f + 1], l1, 0x07030602u)});
          }
        }
      }
      waitv<0>();
      if (lane == 0) flag_st(me + XF_OFF + (size_t)((src * 2 + h) * NSTEP + s) * 4, call);
      if (n == wave) STAMP(16 + wave);
    }
    STAMP(24 + wave);
    return;
  }
  if (b < NCOMM) {
    for (int r = 0; r < 8; ++r) {
      const int id = r * 4 + (b - NPULL), kr = 14 + (id >> 4), panel = id & 15;
      convert_w(p, me, panel, kr * 8 + wave, lane);
      waitv<0>();
      unsigned old = 0;
      if (lane == 0) old = lds_add_rtn(SY_ARR + 4, 1u);
      if (rfl(old) == (unsigned)(8 * r + 7) && lane == 0) flag_st(me + WF_OFF + panel * 128 + kr * 4, call);
      asm volatile("s_waitcnt lgkmcnt(0)" ::: "memory");
      __builtin_amdgcn_s_barrier();
    }
    if (wave == 0) STAMP(2);
    return;
  }

  const int xcd = b & 7, idx = (b >> 3) - NCOMM / 8;
  const int h = xcd & 1, pn = (xcd >> 1) * 4 + (idx & 3), di = idx >> 2;
  const int src = (rank + 1 + di) & 7;
  const bool extra = di < 4;
  {
    const int kr = di * 2 + h;
    convert_w(p, me, pn, kr * 8 + wave, lane);
    waitv<0>();
    unsigned old = 0;
    if (lane == 0) old = lds_add_rtn(SY_ARR + 4, 1u);
    if (rfl(old) == 7u) { if (lane == 0) flag_st(me + WF_OFF + pn * 128 + kr * 4, call); STAMP(2); }
  }
  if (wave < 4) {
    if (extra) consumer<5>(p, me, lds, lane, wave, call, b - NCOMM, src, h, pn, di);
    else consumer<4>(p, me, lds, lane, wave, call, b - NCOMM, src, h, pn, di);
    return;
  }
  bool dead = false;
  if (wave < 6) {
    __builtin_amdgcn_s_setprio(2);
    const char* xs = me + XB_OFF + (size_t)((src * 2 + h) * NSTEP) * XSTG;
    const char* ls = me + XB_OFF + (size_t)((rank * 2 + h) * NSTEP) * XSTG + di * 4096;
    const char* wsr = me + WB_OFF + (size_t)(pn * NSTEP) * WSTG;
    const unsigned voff = lane * 16;
    unsigned slot = (wave - 4) * STG;
    for (int s = wave - 4; s < NSTEP; s += 2) {
      unsigned rdy;
      for (;;) {
        u32x4 d; unsigned r;
        asm volatile("ds_read_b128 %0, %2\n\tds_read_b32 %1, %3\n\ts_waitcnt lgkmcnt(0)" : "=&v"(d), "=&v"(r) : "v"(SY_DONE), "v"(SY_READY) : "memory");
        rdy = rfl(r);
        unsigned dm = rfl(umin(umin(d[0], d[1]), umin(d[2], d[3])));
        if (rdy >= ABORT || (rdy > (unsigned)s && dm + NS > (unsigned)s)) break;
        __builtin_amdgcn_s_sleep(1);
      }
      if (rdy >= ABORT) { lds_st(SY_FULL + 4 * (wave - 4), ABORT); break; }
      const char* xa = xs + (size_t)s * XSTG;
      glds4(voff, xa, slot); glds4(voff, xa + 4096, slot + 4096); glds4(voff, xa + 8192, slot + 8192); glds4(voff, xa + 12288, slot + 12288);
      if (extra) glds4(voff, ls + (size_t)s * XSTG, slot + 16384);
      glds4(voff, wsr + (size_t)s * WSTG, slot + 20480); glds4(voff, wsr + (size_t)s * WSTG + 4096, slot + 24576);
      waitv<0>();
      lds_st(SY_FULL + 4 * (wave - 4), (unsigned)(s + 1));
      slot += 2 * STG; if (slot >= NS * STG) slot -= NS * STG;
    }
    __builtin_amdgcn_s_setprio(0);
  } else if (wave == 6) {
    __builtin_amdgcn_s_setprio(3);
    if (extra) {
      const char* lf = me + XF_OFF + (size_t)((rank * 2 + h) * NSTEP) * 4 + (lane & 31) * 16;
      for (;;) {
        u32x4 f; unsigned t;
        poll16(f, t, lf, tmo);
        if (__builtin_amdgcn_ballot_w64(umin(umin(f[0], f[1]), umin(f[2], f[3])) >= call) == ~0ull) break;
        if (expired(t, call, t0, me)) { dead = true; break; }
        __builtin_amdgcn_s_sleep(2);
      }
    }
    const char* pa = lane < 32 ? me + XF_OFF + (size_t)((src * 2 + h) * NSTEP) * 4 + lane * 16
                               : me + WF_OFF + pn * 128 + (lane & 3) * 16;
    unsigned ready = 0;
    while (!dead && ready < (unsigned)NSTEP) {
      u32x4 f; unsigned t;
      poll16(f, t, pa, tmo);
      unsigned c = f[0] < call ? 0u : f[1] < call ? 1u : f[2] < call ? 2u : f[3] < call ? 3u : 4u;
      unsigned long long part = __builtin_amdgcn_ballot_w64(c < 4u);
      unsigned px = (unsigned)part, pw = (unsigned)(part >> 32) & 0xfu;
      unsigned rx = NSTEP, rw = NSTEP;
      if (px) { unsigned l = __builtin_ctz(px); rx = 4 * l + __builtin_amdgcn_readlane(c, l); }
      if (pw) { unsigned l = __builtin_ctz(pw); rw = 8 * (4 * l + __builtin_amdgcn_readlane(c, 32 + l)); }
      unsigned r = umin(rx, rw);
      if (r > ready) { ready = r; lds_st(SY_READY, ready); continue; }
      if (expired(t, call, t0, me)) { dead = true; break; }
      __builtin_amdgcn_s_sleep(1);
    }
    if (dead) { lds_st(SY_BAD, 1u); lds_st(SY_READY, ABORT); }
    STAMP(3);
    __builtin_amdgcn_s_setprio(0);
  }
  while (rfl(lds_ld(SY_DONE)) < (unsigned)(NSTEP - 1) ) __builtin_amdgcn_s_sleep(8);
  {
    const int l56 = lane < 56 ? lane : 55;
    const char* g0 = me + GR_OFF + (size_t)(((call & 1) * NR + 2 * (wave - 4)) * 256) * 8 + l56 * 32;
    unsigned best = 0;
    for (;;) {
      u32x4 a0, a1, a2, a3; unsigned t;
      asm volatile("global_load_dwordx4 %0, %5, off sc0 sc1\n\tglobal_load_dwordx4 %1, %5, off offset:16 sc0 sc1\n\t"
                   "global_load_dwordx4 %2, %5, off offset:2048 sc0 sc1\n\tglobal_load_dwordx4 %3, %5, off offset:2064 sc0 sc1\n\t"
                   "global_load_dword %4, %6, off sc0 sc1\n\ts_waitcnt vmcnt(0)"
                   : "=&v"(a0), "=&v"(a1), "=&v"(a2), "=&v"(a3), "=&v"(t) : "v"(g0), "v"(tmo) : "memory");
      bool ok = a0[1] == call && a0[3] == call && a1[1] == call && a1[3] == call && a2[1] == call && a2[3] == call && a3[1] == call && a3[3] == call;
      if (rfl(t) == call) { dead = true; break; }
      if (__builtin_amdgcn_ballot_w64(ok) == ~0ull) {
        best = wave_umax(umax(umax(umax(a0[0], a0[2]), umax(a1[0], a1[2])), umax(umax(a2[0], a2[2]), umax(a3[0], a3[2]))));
        break;
      }
      if (expired(t, call, t0, me)) { dead = true; break; }
      __builtin_amdgcn_s_sleep(2);
    }
    if (dead) lds_st(SY_BAD, 1u);
    lds_st(SY_GMAX + 4 * (wave - 4), best);
  }
  STAMP(wave);
  asm volatile("s_waitcnt lgkmcnt(0)" ::: "memory");
  __builtin_amdgcn_s_barrier();
}

}

extern "C" void kernel_launch(int rank, int nranks, ncclComm_t comm, hipStream_t stream,
                              void* const* d_in, const int* in_sizes, int n_in,
                              void* d_out, int out_size, const int* in_dtypes, int out_dtype,
                              void* const* d_out_all, void* const* d_ws_all, size_t ws_bytes) {
  if (nranks != NR || (unsigned)rank >= (unsigned)NR || n_in < 2 || ws_bytes < WS_NEED ||
      in_sizes[0] != 512 * KD || out_size != 4096 * NLOC) {
    fprintf(stderr, "kernel_launch: unexpected problem shape\n");
    return;
  }
  HIP_CHECK(hipFuncSetAttribute((const void*)fused, hipFuncAttributeMaxDynamicSharedMemorySize, LDS_BYTES));
  Params p;
  p.x = (const float*)d_in[0]; p.w = (const float*)d_in[1]; p.out = (char*)d_out; p.rank = rank;
  for (int i = 0; i < NR; ++i) p.ws[i] = (char*)d_ws_all[i];
  fused<<<NWG, 512, LDS_BYTES, stream>>>(p);
  HIP_CHECK(hipGetLastError());
}
```

```cpp
#include <hip/hip_runtime.h>
#include <rccl/rccl.h>
#include <cstdio>
#include <cstdint>

#define HIP_CHECK(expr) do { hipError_t _e = (expr); if (_e != hipSuccess) { \
    fprintf(stderr, "HIP error %s at %s:%d\n", hipGetErrorString(_e), __FILE__, __LINE__); \
    return; } } while (0)

namespace {

typedef unsigned u32x4 __attribute__((ext_vector_type(4)));
typedef unsigned u32x2 __attribute__((ext_vector_type(2)));
typedef float f32x4 __attribute__((ext_vector_type(4)));
typedef float f32x2 __attribute__((ext_vector_type(2)));
typedef __bf16 bf16x8 __attribute__((ext_vector_type(8)));
typedef __bf16 bf16x2 __attribute__((ext_vector_type(2)));
#define LAS __attribute__((address_space(3)))

#define FORCE_RAW 0
#define WLEAD 2
#define PRIOC 0
#define PRIOL 2
#define PRIOP 3
#define SLEEPL 1
#define SLEEPP 4
#define NWP 4

constexpr int NR = 8;
constexpr int KD = 4096, ND = 16384, NLOC = 2048;
constexpr int NSTEP = 128;
constexpr int NPULL = 28, NCOMM = 32, NGEMM = 224, NWG = 256;
constexpr int XSTG = 16384, WSTG = 8192;
constexpr int PKU = 12416;
constexpr size_t XB_OFF = 0;
constexpr size_t WB_OFF = 32u << 20;
constexpr size_t PK_OFF = 48u << 20;
constexpr size_t FL_OFF = 64u << 20;
constexpr size_t CNT_OFF = FL_OFF;
constexpr size_t TMO_OFF = FL_OFF + 0x4000;
constexpr size_t SF_OFF = FL_OFF + 0x5000;
constexpr size_t XF_OFF = FL_OFF + 0x8000;
constexpr size_t WF_OFF = FL_OFF + 0xA000;
constexpr size_t GM_OFF = FL_OFF + 0xC000;
constexpr size_t GR_OFF = FL_OFF + 0x10000;
constexpr size_t ST_OFF = FL_OFF + 0x20000;
constexpr size_t WS_NEED = FL_OFF + 0x40000;
constexpr int NS = 5;
constexpr int STG = 28672;
constexpr int SY = NS * STG;
constexpr int SY_FULL = SY, SY_READY = SY + 8, SY_DONE = SY + 16, SY_ARR = SY + 32, SY_WMAX = SY + 48,
              SY_GMAX = SY + 64, SY_BAD = SY + 80, SY_TKT = SY + 84, SY_PTR = SY + 128;
constexpr int LDS_BYTES = SY + 192;
constexpr unsigned long long DEADLINE = 50000000ull;
constexpr unsigned ABORT = 0x40000000u;

struct Params { const float* x; const float* w; char* out; char* ws[NR]; int rank; };

__device__ __forceinline__ unsigned long long rt() { return __builtin_amdgcn_s_memrealtime(); }
template <int N_> __device__ __forceinline__ void waitv() { asm volatile("s_waitcnt vmcnt(%0)" :: "i"(N_) : "memory"); }
__device__ __forceinline__ void cfence() { asm volatile("" ::: "memory"); }
__device__ __forceinline__ void st16(char* p, u32x4 v) {
  asm volatile("global_store_dwordx4 %0, %1, off sc0 sc1\n\ts_nop 1" :: "v"(p), "v"(v) : "memory");
}
__device__ __forceinline__ void st4(char* p, unsigned v) {
  asm volatile("global_store_dword %0, %1, off sc0 sc1" :: "v"(p), "v"(v) : "memory");
}
__device__ __forceinline__ void flag_st(char* p, unsigned v) {
  __hip_atomic_store((unsigned*)p, v, __ATOMIC_RELAXED, __HIP_MEMORY_SCOPE_SYSTEM);
}
__device__ __forceinline__ void poll16(u32x4& a, unsigned& t, const char* p, const char* tm) {
  asm volatile("global_load_dwordx4 %0, %2, off sc0 sc1\n\tglobal_load_dword %1, %3, off sc0 sc1\n\ts_waitcnt vmcnt(0)"
               : "=&v"(a), "=&v"(t) : "v"(p), "v"(tm) : "memory");
}
__device__ __forceinline__ void lds_st(int addr, unsigned v) {
  asm volatile("ds_write_b32 %0, %1" :: "v"(addr), "v"(v) : "memory");
}
__device__ __forceinline__ unsigned lds_ld(int addr) {
  unsigned v; asm volatile("ds_read_b32 %0, %1\n\ts_waitcnt lgkmcnt(0)" : "=&v"(v) : "v"(addr) : "memory"); return v;
}
__device__ __forceinline__ unsigned lds_add_rtn(int addr, unsigned a) {
  unsigned v; asm volatile("ds_add_rtn_u32 %0, %1, %2\n\ts_waitcnt lgkmcnt(0)" : "=&v"(v) : "v"(addr), "v"(a) : "memory"); return v;
}
__device__ __forceinline__ unsigned rfl(unsigned v) { return __builtin_amdgcn_readfirstlane(v); }
__device__ __forceinline__ unsigned umax(unsigned a, unsigned b) { return a > b ? a : b; }
__device__ __forceinline__ unsigned umin(unsigned a, unsigned b) { return a < b ? a : b; }
__device__ __forceinline__ unsigned wave_umax(unsigned v) {
  v = umax(v, (unsigned)__builtin_amdgcn_update_dpp(0, (int)v, 0x111, 0xf, 0xf, false));
  v = umax(v, (unsigned)__builtin_amdgcn_update_dpp(0, (int)v, 0x112, 0xf, 0xf, false));
  v = umax(v, (unsigned)__builtin_amdgcn_update_dpp(0, (int)v, 0x114, 0xf, 0xf, false));
  v = umax(v, (unsigned)__builtin_amdgcn_update_dpp(0, (int)v, 0x118, 0xf, 0xf, false));
  unsigned a = __builtin_amdgcn_readlane(v, 15), b = __builtin_amdgcn_readlane(v, 31);
  unsigned c = __builtin_amdgcn_readlane(v, 47), d = __builtin_amdgcn_readlane(v, 63);
  return umax(umax(a, b), umax(c, d));
}
__device__ __forceinline__ unsigned pk_bf16(float lo, float hi) {
  f32x2 f = {lo, hi};
  return __builtin_bit_cast(unsigned, __builtin_convertvector(f, bf16x2));
}
__device__ __forceinline__ unsigned perm(unsigned hi, unsigned lo, unsigned sel) { return __builtin_amdgcn_perm(hi, lo, sel); }
__device__ __forceinline__ unsigned rotr(unsigned v, unsigned n) { return __builtin_amdgcn_alignbit(v, v, n); }
__device__ __forceinline__ void glds4(unsigned voff, const char* sbase, unsigned dst) {
  asm volatile("s_mov_b32 m0, %2\n\ts_nop 0\n\t"
               "global_load_lds_dwordx4 %0, %1 sc0 sc1\n\t"
               "global_load_lds_dwordx4 %0, %1 offset:1024 sc0 sc1\n\t"
               "global_load_lds_dwordx4 %0, %1 offset:2048 sc0 sc1\n\t"
               "global_load_lds_dwordx4 %0, %1 offset:3072 sc0 sc1"
               :: "v"(voff), "s"(sbase), "s"(dst) : "memory");
}
__device__ __forceinline__ void glds4s(unsigned voff, const char* sbase, unsigned dst) {
  asm volatile("s_mov_b32 m0, %2\n\ts_nop 0\n\t"
               "global_load_lds_dwordx4 %0, %1 sc0 sc1\n\t"
               "s_add_u32 m0, m0, 4096\n\ts_nop 0\n\t"
               "global_load_lds_dwordx4 %0, %1 offset:1024 sc0 sc1\n\t"
               "s_add_u32 m0, m0, 4096\n\ts_nop 0\n\t"
               "global_load_lds_dwordx4 %0, %1 offset:2048 sc0 sc1\n\t"
               "s_add_u32 m0, m0, 4096\n\ts_nop 0\n\t"
               "global_load_lds_dwordx4 %0, %1 offset:3072 sc0 sc1"
               :: "v"(voff), "s"(sbase), "s"(dst) : "memory", "scc");
}
#define STAMP(i) do {} while (0)

__device__ __forceinline__ bool expired(unsigned tmo_seen, unsigned call, unsigned long long t0, char* me) {
  if (rfl(tmo_seen) == call) return true;
  if (rt() - t0 > DEADLINE) {
    __hip_atomic_store((unsigned*)(me + TMO_OFF), call, __ATOMIC_RELAXED, __HIP_MEMORY_SCOPE_AGENT);
    return true;
  }
  return false;
}

__device__ __forceinline__ void convert_w(const Params& p, char* me, int panel, int s, int lane, unsigned call) {
  const float* wp = p.w + (size_t)(32 * s + 8 * (lane >> 4)) * ND + p.rank * NLOC + panel * 128 + 4 * (lane & 15);
  f32x4 v[2][8];
#pragma unroll
  for (int hf = 0; hf < 2; ++hf)
#pragma unroll
    for (int j = 0; j < 8; ++j) v[hf][j] = __builtin_nontemporal_load((const f32x4*)(wp + (size_t)j * ND + hf * 64));
  char* dst = me + WB_OFF + (size_t)(panel * NSTEP + s) * WSTG + lane * 16;
#pragma unroll
  for (int hf = 0; hf < 2; ++hf)
#pragma unroll
    for (int c = 0; c < 4; ++c) {
      u32x4 o = {pk_bf16(v[hf][0][c], v[hf][1][c]), pk_bf16(v[hf][2][c], v[hf][3][c]),
                 pk_bf16(v[hf][4][c], v[hf][5][c]), pk_bf16(v[hf][6][c], v[hf][7][c])};
      st16(dst + (4 * hf + c) * 1024, o);
    }
  waitv<0>();
  if (lane == 0) flag_st(me + WF_OFF + (size_t)(panel * NSTEP + s) * 4, call);
}

#define MFMA(m, t) asm volatile("v_mfma_f32_16x16x32_bf16 %0, %1, %2, %0" : "+v"(acc[m][t]) : "v"(Wf[t]), "v"(Xc[m]))
#define LDR(dst, addr, off) asm volatile("ds_read_b128 %0, %1 offset:%2" : "=v"(dst) : "v"(addr), "i"(off) : "memory")
template <int N_> __device__ __forceinline__ void waitl() { asm volatile("s_waitcnt lgkmcnt(%0)" :: "i"(N_) : "memory"); }
constexpr int WOF = 20480;
#define MAX2(m, t) asm volatile("v_max3_f32 %0, %2, %3, %0\n\tv_max3_f32 %1, %4, %5, %1" : "+v"(mx0), "+v"(mx1) \
                                : "v"(acc[m][t][0]), "v"(acc[m][t][1]), "v"(acc[m][t][2]), "v"(acc[m][t][3]))
template <int NM, int MODE, int PAR>
__device__ __forceinline__ void mstep(f32x4 (&acc)[NM][8], bf16x8 (&Wf)[8], bf16x8 (&Xc)[NM], bf16x8 (&Xn)[NM],
                                      unsigned& va, unsigned& vx, unsigned va0, unsigned vx0, unsigned vsy, unsigned& vdone,
                                      unsigned& slot, unsigned& fpre, int s, float& mx0, float& mx1, unsigned& nslow) {
  constexpr bool PF = MODE < 2;
  unsigned sf = 0, nslot = 0;
  waitl<PF ? 3 : 2>();
  MFMA(0, 0);
  LDR(Wf[7], va, WOF + 7168);
  MFMA(1, 0);
  if (PF) asm volatile("ds_write_b32 %0, %1 offset:16" :: "v"(vsy), "v"(vdone) : "memory");
  MFMA(2, 0);
  if (PF) asm volatile("s_waitcnt lgkmcnt(3)\n\tv_readfirstlane_b32 %0, %1" : "=s"(sf) : "v"(fpre) : "memory");
  MFMA(3, 0);
  if (PF) asm volatile("v_add_u32 %0, %1, %2" : "=v"(va) : "s"(slot), "v"(va0));
  if (NM == 5) MFMA(NM - 1, 0);
  if (PF && __builtin_expect(sf <= (unsigned)(s + 1), 0)) {
    do __builtin_amdgcn_s_sleep(1); while (rfl(lds_ld(SY_FULL + 4 * (1 - PAR))) <= (unsigned)(s + 1));
  }
#pragma unroll
  for (int t = 1; t < 8; ++t) {
    if (t == 5 && !PF) waitl<2>();
    if (t == 6) waitl<PF ? NM + 6 : 0>();
    MFMA(0, t);
    if (PF) LDR(Wf[t - 1], va, WOF + (t - 1) * 1024); else MAX2(0, t - 1);
    MFMA(1, t);
    if (PF && t == 1) asm volatile("v_add_u32 %0, %1, %2" : "=v"(vx) : "s"(slot), "v"(vx0));
    if (PF && t == 2) asm volatile("v_add_u32 %0, 1, %0" : "+v"(vdone));
    if (MODE == 0 && t == 7) asm volatile("s_add_u32 %0, %1, %2" : "=s"(nslot) : "s"(slot), "i"(STG) : "scc");
    if (!PF) MAX2(1, t - 1);
    MFMA(2, t);
    if (PF && t <= NM) LDR(Xn[t - 1], vx, (t - 1) * 1024);
    if (MODE == 0 && t == 6) asm volatile("ds_read_b32 %0, %1 offset:%2" : "=v"(fpre) : "v"(SY_FULL), "i"(4 * PAR) : "memory");
    if (MODE == 0 && t == 7) asm volatile("s_cmp_lg_u32 %1, %2\n\ts_cselect_b32 %0, %1, 0" : "=s"(slot) : "s"(nslot), "i"(NS * STG) : "scc");
    if (!PF) MAX2(2, t - 1);
    MFMA(3, t);
    if (!PF) MAX2(3, t - 1);
    if (NM == 5) { MFMA(NM - 1, t); if (!PF) MAX2(NM - 1, t - 1); }
  }
  if (!PF) {
    asm volatile("s_nop 7\n\ts_nop 7" ::: "memory");
#pragma unroll
    for (int m = 0; m < NM; ++m) MAX2(m, 7);
  }
}

template <int NM>
__device__ __forceinline__ void consumer(const Params& p, char* me, LAS char* lds, int lane, int wave, unsigned call,
                                         int gidx, int src, int h, int pn, int xj, unsigned long long t0) {
  f32x4 acc[NM][8];
#pragma unroll
  for (int m = 0; m < NM; ++m)
#pragma unroll
    for (int t = 0; t < 8; ++t) { acc[m][t] = f32x4{0.f, 0.f, 0.f, 0.f}; asm volatile("" : "+v"(acc[m][t])); }
  bf16x8 Wf[8], Xa[NM], Xb[NM];
  unsigned slot = STG, fpre, vdone = 1;
  float mx0 = 0.f, mx1 = 0.f;
  unsigned nslow = 0;
  const unsigned va0 = lane * 16, vx0 = lane * 16 + wave * 5120, vsy = SY + 4 * wave;
  unsigned va = va0, vx = vx0;
  while (rfl(lds_ld(SY_FULL)) == 0) __builtin_amdgcn_s_sleep(1);
  STAMP(8 + wave);
#pragma unroll
  for (int t = 0; t < 7; ++t) LDR(Wf[t], va, WOF + t * 1024);
#pragma unroll
  for (int t = 0; t < NM; ++t) LDR(Xa[t], vx, t * 1024);
  asm volatile("ds_read_b32 %0, %1 offset:4\n\ts_waitcnt lgkmcnt(0)" : "=v"(fpre) : "v"(SY_FULL) : "memory");
  for (int s = 0; s < NSTEP - 2; s += 2) {
    mstep<NM, 0, 0>(acc, Wf, Xa, Xb, va, vx, va0, vx0, vsy, vdone, slot, fpre, s, mx0, mx1, nslow);
    mstep<NM, 0, 1>(acc, Wf, Xb, Xa, va, vx, va0, vx0, vsy, vdone, slot, fpre, s + 1, mx0, mx1, nslow);
  }
  mstep<NM, 1, 0>(acc, Wf, Xa, Xb, va, vx, va0, vx0, vsy, vdone, slot, fpre, NSTEP - 2, mx0, mx1, nslow);
  mstep<NM, 2, 1>(acc, Wf, Xb, Xa, va, vx, va0, vx0, vsy, vdone, slot, fpre, NSTEP - 1, mx0, mx1, nslow);
  STAMP(12 + wave);
  unsigned wm = wave_umax(umax(__builtin_bit_cast(unsigned, mx0), __builtin_bit_cast(unsigned, mx1)));
  asm volatile("ds_max_u32 %0, %1" :: "v"(SY_WMAX), "v"(wm) : "memory");
  unsigned old = 0;
  if (lane == 0) old = lds_add_rtn(SY_ARR + 8, 1u);
  if (rfl(old) == 3) {
    unsigned m4 = lds_ld(SY_WMAX);
    if (lane < NR) {
      char* g = (char*)*(LAS unsigned long long*)(lds + SY_PTR + lane * 8) + GR_OFF + (size_t)(((call & 1) * NR + p.rank) * 256 + gidx) * 8;
      asm volatile("global_store_dwordx2 %0, %1, off sc0 sc1\n\ts_nop 1" :: "v"(g), "v"(u32x2{m4, call}) : "memory");
    }
  }
#pragma unroll
  for (int m = 0; m < NM; ++m)
#pragma unroll
    for (int t = 0; t < 8; ++t)
#pragma unroll
      for (int j = 0; j < 4; ++j) asm volatile("v_max_f32 %0, 0, %0" : "+v"(acc[m][t][j]));
  bool bad = false;
  float gmax;
  {
    const char* gp = me + GM_OFF + (blockIdx.x & 7) * 128;
    const char* tm = me + TMO_OFF;
    for (;;) {
      u32x2 g; unsigned t;
      asm volatile("global_load_dwordx2 %0, %2, %3 sc0 sc1\n\tglobal_load_dword %1, %2, %4 sc0 sc1\n\ts_waitcnt vmcnt(0)"
                   : "=&v"(g), "=&v"(t) : "v"(0), "s"(gp), "s"(tm) : "memory");
      gmax = __builtin_bit_cast(float, rfl(g[0]));
      if (rfl(t) == call) { bad = true; break; }
      if (rfl(g[1]) == call) break;
      if (expired(t, call, t0, me)) { bad = true; break; }
      __builtin_amdgcn_s_sleep(1);
    }
  }
  float inv = gmax > 0.f ? 127.0f / gmax : 0.f;
  STAMP(16 + wave);
  char* ob = p.out + pn * 128 + 16 * (lane >> 4);
#pragma unroll
  for (int m = 0; m < NM; ++m) {
    int row = (m < 4 ? src * 512 + 256 * h + 64 * wave + 16 * m : p.rank * 512 + 256 * h + 64 * xj + 16 * wave) + (lane & 15);
#pragma unroll
    for (int hf = 0; hf < 2; ++hf) {
      u32x4 o;
#pragma unroll
      for (int j = 0; j < 4; ++j) {
        unsigned q[4];
#pragma unroll
        for (int c = 0; c < 4; ++c) q[c] = __builtin_bit_cast(unsigned, __builtin_fmaf(acc[m][4 * hf + c][j], inv, 8388608.0f));
        o[j] = perm((q[3] << 8) | q[2], (q[1] << 8) | q[0], 0x05040100u);
      }
      if (bad) o = u32x4{0x80808080u, 0x80808080u, 0x80808080u, 0x80808080u};
      *(u32x4*)(ob + (size_t)row * NLOC + 64 * hf) = o;
    }
  }
  STAMP(20 + wave);
}

__global__ __launch_bounds__(512, 2) void fused(Params p) {
  extern __shared__ char smem[];
  LAS char* lds = (LAS char*)smem;
  const int lane = threadIdx.x & 63;
  const int wave = __builtin_amdgcn_readfirstlane(threadIdx.x >> 6);
  const int b = blockIdx.x;
  const int rank = p.rank;
  char* me = p.ws[rank];
  const char* tmo = me + TMO_OFF;
  if (wave == 0) STAMP(0);

  f32x4 xv[8];
  const int us = b >> 1, uh = b & 1;
  if (wave < 4) {
    const float* xp = p.x + (size_t)(256 * uh + 64 * wave + (lane >> 3)) * KD + 32 * us + 4 * (lane & 7);
#pragma unroll
    for (int i = 0; i < 8; ++i) xv[i] = __builtin_nontemporal_load((const f32x4*)(xp + (size_t)i * 8 * KD));
  } else if (wave == 4) {
    if (lane < 32) lds_st(SY + lane * 4, 0u);
  } else if (wave == 5) {
    if (lane < NR) *(LAS unsigned long long*)(lds + SY_PTR + lane * 8) = (unsigned long long)p.ws[lane];
  }
  unsigned* cntp = (unsigned*)(me + CNT_OFF + b * 64);
  const unsigned call = rfl(__hip_atomic_load(cntp, __ATOMIC_RELAXED, __HIP_MEMORY_SCOPE_SYSTEM)) + 1u;
  const int xcd = b & 7, idx = (b >> 3) - NCOMM / 8;
  const int h = xcd & 1, pn = (xcd >> 1) * 4 + (idx & 3), di = idx >> 2, wj = 2 * di + h;
  const bool extra = di < 4;
  if (b >= NCOMM && wave >= 6) {
    const int r = wj + 14 * (wave - 6);
    convert_w(p, me, pn, r + r / 7, lane, call);
    if (wave == 6) STAMP(2);
  }
  if (wave < 4) {
    float mx = 0.f;
    u32x4 D[4];
    {
      LAS char* tw = lds + wave * 4096 + (lane >> 3) * 16 + ((lane & 7) >> 1) * 256 + (lane & 1) * 8;
#pragma unroll
      for (int i = 0; i < 8; ++i) {
        mx = fmaxf(fmaxf(mx, fmaxf(fabsf(xv[i][0]), fabsf(xv[i][1]))), fmaxf(fabsf(xv[i][2]), fabsf(xv[i][3])));
        *(LAS u32x2*)(tw + (i >> 1) * 1024 + (i & 1) * 128) = u32x2{pk_bf16(xv[i][0], xv[i][1]), pk_bf16(xv[i][2], xv[i][3])};
      }
      if (wave == 0) STAMP(56);
      cfence();
#pragma unroll
      for (int f = 0; f < 4; ++f) D[f] = *(LAS u32x4*)(lds + wave * 4096 + f * 1024 + lane * 16);
    }
    unsigned hi7 = wave_umax((pk_bf16(mx, 0.f) >> 8) & 0x7fu);
    unsigned base = hi7 > 7u ? hi7 - 7u : 0u;
    unsigned B4 = base * 0x01010101u;
    unsigned L[8], T[8], Hr[8], any = 0;
#pragma unroll
    for (int i = 0; i < 8; ++i) {
      unsigned d0 = D[i >> 1][2 * (i & 1)], d1 = D[i >> 1][2 * (i & 1) + 1];
      L[i] = perm(d1, d0, 0x06040200u);
      Hr[i] = perm(d1, d0, 0x07050301u);
      T[i] = (Hr[i] | 0x80808080u) - B4;
      any |= T[i] ^ 0x80808080u;
    }
    u32x4 N;
#pragma unroll
    for (int f = 0; f < 4; ++f) {
      unsigned c1 = (T[2 * f] & 0x07070707u) | (Hr[2 * f] & 0x80808080u);
      unsigned c2 = (T[2 * f + 1] & 0x07070707u) | (Hr[2 * f + 1] & 0x80808080u);
      N[f] = c1 | rotr(c2, 28);
    }
    char* pk = me + PK_OFF + (size_t)b * PKU;
    st16(pk + wave * 3072 + lane * 16, u32x4{L[0], L[1], L[2], L[3]});
    st16(pk + wave * 3072 + 1024 + lane * 16, u32x4{L[4], L[5], L[6], L[7]});
    st16(pk + wave * 3072 + 2048 + lane * 16, N);
    char* hdr = pk + 12288 + wave * 32;
    unsigned cnt = 0;
    if (__builtin_amdgcn_ballot_w64((any & 0xF8F8F8F8u) != 0)) {
#pragma unroll
      for (int i = 0; i < 8; ++i) {
        unsigned E = (T[i] ^ 0x80808080u) & 0xF8F8F8F8u;
        if (__builtin_amdgcn_ballot_w64(E != 0)) {
#pragma unroll
          for (int by = 0; by < 4; ++by) {
            bool fl = ((E >> (8 * by)) & 0xffu) != 0;
            unsigned long long mk = __builtin_amdgcn_ballot_w64(fl);
            unsigned idx = cnt + __builtin_amdgcn_mbcnt_hi((unsigned)(mk >> 32), __builtin_amdgcn_mbcnt_lo((unsigned)mk, 0u));
            if (fl && idx < 7u) st4(hdr + 4 + 4 * idx, ((Hr[i] >> (8 * by)) & 0xffu) | (lane << 8) | (by << 14) | (i << 16));
            cnt += (unsigned)__builtin_popcountll(mk);
          }
        }
      }
    }
    if (cnt > 7u || FORCE_RAW) cnt = 255u;
    if (lane == 0) st4(hdr, base | (cnt << 8));
    char* xl = me + XB_OFF + (size_t)((rank * 2 + uh) * NSTEP + us) * XSTG + wave * 4096 + lane * 16;
#pragma unroll
    for (int f = 0; f < 4; ++f) st16(xl + f * 1024, D[f]);
    if (wave == 0) STAMP(57);
    waitv<0>();
  }
  asm volatile("s_waitcnt lgkmcnt(0)" ::: "memory");
  __builtin_amdgcn_s_barrier();
  cfence();
  if (wave == 0) {
    if (lane < NR) {
      char* f = lane == rank ? me + XF_OFF + (size_t)((rank * 2 + uh) * NSTEP + us) * 4
                             : (char*)*(LAS unsigned long long*)(lds + SY_PTR + lane * 8) + SF_OFF + (size_t)(rank * 256 + b) * 4;
      st4(f, call);
    }
    STAMP(1);
  }
  if (wave == 7 && lane == 0) __hip_atomic_store(cntp, call, __ATOMIC_RELAXED, __HIP_MEMORY_SCOPE_SYSTEM);
  const unsigned long long t0 = rt();

  if (b < NPULL) {
    const int src = (rank + 1 + b % 7) & 7, pp = b / 7;
    const char* sws = p.ws[src];
    const char* sf = me + SF_OFF + src * 1024 + lane * 16;
    bool allset = false;
    const unsigned v0 = lane * 16, v1 = v0 + 4096, v2 = v0 + 8192, vh = 12288 + (lane & 31) * 4;
    bool dead = false;
    for (; wave < NWP;) {
      unsigned tk = 0;
      if (lane == 0) tk = lds_add_rtn(SY_TKT, 1u);
      const int n = (int)rfl(tk);
      if (n >= 64) break;
      const int u = n * 4 + pp, s = u >> 1, h = u & 1;
      if (!allset) {
        for (;;) {
          u32x4 f; unsigned t;
          poll16(f, t, sf, tmo);
          allset = __builtin_amdgcn_ballot_w64(umin(umin(f[0], f[1]), umin(f[2], f[3])) >= call) == ~0ull;
          unsigned mine = pp == 0 ? f[0] : pp == 1 ? f[1] : pp == 2 ? f[2] : f[3];
          if ((unsigned)__builtin_amdgcn_readlane(mine, n) >= call) break;
          if (expired(t, call, t0, me)) { dead = true; break; }
          __builtin_amdgcn_s_sleep(2);
        }
        if (dead) { lds_st(SY_TKT, 0x10000u); break; }
        if (n < 8) STAMP(8 + n);
      }
      u32x4 A[12]; unsigned hd;
      const char* sb = sws + PK_OFF + (size_t)u * PKU;
      asm volatile("s_nop 4\n\t"
                   "global_load_dword %12, %16, %17 sc0 sc1\n\t"
                   "global_load_dwordx4 %0, %13, %17 sc0 sc1\n\t"
                   "global_load_dwordx4 %1, %13, %17 offset:1024 sc0 sc1\n\t"
                   "global_load_dwordx4 %2, %13, %17 offset:2048 sc0 sc1\n\t"
                   "global_load_dwordx4 %3, %13, %17 offset:3072 sc0 sc1\n\t"
                   "global_load_dwordx4 %4, %14, %17 sc0 sc1\n\t"
                   "global_load_dwordx4 %5, %14, %17 offset:1024 sc0 sc1\n\t"
                   "global_load_dwordx4 %6, %14, %17 offset:2048 sc0 sc1\n\t"
                   "global_load_dwordx4 %7, %14, %17 offset:3072 sc0 sc1\n\t"
                   "global_load_dwordx4 %8, %15, %17 sc0 sc1\n\t"
                   "global_load_dwordx4 %9, %15, %17 offset:1024 sc0 sc1\n\t"
                   "global_load_dwordx4 %10, %15, %17 offset:2048 sc0 sc1\n\t"
                   "global_load_dwordx4 %11, %15, %17 offset:3072 sc0 sc1\n\t"
                   "s_waitcnt vmcnt(0)"
                   : "=&v"(A[0]), "=&v"(A[1]), "=&v"(A[2]), "=&v"(A[3]), "=&v"(A[4]), "=&v"(A[5]), "=&v"(A[6]), "=&v"(A[7]),
                     "=&v"(A[8]), "=&v"(A[9]), "=&v"(A[10]), "=&v"(A[11]), "=&v"(hd)
                   : "v"(v0), "v"(v1), "v"(v2), "v"(vh), "s"(sb) : "memory");
      const size_t xo = XB_OFF + (size_t)((src * 2 + h) * NSTEP + s) * XSTG;
      char* xd = me + xo + lane * 16;
#pragma unroll
      for (int q = 0; q < 4; ++q) {
        const unsigned h0 = __builtin_amdgcn_readlane(hd, 8 * q);
        const unsigned cnt = (h0 >> 8) & 0xffu;
        if (cnt > 7u) {
          u32x4 R0, R1, R2, R3;
          const char* rb = sws + xo + q * 4096;
          asm volatile("s_nop 4\n\t"
                       "global_load_dwordx4 %0, %4, %5 sc0 sc1\n\t"
                       "global_load_dwordx4 %1, %4, %5 offset:1024 sc0 sc1\n\t"
                       "global_load_dwordx4 %2, %4, %5 offset:2048 sc0 sc1\n\t"
                       "global_load_dwordx4 %3, %4, %5 offset:3072 sc0 sc1\n\t"
                       "s_waitcnt vmcnt(0)"
                       : "=&v"(R0), "=&v"(R1), "=&v"(R2), "=&v"(R3) : "v"(v0), "s"(rb) : "memory");
          st16(xd + q * 4096, R0); st16(xd + q * 4096 + 1024, R1); st16(xd + q * 4096 + 2048, R2); st16(xd + q * 4096 + 3072, R3);
        } else {
          const unsigned B4 = (h0 & 0x7fu) * 0x01010101u;
          unsigned H[8];
#pragma unroll
          for (int f = 0; f < 4; ++f) {
            H[2 * f] = (A[3 * q + 2][f] & 0x87878787u) + B4;
            H[2 * f + 1] = (rotr(A[3 * q + 2][f], 4) & 0x87878787u) + B4;
          }
          for (unsigned e = 1; e <= cnt; ++e) {
            const unsigned ent = __builtin_amdgcn_readlane(hd, 8 * q + e);
            const unsigned sh = ((ent >> 14) & 3u) * 8u, wi = (ent >> 16) & 7u;
            const unsigned msk = (unsigned)lane == ((ent >> 8) & 63u) ? 0xffu << sh : 0u, val = (ent & 0xffu) << sh;
#pragma unroll
            for (int i = 0; i < 8; ++i) { unsigned mi = wi == (unsigned)i ? msk : 0u; H[i] = (H[i] & ~mi) | (val & mi); }
          }
#pragma unroll
          for (int f = 0; f < 4; ++f) {
            const unsigned l0 = A[3 * q + (f >> 1)][2 * (f & 1)], l1 = A[3 * q + (f >> 1)][2 * (f & 1) + 1];
            st16(xd + q * 4096 + f * 1024, u32x4{perm(H[2 * f], l0, 0x05010400u), perm(H[2 * f], l0, 0x07030602u),
                                                 perm(H[2 * f + 1], l1, 0x05010400u), perm(H[2 * f + 1], l1, 0x07030602u)});
          }
        }
      }
      waitv<0>();
      if (lane == 0) flag_st(me + XF_OFF + (size_t)((src * 2 + h) * NSTEP + s) * 4, call);
      if (n < 8) STAMP(16 + n);
      if ((n & 7) == 7) STAMP(48 + n / 8);
    }
    if (wave < NWP) STAMP(24 + wave);
    if (b >= 8) return;
    if (wave >= NWP) {
      while (rfl(lds_ld(SY_TKT)) < 64u) __builtin_amdgcn_s_sleep(32);
    }
    const int l56 = lane < 56 ? lane : 55;
    const char* g0 = me + GR_OFF + (size_t)(((call & 1) * NR + wave) * 256) * 8 + l56 * 32;
    for (;;) {
      u32x4 a0, a1; unsigned t;
      asm volatile("global_load_dwordx4 %0, %3, off sc0 sc1\n\tglobal_load_dwordx4 %1, %3, off offset:16 sc0 sc1\n\t"
                   "global_load_dword %2, %4, off sc0 sc1\n\ts_waitcnt vmcnt(0)"
                   : "=&v"(a0), "=&v"(a1), "=&v"(t) : "v"(g0), "v"(tmo) : "memory");
      const bool ok = a0[1] == call && a0[3] == call && a1[1] == call && a1[3] == call;
      if (__builtin_amdgcn_ballot_w64(ok) == ~0ull) {
        const unsigned best = wave_umax(umax(umax(a0[0], a0[2]), umax(a1[0], a1[2])));
        asm volatile("ds_max_u32 %0, %1" :: "v"(SY_GMAX), "v"(best) : "memory");
        unsigned old = 0;
        if (lane == 0) old = lds_add_rtn(SY_ARR, 1u);
        if (rfl(old) == 7u) {
          const unsigned gm = lds_ld(SY_GMAX);
          if (lane == 0)
            __hip_atomic_store((unsigned long long*)(me + GM_OFF + b * 128), ((unsigned long long)call << 32) | gm,
                               __ATOMIC_RELAXED, __HIP_MEMORY_SCOPE_SYSTEM);
          STAMP(4);
        }
        break;
      }
      if (expired(t, call, t0, me)) break;
      __builtin_amdgcn_s_sleep(1);
    }
    return;
  }
  if (b < NCOMM) {
    for (int r = 0; r < 8; ++r) {
      const int id = r * 32 + (b - NPULL) * 8 + wave;
      convert_w(p, me, id & 15, (id >> 4) * 8 + 7, lane, call);
    }
    if (wave == 0) STAMP(2);
    return;
  }

  const int src = (rank + 1 + di) & 7;
  if (wave < 4) {
    __builtin_amdgcn_s_setprio(PRIOC);
    if (extra) consumer<5>(p, me, lds, lane, wave, call, b - NCOMM, src, h, pn, di, t0);
    else consumer<4>(p, me, lds, lane, wave, call, b - NCOMM, src, h, pn, di, t0);
    return;
  }
  bool dead = false;
  if (wave < 6) {
    __builtin_amdgcn_s_setprio(PRIOL);
    const char* xs = me + XB_OFF + (size_t)((src * 2 + h) * NSTEP) * XSTG;
    const char* ls = me + XB_OFF + (size_t)((rank * 2 + h) * NSTEP) * XSTG + di * 4096;
    const char* wsr = me + WB_OFF + (size_t)(pn * NSTEP) * WSTG;
    const unsigned voff = lane * 16;
    unsigned slot = (wave - 4) * STG;
    const int fw = SY_FULL + 4 * (wave - 4);
    unsigned pend = 0;
    for (int s = wave - 4; s < NSTEP; s += 2) {
      unsigned rdy;
      for (;;) {
        u32x4 d; unsigned r;
        asm volatile("ds_read_b128 %0, %2\n\tds_read_b32 %1, %3\n\ts_waitcnt lgkmcnt(0)" : "=&v"(d), "=&v"(r) : "v"(SY_DONE), "v"(SY_READY) : "memory");
        rdy = rfl(r);
        unsigned dm = rfl(umin(umin(d[0], d[1]), umin(d[2], d[3])));
        if (rdy >= ABORT || (rdy > (unsigned)s && dm + NS > (unsigned)s)) break;
        if (pend) {
          waitv<0>(); lds_st(fw, pend); pend = 0;
        } else __builtin_amdgcn_s_sleep(SLEEPL);
      }
      if (rdy >= ABORT) { pend = ABORT; break; }
      const char* xa = xs + (size_t)s * XSTG;
      glds4(voff, wsr + (size_t)s * WSTG, slot + WOF); glds4(voff, wsr + (size_t)s * WSTG + 4096, slot + WOF + 4096);
      glds4(voff, xa, slot); glds4(voff, xa + 4096, slot + 5120); glds4(voff, xa + 8192, slot + 10240); glds4(voff, xa + 12288, slot + 15360);
      if (extra) { glds4s(voff, ls + (size_t)s * XSTG, slot + 4096); waitv<28>(); }
      else waitv<24>();
      if (pend) lds_st(fw, pend);
      pend = s + 1;
      slot += 2 * STG; if (slot >= NS * STG) slot -= NS * STG;
    }
    waitv<0>();
    lds_st(fw, pend);
    __builtin_amdgcn_s_setprio(0);
  } else if (wave == 6) {
    __builtin_amdgcn_s_setprio(PRIOP);
    if (extra) {
      const char* lf = me + XF_OFF + (size_t)((rank * 2 + h) * NSTEP) * 4 + (lane & 31) * 16;
      for (;;) {
        u32x4 f; unsigned t;
        poll16(f, t, lf, tmo);
        if (__builtin_amdgcn_ballot_w64(umin(umin(f[0], f[1]), umin(f[2], f[3])) >= call) == ~0ull) break;
        if (expired(t, call, t0, me)) { dead = true; break; }
        __builtin_amdgcn_s_sleep(2);
      }
    }
    const char* pa = lane < 32 ? me + XF_OFF + (size_t)((src * 2 + h) * NSTEP) * 4 + lane * 16
                               : me + WF_OFF + (size_t)(pn * NSTEP) * 4 + (lane & 31) * 16;
    unsigned ready = 0;
    while (!dead && ready < (unsigned)NSTEP) {
      if (ready > rfl(lds_ld(SY_DONE)) + NS + 2) { __builtin_amdgcn_s_sleep(SLEEPP); continue; }
      u32x4 f; unsigned t;
      poll16(f, t, pa, tmo);
      unsigned c = f[0] < call ? 0u : f[1] < call ? 1u : f[2] < call ? 2u : f[3] < call ? 3u : 4u;
      unsigned long long part = __builtin_amdgcn_ballot_w64(c < 4u);
      unsigned px = (unsigned)part, pw = (unsigned)(part >> 32);
      unsigned rx = NSTEP, rw = NSTEP;
      if (px) { unsigned l = __builtin_ctz(px); rx = 4 * l + __builtin_amdgcn_readlane(c, l); }
      if (pw) { unsigned l = __builtin_ctz(pw); rw = 4 * l + __builtin_amdgcn_readlane(c, 32 + l); }
      unsigned r = umin(rx, rw);
      if (r > ready) {
        ready = r; lds_st(SY_READY, ready); continue; }
      if (expired(t, call, t0, me)) { dead = true; break; }
      __builtin_amdgcn_s_sleep(1);
    }
    if (dead) lds_st(SY_READY, ABORT);
    STAMP(3);
  } else {
    for (int i = 2; i < 8; ++i) {
      const int r = wj + 14 * i;
      while (rfl(lds_ld(SY_DONE)) < (unsigned)(16 * (i - 2) + WLEAD + wj)) __builtin_amdgcn_s_sleep(16);
      convert_w(p, me, pn, r + r / 7, lane, call);
    }
  }
}

}

extern "C" void kernel_launch(int rank, int nranks, ncclComm_t comm, hipStream_t stream,
                              void* const* d_in, const int* in_sizes, int n_in,
                              void* d_out, int out_size, const int* in_dtypes, int out_dtype,
                              void* const* d_out_all, void* const* d_ws_all, size_t ws_bytes) {
  if (nranks != NR || (unsigned)rank >= (unsigned)NR || n_in < 2 || ws_bytes < WS_NEED ||
      in_sizes[0] != 512 * KD || out_size != 4096 * NLOC) {
    fprintf(stderr, "kernel_launch: unexpected problem shape\n");
    return;
  }
  HIP_CHECK(hipFuncSetAttribute((const void*)fused, hipFuncAttributeMaxDynamicSharedMemorySize, LDS_BYTES));
  Params p;
  p.x = (const float*)d_in[0]; p.w = (const float*)d_in[1]; p.out = (char*)d_out; p.rank = rank;
  for (int i = 0; i < NR; ++i) p.ws[i] = (char*)d_ws_all[i];
  fused<<<NWG, 512, LDS_BYTES, stream>>>(p);
  HIP_CHECK(hipGetLastError());
}
```

```cpp
#include <hip/hip_runtime.h>
#include <rccl/rccl.h>
#include <cstdio>
#include <cstdint>

#define HIP_CHECK(expr) do { hipError_t _e = (expr); if (_e != hipSuccess) { \
    fprintf(stderr, "HIP error %s at %s:%d\n", hipGetErrorString(_e), __FILE__, __LINE__); \
    return; } } while (0)

namespace {

typedef unsigned u32x4 __attribute__((ext_vector_type(4)));
typedef unsigned u32x3 __attribute__((ext_vector_type(3)));
typedef unsigned u32x2 __attribute__((ext_vector_type(2)));
typedef unsigned short u16x2 __attribute__((ext_vector_type(2)));
typedef float f32x4 __attribute__((ext_vector_type(4)));
typedef float f32x2 __attribute__((ext_vector_type(2)));
typedef __bf16 bf16x8 __attribute__((ext_vector_type(8)));
typedef __bf16 bf16x2 __attribute__((ext_vector_type(2)));
#define LAS __attribute__((address_space(3)))

#define FORCE_RAW 0
#define UPF 1
#define PRIOC 3
#define PRIOL 0
#define PRIOP 0
#define SLEEPL 1
#define SLEEPP 4
#define STAG 8
#define NWP 6

constexpr int NR = 8;
constexpr int KD = 4096, ND = 16384, NLOC = 2048;
constexpr int NSTEP = 128;
constexpr int NPULL = 28, NCOMM = 48, NCONV = NCOMM - NPULL, NGEMM = 208, NWG = 256;
constexpr int XSTG = 16384, WSTG = 8192;
constexpr int PK_C = 8192, PK_E = 11264, PK_R = 192, PKU = PK_E + 2 * PK_R;
constexpr unsigned EXC = (PK_R - 8) / 2;
constexpr size_t XB_OFF = 0;
constexpr size_t WB_OFF = 32u << 20;
constexpr size_t PK_OFF = 48u << 20;
constexpr size_t FL_OFF = 64u << 20;
constexpr size_t CNT_OFF = FL_OFF;
constexpr size_t TMO_OFF = FL_OFF + 0x4000;
constexpr size_t SF_OFF = FL_OFF + 0x5000;
constexpr size_t XF_OFF = FL_OFF + 0x8000;
constexpr size_t WF_OFF = FL_OFF + 0xA000;
constexpr size_t GM_OFF = FL_OFF + 0xC000;
constexpr size_t TK_OFF = FL_OFF + 0xD000;
constexpr size_t GR_OFF = FL_OFF + 0x10000;
constexpr size_t ST_OFF = FL_OFF + 0x20000;
constexpr size_t WS_NEED = FL_OFF + 0x40000;
constexpr int NS = 5;
constexpr int STG = 28672;
constexpr int SY = NS * STG;
constexpr int SY_FULL = SY, SY_READY = SY + 8, SY_DONE = SY + 16, SY_ARR = SY + 32, SY_WMAX = SY + 48,
              SY_GMAX = SY + 64, SY_TKT = SY + 84, SY_PACK = SY + 96, SY_PTR = SY + 128;
constexpr int LDS_BYTES = SY + 192;
constexpr unsigned long long DEADLINE = 50000000ull;
constexpr unsigned ABORT = 0x40000000u;

struct Params { const float* x; const float* w; char* out; char* ws[NR]; int rank; };

__device__ __forceinline__ unsigned long long rt() { return __builtin_amdgcn_s_memrealtime(); }
template <int N_> __device__ __forceinline__ void waitv() { asm volatile("s_waitcnt vmcnt(%0)" :: "i"(N_) : "memory"); }
__device__ __forceinline__ void cfence() { asm volatile("" ::: "memory"); }
__device__ __forceinline__ void st16(char* p, u32x4 v) {
  asm volatile("global_store_dwordx4 %0, %1, off sc0 sc1\n\ts_nop 1" :: "v"(p), "v"(v) : "memory");
}
__device__ __forceinline__ void st4(char* p, unsigned v) {
  asm volatile("global_store_dword %0, %1, off sc0 sc1" :: "v"(p), "v"(v) : "memory");
}
__device__ __forceinline__ void flag_st(const char* p, unsigned v) {
  asm volatile("s_mov_b64 exec, 1\n\ts_nop 3\n\tglobal_store_dword %0, %1, %2 sc0 sc1\n\ts_mov_b64 exec, -1" :: "v"(0), "v"(v), "s"(p) : "memory");
}
__device__ __forceinline__ unsigned lds_ticket(int addr) {
  unsigned v, r;
  asm volatile("s_mov_b64 exec, 1\n\tds_add_rtn_u32 %0, %2, %3\n\ts_mov_b64 exec, -1\n\ts_waitcnt lgkmcnt(0)\n\tv_readfirstlane_b32 %1, %0"
               : "=&v"(v), "=s"(r) : "v"(addr), "v"(1u) : "memory");
  return r;
}
__device__ __forceinline__ void poll16(u32x4& a, unsigned& t, const char* p, const char* tm) {
  asm volatile("global_load_dwordx4 %0, %2, off sc0 sc1\n\tglobal_load_dword %1, %3, off sc0 sc1\n\ts_waitcnt vmcnt(0)"
               : "=&v"(a), "=&v"(t) : "v"(p), "v"(tm) : "memory");
}
__device__ __forceinline__ void lds_st(int addr, unsigned v) {
  asm volatile("ds_write_b32 %0, %1" :: "v"(addr), "v"(v) : "memory");
}
__device__ __forceinline__ unsigned lds_ld(int addr) {
  unsigned v; asm volatile("ds_read_b32 %0, %1\n\ts_waitcnt lgkmcnt(0)" : "=&v"(v) : "v"(addr) : "memory"); return v;
}
__device__ __forceinline__ unsigned rfl(unsigned v) { return __builtin_amdgcn_readfirstlane(v); }
__device__ __forceinline__ unsigned umax(unsigned a, unsigned b) { return a > b ? a : b; }
__device__ __forceinline__ unsigned umin(unsigned a, unsigned b) { return a < b ? a : b; }
__device__ __forceinline__ unsigned wave_umax(unsigned v) {
  v = umax(v, (unsigned)__builtin_amdgcn_update_dpp(0, (int)v, 0x111, 0xf, 0xf, false));
  v = umax(v, (unsigned)__builtin_amdgcn_update_dpp(0, (int)v, 0x112, 0xf, 0xf, false));
  v = umax(v, (unsigned)__builtin_amdgcn_update_dpp(0, (int)v, 0x114, 0xf, 0xf, false));
  v = umax(v, (unsigned)__builtin_amdgcn_update_dpp(0, (int)v, 0x118, 0xf, 0xf, false));
  unsigned a = __builtin_amdgcn_readlane(v, 15), b = __builtin_amdgcn_readlane(v, 31);
  unsigned c = __builtin_amdgcn_readlane(v, 47), d = __builtin_amdgcn_readlane(v, 63);
  return umax(umax(a, b), umax(c, d));
}
__device__ __forceinline__ unsigned wave_fsum(float v) {
#define DPPADD(ctrl, rm) v += __builtin_bit_cast(float, __builtin_amdgcn_update_dpp(0, __builtin_bit_cast(int, v), ctrl, rm, 0xf, false))
  DPPADD(0x111, 0xf); DPPADD(0x112, 0xf); DPPADD(0x114, 0xf); DPPADD(0x118, 0xf); DPPADD(0x142, 0xa); DPPADD(0x143, 0xc);
  return __builtin_amdgcn_readlane(__builtin_bit_cast(unsigned, v), 63);
}
__device__ __forceinline__ unsigned bfi(unsigned m, unsigned a, unsigned b) { return (a & m) | (b & ~m); }
__device__ __forceinline__ unsigned pk_sub16(unsigned a, unsigned b) {
  return __builtin_bit_cast(unsigned, (u16x2)(__builtin_bit_cast(u16x2, a) - __builtin_bit_cast(u16x2, b)));
}
__device__ __forceinline__ unsigned pk_min16(unsigned a, unsigned b) {
  return __builtin_bit_cast(unsigned, __builtin_elementwise_min(__builtin_bit_cast(u16x2, a), __builtin_bit_cast(u16x2, b)));
}
__device__ __forceinline__ unsigned pk_bf16(float lo, float hi) {
  f32x2 f = {lo, hi};
  return __builtin_bit_cast(unsigned, __builtin_convertvector(f, bf16x2));
}
__device__ __forceinline__ unsigned perm(unsigned hi, unsigned lo, unsigned sel) { return __builtin_amdgcn_perm(hi, lo, sel); }
__device__ __forceinline__ unsigned rotr(unsigned v, unsigned n) { return __builtin_amdgcn_alignbit(v, v, n); }
__device__ __forceinline__ void glds4(unsigned voff, const char* sbase, unsigned dst) {
  asm volatile("s_mov_b32 m0, %2\n\ts_nop 0\n\t"
               "global_load_lds_dwordx4 %0, %1 sc0 sc1\n\t"
               "global_load_lds_dwordx4 %0, %1 offset:1024 sc0 sc1\n\t"
               "global_load_lds_dwordx4 %0, %1 offset:2048 sc0 sc1\n\t"
               "global_load_lds_dwordx4 %0, %1 offset:3072 sc0 sc1"
               :: "v"(voff), "s"(sbase), "s"(dst) : "memory");
}
__device__ __forceinline__ void glds4s(unsigned voff, const char* sbase, unsigned dst) {
  asm volatile("s_mov_b32 m0, %2\n\ts_nop 0\n\t"
               "global_load_lds_dwordx4 %0, %1 sc0 sc1\n\t"
               "s_add_u32 m0, m0, 4096\n\ts_nop 0\n\t"
               "global_load_lds_dwordx4 %0, %1 offset:1024 sc0 sc1\n\t"
               "s_add_u32 m0, m0, 4096\n\ts_nop 0\n\t"
               "global_load_lds_dwordx4 %0, %1 offset:2048 sc0 sc1\n\t"
               "s_add_u32 m0, m0, 4096\n\ts_nop 0\n\t"
               "global_load_lds_dwordx4 %0, %1 offset:3072 sc0 sc1"
               :: "v"(voff), "s"(sbase), "s"(dst) : "memory", "scc");
}
#define STAMP(i) do {} while (0)

__device__ __forceinline__ bool expired(unsigned tmo_seen, unsigned call, unsigned long long t0, char* me) {
  if (rfl(tmo_seen) == call) return true;
  if (rt() - t0 > DEADLINE) {
    __hip_atomic_store((unsigned*)(me + TMO_OFF), call, __ATOMIC_RELAXED, __HIP_MEMORY_SCOPE_AGENT);
    return true;
  }
  return false;
}

struct WRegs { f32x4 v[2][8]; };
__device__ __forceinline__ void w_load(WRegs& r, const Params& p, int panel, int s, int lane) {
  const float* wp = p.w + (size_t)(32 * s + 8 * (lane >> 4)) * ND + p.rank * NLOC + panel * 128 + 4 * (lane & 15);
#pragma unroll
  for (int hf = 0; hf < 2; ++hf)
#pragma unroll
    for (int j = 0; j < 8; ++j) r.v[hf][j] = __builtin_nontemporal_load((const f32x4*)(wp + (size_t)j * ND + hf * 64));
}
__device__ __forceinline__ void w_store(const WRegs& r, char* me, int panel, int s, int lane) {
  char* dst = me + WB_OFF + (size_t)(panel * NSTEP + s) * WSTG + lane * 16;
#pragma unroll
  for (int hf = 0; hf < 2; ++hf)
#pragma unroll
    for (int c = 0; c < 4; ++c) {
      u32x4 o = {pk_bf16(r.v[hf][0][c], r.v[hf][1][c]), pk_bf16(r.v[hf][2][c], r.v[hf][3][c]),
                 pk_bf16(r.v[hf][4][c], r.v[hf][5][c]), pk_bf16(r.v[hf][6][c], r.v[hf][7][c])};
      st16(dst + (4 * hf + c) * 1024, o);
    }
}
__device__ __forceinline__ void w_flag(char* me, int panel, int s, int lane, unsigned call) {
  waitv<0>();
  flag_st(me + WF_OFF + (size_t)(panel * NSTEP + s) * 4, call);
}
__device__ __forceinline__ void convert_w(const Params& p, char* me, int panel, int s, int lane, unsigned call) {
  WRegs r;
  w_load(r, p, panel, s, lane);
  w_store(r, me, panel, s, lane);
  w_flag(me, panel, s, lane, call);
}

#define MFMA(m, t) asm volatile("v_mfma_f32_16x16x32_bf16 %0, %1, %2, %0" : "+v"(acc[m][t]) : "v"(Wf[t]), "v"(Xc[m]))
#define LDR(dst, addr, off) asm volatile("ds_read_b128 %0, %1 offset:%2" : "=v"(dst) : "v"(addr), "i"(off) : "memory")
template <int N_> __device__ __forceinline__ void waitl() { asm volatile("s_waitcnt lgkmcnt(%0)" :: "i"(N_) : "memory"); }
constexpr int WOF = 20480;
#define MAX2(m, t) asm volatile("v_max3_f32 %0, %2, %3, %0\n\tv_max3_f32 %1, %4, %5, %1" : "+v"(mx0), "+v"(mx1) \
                                : "v"(acc[m][t][0]), "v"(acc[m][t][1]), "v"(acc[m][t][2]), "v"(acc[m][t][3]))
template <int NM, int MODE, int PAR>
__device__ __forceinline__ void mstep(f32x4 (&acc)[NM][8], bf16x8 (&Wf)[8], bf16x8 (&Xc)[NM], bf16x8 (&Xn)[NM],
                                      unsigned& va, unsigned& vx, unsigned va0, unsigned vx0, unsigned vsy, unsigned& vdone,
                                      unsigned& slot, unsigned& fpre, int s, float& mx0, float& mx1, unsigned& nslow) {
  constexpr bool PF = MODE < 2;
  unsigned sf = 0, nslot = 0;
  waitl<PF ? 3 : 2>();
  MFMA(0, 0);
  LDR(Wf[7], va, WOF + 7168);
  MFMA(1, 0);
  if (PF) asm volatile("ds_write_b32 %0, %1 offset:16" :: "v"(vsy), "v"(vdone) : "memory");
  MFMA(2, 0);
  if (PF) asm volatile("s_waitcnt lgkmcnt(3)\n\tv_readfirstlane_b32 %0, %1" : "=s"(sf) : "v"(fpre) : "memory");
  MFMA(3, 0);
  if (PF) asm volatile("v_add_u32 %0, %1, %2" : "=v"(va) : "s"(slot), "v"(va0));
  if (NM == 5) MFMA(NM - 1, 0);
  if (PF && __builtin_expect(sf <= (unsigned)(s + 1), 0)) {
    do __builtin_amdgcn_s_sleep(1); while (rfl(lds_ld(SY_FULL + 4 * (1 - PAR))) <= (unsigned)(s + 1));
  }
#pragma unroll
  for (int t = 1; t < 8; ++t) {
    if (t == 5 && !PF) waitl<2>();
    if (t == 6) waitl<PF ? NM + 6 : 0>();
    MFMA(0, t);
    if (PF) LDR(Wf[t - 1], va, WOF + (t - 1) * 1024); else MAX2(0, t - 1);
    MFMA(1, t);
    if (PF && t == 1) asm volatile("v_add_u32 %0, %1, %2" : "=v"(vx) : "s"(slot), "v"(vx0));
    if (PF && t == 2) asm volatile("v_add_u32 %0, 1, %0" : "+v"(vdone));
    if (MODE == 0 && t == 7) asm volatile("s_add_u32 %0, %1, %2" : "=s"(nslot) : "s"(slot), "i"(STG) : "scc");
    if (!PF) MAX2(1, t - 1);
    MFMA(2, t);
    if (PF && t <= NM) LDR(Xn[t - 1], vx, (t - 1) * 1024);
    if (MODE == 0 && t == 6) asm volatile("ds_read_b32 %0, %1 offset:%2" : "=v"(fpre) : "v"(SY_FULL), "i"(4 * PAR) : "memory");
    if (MODE == 0 && t == 7) asm volatile("s_cmp_lg_u32 %1, %2\n\ts_cselect_b32 %0, %1, 0" : "=s"(slot) : "s"(nslot), "i"(NS * STG) : "scc");
    if (!PF) MAX2(2, t - 1);
    MFMA(3, t);
    if (!PF) MAX2(3, t - 1);
    if (NM == 5) { MFMA(NM - 1, t); if (!PF) MAX2(NM - 1, t - 1); }
  }
  if (!PF) {
    asm volatile("s_nop 7\n\ts_nop 7" ::: "memory");
#pragma unroll
    for (int m = 0; m < NM; ++m) MAX2(m, 7);
  }
}

template <int NM>
__device__ __forceinline__ void consumer(const Params& p, char* me, LAS char* lds, int lane, int wave, unsigned call,
                                         int gidx, int row0, int row4, int pn, unsigned long long t0) {
  f32x4 acc[NM][8];
#pragma unroll
  for (int m = 0; m < NM; ++m)
#pragma unroll
    for (int t = 0; t < 8; ++t) { acc[m][t] = f32x4{0.f, 0.f, 0.f, 0.f}; asm volatile("" : "+v"(acc[m][t])); }
  bf16x8 Wf[8], Xa[NM], Xb[NM];
  unsigned slot = STG, fpre, vdone = 1;
  float mx0 = 0.f, mx1 = 0.f;
  unsigned nslow = 0;
  const unsigned va0 = lane * 16, vx0 = lane * 16 + wave * 5120, vsy = SY + 4 * wave;
  unsigned va = va0, vx = vx0;
  while (rfl(lds_ld(SY_FULL)) == 0) __builtin_amdgcn_s_sleep(1);
  STAMP(8 + wave);
#pragma unroll
  for (int t = 0; t < 7; ++t) LDR(Wf[t], va, WOF + t * 1024);
#pragma unroll
  for (int t = 0; t < NM; ++t) LDR(Xa[t], vx, t * 1024);
  asm volatile("ds_read_b32 %0, %1 offset:4\n\ts_waitcnt lgkmcnt(0)" : "=v"(fpre) : "v"(SY_FULL) : "memory");
  for (int s = 0; s < NSTEP - 2; s += 2) {
    mstep<NM, 0, 0>(acc, Wf, Xa, Xb, va, vx, va0, vx0, vsy, vdone, slot, fpre, s, mx0, mx1, nslow);
    mstep<NM, 0, 1>(acc, Wf, Xb, Xa, va, vx, va0, vx0, vsy, vdone, slot, fpre, s + 1, mx0, mx1, nslow);
  }
  mstep<NM, 1, 0>(acc, Wf, Xa, Xb, va, vx, va0, vx0, vsy, vdone, slot, fpre, NSTEP - 2, mx0, mx1, nslow);
  mstep<NM, 2, 1>(acc, Wf, Xb, Xa, va, vx, va0, vx0, vsy, vdone, slot, fpre, NSTEP - 1, mx0, mx1, nslow);
  STAMP(12 + wave);
  unsigned wm = wave_umax(umax(__builtin_bit_cast(unsigned, mx0), __builtin_bit_cast(unsigned, mx1)));
  asm volatile("ds_max_u32 %0, %1" :: "v"(SY_WMAX), "v"(wm) : "memory");
  if (lds_ticket(SY_ARR + 8) == 3) {
    unsigned m4 = lds_ld(SY_WMAX);
    if (lane < NR) {
      char* g = (char*)*(LAS unsigned long long*)(lds + SY_PTR + lane * 8) + GR_OFF + (size_t)(((call & 1) * NR + p.rank) * 256 + gidx) * 8;
      asm volatile("global_store_dwordx2 %0, %1, off sc0 sc1\n\ts_nop 1" :: "v"(g), "v"(u32x2{m4, call}) : "memory");
    }
  }
#pragma unroll
  for (int m = 0; m < NM; ++m)
#pragma unroll
    for (int t = 0; t < 8; ++t)
#pragma unroll
      for (int j = 0; j < 4; ++j) asm volatile("v_max_f32 %0, 0, %0" : "+v"(acc[m][t][j]));
  bool bad = false;
  float gmax;
  {
    const char* gp = me + GM_OFF + (blockIdx.x & 7) * 128;
    const char* tm = me + TMO_OFF;
    for (;;) {
      u32x2 g; unsigned t;
      asm volatile("global_load_dwordx2 %0, %2, %3 sc0 sc1\n\tglobal_load_dword %1, %2, %4 sc0 sc1\n\ts_waitcnt vmcnt(0)"
                   : "=&v"(g), "=&v"(t) : "v"(0), "s"(gp), "s"(tm) : "memory");
      gmax = __builtin_bit_cast(float, rfl(g[0]));
      if (rfl(t) == call) { bad = true; break; }
      if (rfl(g[1]) == call) break;
      if (expired(t, call, t0, me)) { bad = true; break; }
      __builtin_amdgcn_s_sleep(1);
    }
  }
  float inv = gmax > 0.f ? 127.0f / gmax : 0.f;
  STAMP(16 + wave);
  char* ob = p.out + pn * 128 + 16 * (lane >> 4);
#pragma unroll
  for (int m = 0; m < NM; ++m) {
    int row = (m < 4 ? row0 + 64 * wave + 16 * m : row4 + 16 * wave) + (lane & 15);
#pragma unroll
    for (int hf = 0; hf < 2; ++hf) {
      u32x4 o;
#pragma unroll
      for (int j = 0; j < 4; ++j) {
        unsigned q[4];
#pragma unroll
        for (int c = 0; c < 4; ++c) q[c] = __builtin_bit_cast(unsigned, __builtin_fmaf(acc[m][4 * hf + c][j], inv, 8388608.0f));
        o[j] = perm((q[3] << 8) | q[2], (q[1] << 8) | q[0], 0x05040100u);
      }
      if (bad) o = u32x4{0x80808080u, 0x80808080u, 0x80808080u, 0x80808080u};
      *(u32x4*)(ob + (size_t)row * NLOC + 64 * hf) = o;
    }
  }
  STAMP(20 + wave);
}

__global__ __launch_bounds__(512, 2) void fused(Params p) {
  extern __shared__ char smem[];
  LAS char* lds = (LAS char*)smem;
  const int lane = threadIdx.x & 63;
  const int wave = __builtin_amdgcn_readfirstlane(threadIdx.x >> 6);
  const int b = blockIdx.x;
  const int rank = p.rank;
  char* me = p.ws[rank];
  const char* tmo = me + TMO_OFF;
  if (wave == 0) STAMP(0);

  unsigned* cntp = (unsigned*)(me + CNT_OFF + b * 64);
  unsigned cnt0, cnt1;
  asm volatile("global_load_dword %0, %1, off sc0 sc1" : "=&v"(cnt0) : "v"(cntp) : "memory");
#define GETCNT(N) asm volatile("s_waitcnt vmcnt(" #N ")\n\tv_readfirstlane_b32 %0, %1" : "=s"(cnt1) : "v"(cnt0) : "memory")
  const int wt0 = b - NCOMM + NGEMM * (7 - wave);
  const bool wup = b >= NCOMM && wave >= 8 - UPF;
  f32x4 xv[8];
  WRegs w0;
  const int us = b >> 1, uh = b & 1;
  if (wave < 4) {
    const float* xp = p.x + (size_t)(256 * uh + 64 * wave + (lane >> 3)) * KD + 32 * us + 4 * (lane & 7);
#pragma unroll
    for (int i = 0; i < 8; ++i) xv[i] = __builtin_nontemporal_load((const f32x4*)(xp + (size_t)i * 8 * KD));
    GETCNT(8);
  } else if (wup) {
    w_load(w0, p, wt0 & 15, wt0 >> 4, lane);
    GETCNT(16);
  } else {
    if (wave == 4 && lane < 32) lds_st(SY + lane * 4, 0u);
    if (wave == 5 && lane < NR) *(LAS unsigned long long*)(lds + SY_PTR + lane * 8) = (unsigned long long)p.ws[lane];
    GETCNT(0);
  }
  const unsigned call = cnt1 + 1u;
  asm volatile("s_waitcnt lgkmcnt(0)" ::: "memory");
  __builtin_amdgcn_s_barrier();
  cfence();
  if (wave == 7 && lane == 0) __hip_atomic_store(cntp, call, __ATOMIC_RELAXED, __HIP_MEMORY_SCOPE_SYSTEM);
  if (wup) {
    w_store(w0, me, wt0 & 15, wt0 >> 4, lane);
    w_flag(me, wt0 & 15, wt0 >> 4, lane, call);
    if (wave == 7) STAMP(2);
  }
  if (wave < 4) {
    float mx = 0.f, sm = 0.f;
    u32x4 D[4];
    {
      LAS char* tw = lds + wave * XSTG + (lane >> 3) * 16 + ((lane & 7) >> 1) * 256 + (lane & 1) * 8;
#pragma unroll
      for (int i = 0; i < 8; ++i) {
        mx = fmaxf(fmaxf(mx, fmaxf(fabsf(xv[i][0]), fabsf(xv[i][1]))), fmaxf(fabsf(xv[i][2]), fabsf(xv[i][3])));
        sm += fabsf(xv[i][i & 3]);
        *(LAS u32x2*)(tw + (i >> 1) * 1024 + (i & 1) * 128) = u32x2{pk_bf16(xv[i][0], xv[i][1]), pk_bf16(xv[i][2], xv[i][3])};
      }
      if (wave == 0) STAMP(56);
      cfence();
#pragma unroll
      for (int f = 0; f < 4; ++f) D[f] = *(LAS u32x4*)(lds + wave * XSTG + f * 1024 + lane * 16);
    }
    char* xl = me + XB_OFF + (size_t)((rank * 2 + uh) * NSTEP + us) * XSTG + wave * 4096 + lane * 16;
#pragma unroll
    for (int f = 0; f < 4; ++f) st16(xl + f * 1024, D[f]);
    const unsigned emax = wave_umax((pk_bf16(mx, 0.f) >> 7) & 0xffu);
    const unsigned emean = (wave_fsum(sm * (3.32f / 512.f)) >> 23) & 0xffu;
    const unsigned etop = umin(emax, emean);
    const unsigned base = etop > 7u ? etop - 7u : 0u;
    const unsigned B2 = base * 0x00800080u;
    char* pk = me + PK_OFF + (size_t)b * PKU;
    {
      unsigned S[8];
#pragma unroll
      for (int i = 0; i < 8; ++i) {
        const unsigned d0 = D[i >> 1][2 * (i & 1)], d1 = D[i >> 1][2 * (i & 1) + 1];
        S[i] = bfi(0x80808080u, perm(d1, d0, 0x07050301u), perm(d1, d0, 0x06040200u));
      }
      st16(pk + wave * 2048 + lane * 16, u32x4{S[0], S[1], S[2], S[3]});
      st16(pk + wave * 2048 + 1024 + lane * 16, u32x4{S[4], S[5], S[6], S[7]});
    }
    unsigned C[16], Ma = 0, Mb = 0;
#pragma unroll
    for (int i = 0; i < 16; ++i) {
      const unsigned t = pk_sub16(D[i >> 2][i & 3] & 0x7f807f80u, B2);
      C[i] = t & 0x03800380u;
      const unsigned y = pk_min16(t, 0x04000400u);
      if (i < 8) Ma = bfi(0x04000400u, y, Ma >> 1); else Mb = bfi(0x04000400u, y, Mb >> 1);
    }
    {
      u32x3 P;
#pragma unroll
      for (int g = 0; g < 3; ++g)
        P[g] = C[5 * g] | (C[5 * g + 1] << 3) | (C[5 * g + 2] << 6) | (((C[5 * g + 3] << 3) | C[5 * g + 4]) >> 6) | ((C[15] >> (7 + g)) & 0x00010001u);
      asm volatile("global_store_dwordx3 %0, %1, off sc0 sc1\n\ts_nop 1" :: "v"(pk + PK_C + wave * 768 + lane * 12), "v"(P) : "memory");
    }
    unsigned M = (Ma >> 3) | (Mb << 5), cnt = 0;
    bool raw = FORCE_RAW;
    const int eb = wave * XSTG + 4096;
    for (;;) {
      const unsigned long long act = __builtin_amdgcn_ballot_w64(M != 0);
      if (!act) break;
      const unsigned k = M ? __builtin_ctz(M) : 0u;
      const unsigned pos = (k & 12u) * 128u + lane * 8u + (k & 3u) * 2u + (k >> 4);
      const unsigned v = *(LAS unsigned short*)(lds + wave * XSTG + pos * 2);
      const unsigned hi = (unsigned)(((int)((v >> 7) & 0xffu) - (int)base) >> 3) + 28u;
      const unsigned sl = cnt + __builtin_amdgcn_mbcnt_hi((unsigned)(act >> 32), __builtin_amdgcn_mbcnt_lo((unsigned)act, 0u));
      const unsigned slc = M != 0 && sl < EXC ? sl : 127u - 2u;
      *(LAS unsigned short*)(lds + eb + ((wave & 1) && slc < EXC ? PK_R - 6 - 2 * slc : 4 + 2 * slc)) = (unsigned short)(pos | (hi << 11));
      if (__builtin_amdgcn_ballot_w64(M != 0 && hi > 31u)) raw = true;
      cnt += (unsigned)__builtin_popcountll(act);
      M &= M - 1;
      if (cnt > EXC) { raw = true; break; }
    }
    {
      const unsigned nd = ((raw ? 0u : cnt) + 1u) / 2u + 1u;
      const int o = (wave & 1) ? PK_R - 4 - 4 * lane : 4 * lane;
      cfence();
      unsigned v = *(LAS unsigned*)(lds + eb + ((unsigned)lane < nd ? o : 0));
      cfence();
      lds_st(SY_PACK + 4 * wave, 1u);
      if (lane == 0) v = base | ((raw ? 255u : cnt) << 8);
      if ((unsigned)lane < nd) st4(pk + PK_E + (wave >> 1) * PK_R + o, v);
    }
    if (wave == 0) STAMP(57);
    waitv<0>();
    if (lds_ticket(SY_ARR + 4) == 3) {
      if (lane < NR) {
        char* f = lane == rank ? me + XF_OFF + (size_t)((rank * 2 + uh) * NSTEP + us) * 4
                               : (char*)*(LAS unsigned long long*)(lds + SY_PTR + lane * 8) + SF_OFF + (size_t)(rank * 256 + b) * 4;
        st4(f, call);
      }
      STAMP(1);
    }
  }
  const unsigned long long t0 = rt();

  if (b < NPULL) {
    const int src = (rank + 1 + b % 7) & 7, pp = b / 7;
    const char* sws = p.ws[src];
    const char* sf = me + SF_OFF + src * 1024 + lane * 16;
    const unsigned v0 = lane * 16, v1 = v0 + 4096, vc = PK_C + lane * 12, vh = PK_E + (lane < 48 ? lane : 47) * 8;
    LAS char* img = lds + wave * XSTG;
    bool dead = false;
    const char* tkp = me + TK_OFF + ((call & 1) * NR + src) * 128;
    if (pp == 0 && wave == 7 && lane == 0) st4(me + TK_OFF + (((call + 1) & 1) * NR + src) * 128, 0u);
    unsigned long long seen[4] = {0, 0, 0, 0};
    int u = wave * 4 + pp;
    for (bool first = true; wave < NWP; first = false) {
      if (u >= 256) break;
      const int s = u >> 1, h = u & 1, n = u >> 2;
      {
        const int k = u & 3;
        while (!(((k == 0 ? seen[0] : k == 1 ? seen[1] : k == 2 ? seen[2] : seen[3]) >> n) & 1)) {
          u32x4 f; unsigned t;
          poll16(f, t, sf, tmo);
#pragma unroll
          for (int i = 0; i < 4; ++i) seen[i] = __builtin_amdgcn_ballot_w64(f[i] >= call);
          if (((k == 0 ? seen[0] : k == 1 ? seen[1] : k == 2 ? seen[2] : seen[3]) >> n) & 1) break;
          if (expired(t, call, t0, me)) { dead = true; break; }
          __builtin_amdgcn_s_sleep(2);
        }
        if (dead) break;
        if (first) for (int i = 0; i < u; ++i) __builtin_amdgcn_s_sleep(STAG);
        if (n < 8) STAMP(8 + n);
      }
      u32x4 A[8]; u32x3 P[4]; u32x2 E;
      const char* sb = sws + PK_OFF + (size_t)u * PKU;
      asm volatile("s_nop 4\n\t"
                   "global_load_dwordx2 %12, %16, %17 sc0 sc1\n\t"
                   "global_load_dwordx3 %8, %15, %17 sc0 sc1\n\t"
                   "global_load_dwordx3 %9, %15, %17 offset:768 sc0 sc1\n\t"
                   "global_load_dwordx3 %10, %15, %17 offset:1536 sc0 sc1\n\t"
                   "global_load_dwordx3 %11, %15, %17 offset:2304 sc0 sc1\n\t"
                   "global_load_dwordx4 %0, %13, %17 sc0 sc1\n\t"
                   "global_load_dwordx4 %1, %13, %17 offset:1024 sc0 sc1\n\t"
                   "global_load_dwordx4 %2, %13, %17 offset:2048 sc0 sc1\n\t"
                   "global_load_dwordx4 %3, %13, %17 offset:3072 sc0 sc1\n\t"
                   "global_load_dwordx4 %4, %14, %17 sc0 sc1\n\t"
                   "global_load_dwordx4 %5, %14, %17 offset:1024 sc0 sc1\n\t"
                   "global_load_dwordx4 %6, %14, %17 offset:2048 sc0 sc1\n\t"
                   "global_load_dwordx4 %7, %14, %17 offset:3072 sc0 sc1\n\t"
                   "s_waitcnt vmcnt(0)"
                   : "=&v"(A[0]), "=&v"(A[1]), "=&v"(A[2]), "=&v"(A[3]), "=&v"(A[4]), "=&v"(A[5]), "=&v"(A[6]), "=&v"(A[7]),
                     "=&v"(P[0]), "=&v"(P[1]), "=&v"(P[2]), "=&v"(P[3]), "=&v"(E)
                   : "v"(v0), "v"(v1), "v"(vc), "v"(vh), "s"(sb) : "memory");
      const size_t xo = XB_OFF + (size_t)((src * 2 + h) * NSTEP + s) * XSTG;
      char* xd = me + xo + lane * 16;
      const unsigned hd[4] = {(unsigned)__builtin_amdgcn_readlane(E[0], 0), (unsigned)__builtin_amdgcn_readlane(E[1], 23),
                              (unsigned)__builtin_amdgcn_readlane(E[0], 24), (unsigned)__builtin_amdgcn_readlane(E[1], 47)};
      unsigned cn[4];
      bool raw[4];
#pragma unroll
      for (int r = 0; r < 2; ++r) {
        const unsigned c0 = (hd[2 * r] >> 8) & 0xffu, c1 = (hd[2 * r + 1] >> 8) & 0xffu;
        const unsigned n0 = c0 > EXC ? 0u : c0, n1 = c1 > EXC ? 0u : c1;
        const bool clash = ((n0 + 1u) & ~1u) + ((n1 + 1u) & ~1u) > EXC;
        raw[2 * r] = clash || c0 > EXC; raw[2 * r + 1] = clash || c1 > EXC;
        cn[2 * r] = clash ? 0u : n0; cn[2 * r + 1] = clash ? 0u : n1;
      }
#pragma unroll
      for (int q = 0; q < 4; ++q) {
        const unsigned B2 = (hd[q] & 0xffu) * 0x00800080u;
#pragma unroll
        for (int f = 0; f < 4; ++f) {
          u32x4 d;
#pragma unroll
          for (int j = 0; j < 4; ++j) {
            const int i = 4 * f + j, g = i / 5, r = i % 5;
            const unsigned c = i == 15 ? ((P[q][0] & 0x00010001u) << 7) | ((P[q][1] & 0x00010001u) << 8) | ((P[q][2] & 0x00010001u) << 9)
                                       : (r < 3 ? P[q][g] >> (3 * r) : P[q][g] << (3 * r - 6)) & 0x03800380u;
            const unsigned sm = A[2 * q + (f >> 1)][2 * (f & 1) + (j >> 1)];
            d[j] = (perm(sm, sm, (j & 1) ? 0x03030202u : 0x01010000u) & 0x807f807fu) | (c + B2);
          }
          *(LAS u32x4*)(img + q * 4096 + f * 1024 + lane * 16) = d;
        }
      }
      cfence();
      {
        const bool r1 = lane >= 24;
        const unsigned hw0 = 4u * (r1 ? lane - 24 : lane), ce = r1 ? cn[2] : cn[0], co = r1 ? cn[3] : cn[1];
#pragma unroll
        for (int c = 0; c < 4; ++c) {
          const unsigned ent = (E[c >> 1] >> (16 * (c & 1))) & 0xffffu, hw = hw0 + c;
          const bool ev = hw - 2u < ce, od = (PK_R / 2 - 3) - hw < co;
          const unsigned dl = (ev || od) && lane < 48 ? ((ent >> 11) - 28u) << (10u + (ent & 1u) * 16u) : 0u;
          const unsigned ad = wave * XSTG + (r1 ? 8192u : 0u) + (od ? 4096u : 0u) + (ent & 0x7feu) * 2u;
          asm volatile("ds_add_u32 %0, %1" :: "v"(ad), "v"(dl) : "memory");
        }
      }
      cfence();
#pragma unroll
      for (int q = 0; q < 4; ++q) {
        u32x4 R0, R1, R2, R3;
        if (raw[q]) {
          const char* rb = sws + xo + q * 4096;
          asm volatile("s_nop 4\n\t"
                       "global_load_dwordx4 %0, %4, %5 sc0 sc1\n\t"
                       "global_load_dwordx4 %1, %4, %5 offset:1024 sc0 sc1\n\t"
                       "global_load_dwordx4 %2, %4, %5 offset:2048 sc0 sc1\n\t"
                       "global_load_dwordx4 %3, %4, %5 offset:3072 sc0 sc1\n\t"
                       "s_waitcnt vmcnt(0)"
                       : "=&v"(R0), "=&v"(R1), "=&v"(R2), "=&v"(R3) : "v"(v0), "s"(rb) : "memory");
        } else {
          R0 = *(LAS u32x4*)(img + q * 4096 + lane * 16); R1 = *(LAS u32x4*)(img + q * 4096 + 1024 + lane * 16);
          R2 = *(LAS u32x4*)(img + q * 4096 + 2048 + lane * 16); R3 = *(LAS u32x4*)(img + q * 4096 + 3072 + lane * 16);
        }
        st16(xd + q * 4096, R0); st16(xd + q * 4096 + 1024, R1); st16(xd + q * 4096 + 2048, R2); st16(xd + q * 4096 + 3072, R3);
      }
      unsigned tk;
      asm volatile("s_mov_b64 exec, 1\n\ts_nop 3\n\tglobal_atomic_add %0, %1, %2, %3 sc0\n\ts_mov_b64 exec, -1\n\ts_waitcnt vmcnt(0)"
                   : "=&v"(tk) : "v"(0), "v"(1u), "s"(tkp) : "memory");
      flag_st(me + XF_OFF + (size_t)((src * 2 + h) * NSTEP + s) * 4, call);
      if (n < 8) STAMP(16 + n);
      if ((n & 7) == 7) STAMP(48 + n / 8);
      u = (int)umin(rfl(tk), 256u) + 4 * NWP;
    }
    if (wave < NWP) { STAMP(24 + wave); lds_st(SY_TKT, 64u); }
    if (b >= 8) return;
    if (wave >= NWP) {
      while (rfl(lds_ld(SY_TKT)) < 64u) __builtin_amdgcn_s_sleep(32);
    }
    const int lg = lane < NGEMM / 4 ? lane : NGEMM / 4 - 1;
    const char* g0 = me + GR_OFF + (size_t)(((call & 1) * NR + wave) * 256) * 8 + lg * 32;
    for (;;) {
      u32x4 a0, a1; unsigned t;
      asm volatile("global_load_dwordx4 %0, %3, off sc0 sc1\n\tglobal_load_dwordx4 %1, %3, off offset:16 sc0 sc1\n\t"
                   "global_load_dword %2, %4, off sc0 sc1\n\ts_waitcnt vmcnt(0)"
                   : "=&v"(a0), "=&v"(a1), "=&v"(t) : "v"(g0), "v"(tmo) : "memory");
      const bool ok = a0[1] == call && a0[3] == call && a1[1] == call && a1[3] == call;
      if (__builtin_amdgcn_ballot_w64(ok) == ~0ull) {
        const unsigned best = wave_umax(umax(umax(a0[0], a0[2]), umax(a1[0], a1[2])));
        asm volatile("ds_max_u32 %0, %1" :: "v"(SY_GMAX), "v"(best) : "memory");
        if (lds_ticket(SY_ARR) == 7u) {
          const unsigned gm = lds_ld(SY_GMAX);
          if (lane == 0)
            __hip_atomic_store((unsigned long long*)(me + GM_OFF + b * 128), ((unsigned long long)call << 32) | gm,
                               __ATOMIC_RELAXED, __HIP_MEMORY_SCOPE_SYSTEM);
          STAMP(4);
        }
        break;
      }
      if (expired(t, call, t0, me)) break;
      __builtin_amdgcn_s_sleep(1);
    }
    return;
  }
  if (b < NCOMM) {
    for (;;) {
      const int t = NGEMM * UPF + (b - NPULL) + NCONV * (int)lds_ticket(SY_TKT);
      if (t >= 16 * NSTEP) break;
      if ((t & 255) < NCONV) STAMP(8 + (t >> 8));
      convert_w(p, me, t & 15, t >> 4, lane, call);
    }
    if (wave == 0) STAMP(2);
    return;
  }

  const int xcd = b & 7, idx = (b >> 3) - NCOMM / 8, xh = xcd & 1;
  const bool extra = idx < 24;
  const int di = extra ? idx >> 2 : 6;
  const int pn = (xcd >> 1) * 4 + (extra ? idx & 3 : 2 * xh + (idx & 1));
  const int src = (rank + 1 + di) & 7, h = extra ? xh : 0;
  const int esrc = di < 4 ? rank : (rank + 7) & 7, eh = di < 4 ? xh : extra ? 1 : 0, eq = di < 4 ? di : 2 * xh + (di & 1);
  if (wave < 4) {
    __builtin_amdgcn_s_setprio(PRIOC);
    if (extra) consumer<5>(p, me, lds, lane, wave, call, b - NCOMM, src * 512 + h * 256, esrc * 512 + eh * 256 + eq * 64, pn, t0);
    else consumer<4>(p, me, lds, lane, wave, call, b - NCOMM, src * 512 + h * 256, 0, pn, t0);
    return;
  }
  bool dead = false;
  if (wave < 6) {
    __builtin_amdgcn_s_setprio(PRIOL);
    const char* xs = me + XB_OFF + (size_t)((src * 2 + h) * NSTEP) * XSTG;
    const char* ls = me + XB_OFF + (size_t)((esrc * 2 + eh) * NSTEP) * XSTG + eq * 4096;
    const char* wsr = me + WB_OFF + (size_t)(pn * NSTEP) * WSTG;
    const unsigned voff = lane * 16;
    unsigned slot = (wave - 4) * STG;
    const int fw = SY_FULL + 4 * (wave - 4);
    unsigned pend = 0;
    for (;;) {
      u32x4 d;
      asm volatile("ds_read_b128 %0, %1\n\ts_waitcnt lgkmcnt(0)" : "=&v"(d) : "v"(SY_PACK) : "memory");
      if (rfl(d[0] & d[1] & d[2] & d[3])) break;
      __builtin_amdgcn_s_sleep(2);
    }
    for (int s = wave - 4; s < NSTEP; s += 2) {
      unsigned rdy;
      for (;;) {
        u32x4 d; unsigned r;
        asm volatile("ds_read_b128 %0, %2\n\tds_read_b32 %1, %3\n\ts_waitcnt lgkmcnt(0)" : "=&v"(d), "=&v"(r) : "v"(SY_DONE), "v"(SY_READY) : "memory");
        rdy = rfl(r);
        unsigned dm = rfl(umin(umin(d[0], d[1]), umin(d[2], d[3])));
        if (rdy >= ABORT || (rdy > (unsigned)s && dm + NS > (unsigned)s)) break;
        if (pend) {
          waitv<0>(); lds_st(fw, pend); pend = 0;
        } else __builtin_amdgcn_s_sleep(SLEEPL);
      }
      if (rdy >= ABORT) { pend = ABORT; break; }
      const char* xa = xs + (size_t)s * XSTG;
      glds4(voff, wsr + (size_t)s * WSTG, slot + WOF); glds4(voff, wsr + (size_t)s * WSTG + 4096, slot + WOF + 4096);
      glds4(voff, xa, slot); glds4(voff, xa + 4096, slot + 5120); glds4(voff, xa + 8192, slot + 10240); glds4(voff, xa + 12288, slot + 15360);
      if (extra) { glds4s(voff, ls + (size_t)s * XSTG, slot + 4096); waitv<28>(); }
      else waitv<24>();
      if (pend) lds_st(fw, pend);
      pend = s + 1;
      slot += 2 * STG; if (slot >= NS * STG) slot -= NS * STG;
    }
    waitv<0>();
    lds_st(fw, pend);
    __builtin_amdgcn_s_setprio(0);
  } else if (wave == 6) {
    __builtin_amdgcn_s_setprio(PRIOP);
    const char* pa = lane < 32 ? me + XF_OFF + (size_t)((src * 2 + h) * NSTEP) * 4 + lane * 16
                               : me + WF_OFF + (size_t)(pn * NSTEP) * 4 + (lane & 31) * 16;
    const char* pb = lane < 32 ? me + XF_OFF + (size_t)((esrc * 2 + eh) * NSTEP) * 4 + lane * 16 : pa;
    unsigned ready = 0;
    while (!dead && ready < (unsigned)NSTEP) {
      if (ready > rfl(lds_ld(SY_DONE)) + NS + 2) { __builtin_amdgcn_s_sleep(SLEEPP); continue; }
      u32x4 f, g; unsigned t;
      asm volatile("global_load_dwordx4 %0, %3, off sc0 sc1\n\tglobal_load_dwordx4 %1, %4, off sc0 sc1\n\t"
                   "global_load_dword %2, %5, off sc0 sc1\n\ts_waitcnt vmcnt(0)"
                   : "=&v"(f), "=&v"(g), "=&v"(t) : "v"(pa), "v"(pb), "v"(tmo) : "memory");
#pragma unroll
      for (int i = 0; i < 4; ++i) f[i] = umin(f[i], g[i]);
      unsigned c = f[0] < call ? 0u : f[1] < call ? 1u : f[2] < call ? 2u : f[3] < call ? 3u : 4u;
      unsigned long long part = __builtin_amdgcn_ballot_w64(c < 4u);
      unsigned px = (unsigned)part, pw = (unsigned)(part >> 32);
      unsigned rx = NSTEP, rw = NSTEP;
      if (px) { unsigned l = __builtin_ctz(px); rx = 4 * l + __builtin_amdgcn_readlane(c, l); }
      if (pw) { unsigned l = __builtin_ctz(pw); rw = 4 * l + __builtin_amdgcn_readlane(c, 32 + l); }
      unsigned r = umin(rx, rw);
      if (r > ready) {
        ready = r; lds_st(SY_READY, ready); continue; }
      if (expired(t, call, t0, me)) { dead = true; break; }
      __builtin_amdgcn_s_sleep(1);
    }
    if (dead) lds_st(SY_READY, ABORT);
    STAMP(3);
  }
}

}

extern "C" void kernel_launch(int rank, int nranks, ncclComm_t comm, hipStream_t stream,
                              void* const* d_in, const int* in_sizes, int n_in,
                              void* d_out, int out_size, const int* in_dtypes, int out_dtype,
                              void* const* d_out_all, void* const* d_ws_all, size_t ws_bytes) {
  if (nranks != NR || (unsigned)rank >= (unsigned)NR || n_in < 2 || ws_bytes < WS_NEED ||
      in_sizes[0] != 512 * KD || out_size != 4096 * NLOC) {
    fprintf(stderr, "kernel_launch: unexpected problem shape\n");
    return;
  }
  HIP_CHECK(hipFuncSetAttribute((const void*)fused, hipFuncAttributeMaxDynamicSharedMemorySize, LDS_BYTES));
  Params p;
  p.x = (const float*)d_in[0]; p.w = (const float*)d_in[1]; p.out = (char*)d_out; p.rank = rank;
  for (int i = 0; i < NR; ++i) p.ws[i] = (char*)d_ws_all[i];
  fused<<<NWG, 512, LDS_BYTES, stream>>>(p);
  HIP_CHECK(hipGetLastError());
}
```

```cpp
#include <hip/hip_runtime.h>
#include <rccl/rccl.h>
#include <cstdio>
#include <cstdint>

#define HIP_CHECK(expr) do { hipError_t _e = (expr); if (_e != hipSuccess) { \
    fprintf(stderr, "HIP error %s at %s:%d\n", hipGetErrorString(_e), __FILE__, __LINE__); \
    return; } } while (0)

namespace {

typedef unsigned u32x4 __attribute__((ext_vector_type(4)));
typedef unsigned u32x3 __attribute__((ext_vector_type(3)));
typedef unsigned u32x2 __attribute__((ext_vector_type(2)));
typedef unsigned short u16x2 __attribute__((ext_vector_type(2)));
typedef float f32x4 __attribute__((ext_vector_type(4)));
typedef float f32x2 __attribute__((ext_vector_type(2)));
typedef __bf16 bf16x8 __attribute__((ext_vector_type(8)));
typedef __bf16 bf16x2 __attribute__((ext_vector_type(2)));
#define LAS __attribute__((address_space(3)))

#define FORCE_RAW 0
#define UPF 1
#define PRIOC 3
#define PRIOL 0
#define PRIOP 0
#define SLEEPL 1
#define SLEEPP 4
#define STAG 4
#define LSC " sc0 sc1"
#define GNOP 2
#define GREP 1
#define STR2(x) #x
#define STR(x) STR2(x)
#define GAP ".rept " STR(GREP) "\n\ts_nop " STR(GNOP) "\n\t.endr\n\t"
#define NWP 6

constexpr int NR = 8;
constexpr int KD = 4096, ND = 16384, NLOC = 2048;
constexpr int NSTEP = 128;
constexpr int NPULL = 28, NCOMM = 48, NCONV = NCOMM - NPULL, NGEMM = 208, NWG = 256;
constexpr int XSTG = 16384, WSTG = 8192;
constexpr int PK_C = 8192, PK_E = 11264, PK_R = 192, PKU = PK_E + 2 * PK_R;
constexpr unsigned EXC = (PK_R - 8) / 2;
constexpr size_t XB_OFF = 0;
constexpr size_t WB_OFF = 32u << 20;
constexpr size_t PK_OFF = 48u << 20;
constexpr size_t FL_OFF = 64u << 20;
constexpr size_t CNT_OFF = FL_OFF;
constexpr size_t TMO_OFF = FL_OFF + 0x4000;
constexpr size_t SF_OFF = FL_OFF + 0x5000;
constexpr size_t XF_OFF = FL_OFF + 0x60000;
constexpr size_t WF_OFF = FL_OFF + 0xA000;
constexpr size_t GM_OFF = FL_OFF + 0xC000;
constexpr size_t TK_OFF = FL_OFF + 0xD000;
constexpr size_t GR_OFF = FL_OFF + 0x10000;
constexpr size_t ST_OFF = FL_OFF + 0x20000;
constexpr size_t IO_OFF = FL_OFF + 0x40000;
constexpr size_t WS_NEED = FL_OFF + 0x64000;
constexpr int NS = 5;
constexpr int STG = 28672;
constexpr int SY = NS * STG;
constexpr int SY_FULL = SY, SY_READY = SY + 8, SY_DONE = SY + 16, SY_ARR = SY + 32, SY_WMAX = SY + 48,
              SY_GMAX = SY + 64, SY_TKT = SY + 84, SY_PACK = SY + 96, SY_SRC = SY + 112, SY_PTR = SY + 144;
constexpr int LDS_BYTES = SY + 208;
constexpr unsigned long long DEADLINE = 50000000ull;
constexpr unsigned ABORT = 0x40000000u;

struct Params { const float* x; const float* w; char* out; char* ws[NR]; int rank; };

__device__ __forceinline__ unsigned long long rt() { return __builtin_amdgcn_s_memrealtime(); }
template <int N_> __device__ __forceinline__ void waitv() { asm volatile("s_waitcnt vmcnt(%0)" :: "i"(N_) : "memory"); }
__device__ __forceinline__ void cfence() { asm volatile("" ::: "memory"); }
__device__ __forceinline__ void st16(char* p, u32x4 v) {
  asm volatile("global_store_dwordx4 %0, %1, off sc0 sc1\n\ts_nop 1" :: "v"(p), "v"(v) : "memory");
}
__device__ __forceinline__ void st4(char* p, unsigned v) {
  asm volatile("global_store_dword %0, %1, off sc0 sc1" :: "v"(p), "v"(v) : "memory");
}
__device__ __forceinline__ void flag_st(const char* p, unsigned v) {
  asm volatile("s_mov_b64 exec, 1\n\ts_nop 3\n\tglobal_store_dword %0, %1, %2 sc0 sc1\n\ts_mov_b64 exec, -1" :: "v"(0), "v"(v), "s"(p) : "memory");
}
__device__ __forceinline__ unsigned lds_ticket(int addr) {
  unsigned v, r;
  asm volatile("s_mov_b64 exec, 1\n\tds_add_rtn_u32 %0, %2, %3\n\ts_mov_b64 exec, -1\n\ts_waitcnt lgkmcnt(0)\n\tv_readfirstlane_b32 %1, %0"
               : "=&v"(v), "=s"(r) : "v"(addr), "v"(1u) : "memory");
  return r;
}
__device__ __forceinline__ void poll16(u32x4& a, unsigned& t, const char* p, const char* tm) {
  asm volatile("global_load_dwordx4 %0, %2, off sc0 sc1\n\tglobal_load_dword %1, %3, off sc0 sc1\n\ts_waitcnt vmcnt(0)"
               : "=&v"(a), "=&v"(t) : "v"(p), "v"(tm) : "memory");
}
__device__ __forceinline__ void lds_st(int addr, unsigned v) {
  asm volatile("ds_write_b32 %0, %1" :: "v"(addr), "v"(v) : "memory");
}
__device__ __forceinline__ unsigned lds_ld(int addr) {
  unsigned v; asm volatile("ds_read_b32 %0, %1\n\ts_waitcnt lgkmcnt(0)" : "=&v"(v) : "v"(addr) : "memory"); return v;
}
__device__ __forceinline__ unsigned rfl(unsigned v) { return __builtin_amdgcn_readfirstlane(v); }
__device__ __forceinline__ unsigned umax(unsigned a, unsigned b) { return a > b ? a : b; }
__device__ __forceinline__ unsigned umin(unsigned a, unsigned b) { return a < b ? a : b; }
__device__ __forceinline__ unsigned wave_umax(unsigned v) {
  v = umax(v, (unsigned)__builtin_amdgcn_update_dpp(0, (int)v, 0x111, 0xf, 0xf, false));
  v = umax(v, (unsigned)__builtin_amdgcn_update_dpp(0, (int)v, 0x112, 0xf, 0xf, false));
  v = umax(v, (unsigned)__builtin_amdgcn_update_dpp(0, (int)v, 0x114, 0xf, 0xf, false));
  v = umax(v, (unsigned)__builtin_amdgcn_update_dpp(0, (int)v, 0x118, 0xf, 0xf, false));
  unsigned a = __builtin_amdgcn_readlane(v, 15), b = __builtin_amdgcn_readlane(v, 31);
  unsigned c = __builtin_amdgcn_readlane(v, 47), d = __builtin_amdgcn_readlane(v, 63);
  return umax(umax(a, b), umax(c, d));
}
__device__ __forceinline__ unsigned wave_fsum(float v) {
#define DPPADD(ctrl, rm) v += __builtin_bit_cast(float, __builtin_amdgcn_update_dpp(0, __builtin_bit_cast(int, v), ctrl, rm, 0xf, false))
  DPPADD(0x111, 0xf); DPPADD(0x112, 0xf); DPPADD(0x114, 0xf); DPPADD(0x118, 0xf); DPPADD(0x142, 0xa); DPPADD(0x143, 0xc);
  return __builtin_amdgcn_readlane(__builtin_bit_cast(unsigned, v), 63);
}
__device__ __forceinline__ unsigned bfi(unsigned m, unsigned a, unsigned b) { return (a & m) | (b & ~m); }
__device__ __forceinline__ unsigned pk_sub16(unsigned a, unsigned b) {
  return __builtin_bit_cast(unsigned, (u16x2)(__builtin_bit_cast(u16x2, a) - __builtin_bit_cast(u16x2, b)));
}
__device__ __forceinline__ unsigned pk_min16(unsigned a, unsigned b) {
  return __builtin_bit_cast(unsigned, __builtin_elementwise_min(__builtin_bit_cast(u16x2, a), __builtin_bit_cast(u16x2, b)));
}
__device__ __forceinline__ unsigned pk_bf16(float lo, float hi) {
  f32x2 f = {lo, hi};
  return __builtin_bit_cast(unsigned, __builtin_convertvector(f, bf16x2));
}
__device__ __forceinline__ unsigned perm(unsigned hi, unsigned lo, unsigned sel) { return __builtin_amdgcn_perm(hi, lo, sel); }
__device__ __forceinline__ unsigned rotr(unsigned v, unsigned n) { return __builtin_amdgcn_alignbit(v, v, n); }
__device__ __forceinline__ void glds4(unsigned voff, const char* sbase, unsigned dst) {
  asm volatile("s_mov_b32 m0, %2\n\ts_nop 0\n\t"
               "global_load_lds_dwordx4 %0, %1" LSC "\n\t" GAP
               "global_load_lds_dwordx4 %0, %1 offset:1024" LSC "\n\t" GAP
               "global_load_lds_dwordx4 %0, %1 offset:2048" LSC "\n\t" GAP
               "global_load_lds_dwordx4 %0, %1 offset:3072" LSC "\n\t" GAP
               :: "v"(voff), "s"(sbase), "s"(dst) : "memory");
}
__device__ __forceinline__ void glds4s(unsigned voff, const char* sbase, unsigned dst) {
  asm volatile("s_mov_b32 m0, %2\n\ts_nop 0\n\t"
               "global_load_lds_dwordx4 %0, %1" LSC "\n\t" GAP
               "s_add_u32 m0, m0, 4096\n\ts_nop 0\n\t"
               "global_load_lds_dwordx4 %0, %1 offset:1024" LSC "\n\t" GAP
               "s_add_u32 m0, m0, 4096\n\ts_nop 0\n\t"
               "global_load_lds_dwordx4 %0, %1 offset:2048" LSC "\n\t" GAP
               "s_add_u32 m0, m0, 4096\n\ts_nop 0\n\t"
               "global_load_lds_dwordx4 %0, %1 offset:3072" LSC "\n\t" GAP
               :: "v"(voff), "s"(sbase), "s"(dst) : "memory", "scc");
}
#define STAMP(i) do {} while (0)

__device__ __forceinline__ bool expired(unsigned tmo_seen, unsigned call, unsigned long long t0, char* me) {
  if (rfl(tmo_seen) == call) return true;
  if (rt() - t0 > DEADLINE) {
    __hip_atomic_store((unsigned*)(me + TMO_OFF), call, __ATOMIC_RELAXED, __HIP_MEMORY_SCOPE_AGENT);
    return true;
  }
  return false;
}

struct WRegs { f32x4 v[2][8]; };
__device__ __forceinline__ void w_load(WRegs& r, const Params& p, int panel, int s, int lane) {
  const float* wp = p.w + (size_t)(32 * s + 8 * (lane >> 4)) * ND + p.rank * NLOC + panel * 128 + 4 * (lane & 15);
#pragma unroll
  for (int hf = 0; hf < 2; ++hf)
#pragma unroll
    for (int j = 0; j < 8; ++j) r.v[hf][j] = __builtin_nontemporal_load((const f32x4*)(wp + (size_t)j * ND + hf * 64));
}
__device__ __forceinline__ void w_store(const WRegs& r, char* me, int panel, int s, int lane) {
  char* dst = me + WB_OFF + (size_t)(panel * NSTEP + s) * WSTG + lane * 16;
#pragma unroll
  for (int hf = 0; hf < 2; ++hf)
#pragma unroll
    for (int c = 0; c < 4; ++c) {
      u32x4 o = {pk_bf16(r.v[hf][0][c], r.v[hf][1][c]), pk_bf16(r.v[hf][2][c], r.v[hf][3][c]),
                 pk_bf16(r.v[hf][4][c], r.v[hf][5][c]), pk_bf16(r.v[hf][6][c], r.v[hf][7][c])};
      st16(dst + (4 * hf + c) * 1024, o);
    }
}
__device__ __forceinline__ void w_flag(char* me, int panel, int s, int lane, unsigned call) {
  waitv<0>();
  flag_st(me + WF_OFF + (size_t)(panel * NSTEP + s) * 4, call);
}
__device__ __forceinline__ void convert_w(const Params& p, char* me, int panel, int s, int lane, unsigned call) {
  WRegs r;
  w_load(r, p, panel, s, lane);
  w_store(r, me, panel, s, lane);
  w_flag(me, panel, s, lane, call);
}

#define MFMA(m, t) asm volatile("v_mfma_f32_16x16x32_bf16 %0, %1, %2, %0" : "+v"(acc[m][t]) : "v"(Wf[t]), "v"(Xc[m]))
#define LDR(dst, addr, off) asm volatile("ds_read_b128 %0, %1 offset:%2" : "=v"(dst) : "v"(addr), "i"(off) : "memory")
template <int N_> __device__ __forceinline__ void waitl() { asm volatile("s_waitcnt lgkmcnt(%0)" :: "i"(N_) : "memory"); }
constexpr int WOF = 20480;
#define MAX2(m, t) asm volatile("v_max3_f32 %0, %2, %3, %0\n\tv_max3_f32 %1, %4, %5, %1" : "+v"(mx0), "+v"(mx1) \
                                : "v"(acc[m][t][0]), "v"(acc[m][t][1]), "v"(acc[m][t][2]), "v"(acc[m][t][3]))
template <int NM, int MODE, int PAR>
__device__ __forceinline__ void mstep(f32x4 (&acc)[NM][8], bf16x8 (&Wf)[8], bf16x8 (&Xc)[NM], bf16x8 (&Xn)[NM],
                                      unsigned& va, unsigned& vx, unsigned va0, unsigned vx0, unsigned vsy, unsigned& vdone,
                                      unsigned& slot, unsigned& fpre, int s, float& mx0, float& mx1, unsigned& nslow) {
  constexpr bool PF = MODE < 2;
  unsigned sf = 0, nslot = 0;
  constexpr int NX2 = 2 + (NM == 5) + (MODE == 0);
  waitl<5>();
  MFMA(0, 0);
  LDR(Wf[7], va, WOF + 7168);
  MFMA(1, 0);
  if (PF) asm volatile("ds_write_b32 %0, %1 offset:16" :: "v"(vsy), "v"(vdone) : "memory");
  MFMA(2, 0);
  if (PF) asm volatile("v_readfirstlane_b32 %0, %1" : "=s"(sf) : "v"(fpre) : "memory");
  MFMA(3, 0);
  if (PF) asm volatile("v_add_u32 %0, %1, %2" : "=v"(va) : "s"(slot), "v"(va0));
  if (NM == 5) MFMA(NM - 1, 0);
  if (PF) asm volatile("v_add_u32 %0, %1, %2" : "=v"(vx) : "s"(slot), "v"(vx0));
  if (PF && __builtin_expect(sf <= (unsigned)(s + 1), 0)) {
    do __builtin_amdgcn_s_sleep(1); while (rfl(lds_ld(SY_FULL + 4 * (1 - PAR))) <= (unsigned)(s + 1));
  }
#pragma unroll
  for (int t = 1; t < 8; ++t) {
    if (t == 2) waitl<PF ? 10 : 5>();
    if (t == 3) waitl<PF ? 9 + NX2 : 4>();
    if (t == 4) waitl<PF ? 9 + NX2 : 3>();
    if (t == 5) waitl<PF ? 9 + NX2 : 2>();
    if (t == 6) waitl<PF ? 9 + NX2 : 1>();
    if (t == 7) waitl<PF ? 9 + NX2 : 0>();
    MFMA(0, t);
    if (PF) LDR(Wf[t - 1], va, WOF + (t - 1) * 1024); else MAX2(0, t - 1);
    MFMA(1, t);
    if (PF && t == 1) LDR(Xn[0], vx, 0);
    if (PF && t == 2) LDR(Xn[3], vx, 3 * 1024);
    if (MODE == 0 && t == 5) asm volatile("s_add_u32 %0, %1, %2" : "=s"(nslot) : "s"(slot), "i"(STG) : "scc");
    if (MODE == 0 && t == 6) asm volatile("s_cmp_lg_u32 %1, %2\n\ts_cselect_b32 %0, %1, 0" : "=s"(slot) : "s"(nslot), "i"(NS * STG) : "scc");
    if (!PF) MAX2(1, t - 1);
    MFMA(2, t);
    if (PF && t == 1) LDR(Xn[1], vx, 1024);
    if (PF && t == 2 && NM == 5) LDR(Xn[NM - 1], vx, 4 * 1024);
    if (PF && t == 4) asm volatile("v_add_u32 %0, 1, %0" : "+v"(vdone));
    if (!PF) MAX2(2, t - 1);
    MFMA(3, t);
    if (PF && t == 1) LDR(Xn[2], vx, 2 * 1024);
    if (MODE == 0 && t == 2) asm volatile("ds_read_b32 %0, %1 offset:%2" : "=v"(fpre) : "v"(SY_FULL), "i"(4 * PAR) : "memory");
    if (!PF) MAX2(3, t - 1);
    if (NM == 5) { MFMA(NM - 1, t); if (!PF) MAX2(NM - 1, t - 1); }
  }
  if (!PF) {
    asm volatile("s_nop 7\n\ts_nop 7" ::: "memory");
#pragma unroll
    for (int m = 0; m < NM; ++m) MAX2(m, 7);
  }
}

template <int NM>
__device__ __forceinline__ void consumer(const Params& p, char* me, LAS char* lds, int lane, int wave, unsigned call,
                                         int gidx, int row0, int row4, int pn, unsigned long long t0) {
  f32x4 acc[NM][8];
#pragma unroll
  for (int m = 0; m < NM; ++m)
#pragma unroll
    for (int t = 0; t < 8; ++t) { acc[m][t] = f32x4{0.f, 0.f, 0.f, 0.f}; asm volatile("" : "+v"(acc[m][t])); }
  bf16x8 Wf[8], Xa[NM], Xb[NM];
  unsigned slot = STG, fpre, vdone = 1;
  float mx0 = 0.f, mx1 = 0.f;
  unsigned nslow = 0;
  const unsigned va0 = lane * 16, vx0 = lane * 16 + wave * 5120, vsy = SY + 4 * wave;
  unsigned va = va0, vx = vx0;
  while (rfl(lds_ld(SY_FULL)) == 0) __builtin_amdgcn_s_sleep(1);
  STAMP(8 + wave);
#pragma unroll
  for (int t = 0; t < 7; ++t) LDR(Wf[t], va, WOF + t * 1024);
#pragma unroll
  for (int t = 0; t < NM; ++t) LDR(Xa[t], vx, t * 1024);
  asm volatile("ds_read_b32 %0, %1 offset:4\n\ts_waitcnt lgkmcnt(0)" : "=v"(fpre) : "v"(SY_FULL) : "memory");
  for (int s = 0; s < NSTEP - 2; s += 2) {
    mstep<NM, 0, 0>(acc, Wf, Xa, Xb, va, vx, va0, vx0, vsy, vdone, slot, fpre, s, mx0, mx1, nslow);
    mstep<NM, 0, 1>(acc, Wf, Xb, Xa, va, vx, va0, vx0, vsy, vdone, slot, fpre, s + 1, mx0, mx1, nslow);
  }
  mstep<NM, 1, 0>(acc, Wf, Xa, Xb, va, vx, va0, vx0, vsy, vdone, slot, fpre, NSTEP - 2, mx0, mx1, nslow);
  mstep<NM, 2, 1>(acc, Wf, Xb, Xa, va, vx, va0, vx0, vsy, vdone, slot, fpre, NSTEP - 1, mx0, mx1, nslow);
  STAMP(12 + wave);
  unsigned wm = wave_umax(umax(__builtin_bit_cast(unsigned, mx0), __builtin_bit_cast(unsigned, mx1)));
  asm volatile("ds_max_u32 %0, %1" :: "v"(SY_WMAX), "v"(wm) : "memory");
  if (lds_ticket(SY_ARR + 8) == 3) {
    unsigned m4 = lds_ld(SY_WMAX);
    if (lane < NR) {
      char* g = (char*)*(LAS unsigned long long*)(lds + SY_PTR + lane * 8) + GR_OFF + (size_t)(((call & 1) * NR + p.rank) * 256 + gidx) * 8;
      asm volatile("global_store_dwordx2 %0, %1, off sc0 sc1\n\ts_nop 1" :: "v"(g), "v"(u32x2{m4, call}) : "memory");
    }
  }
#pragma unroll
  for (int m = 0; m < NM; ++m)
#pragma unroll
    for (int t = 0; t < 8; ++t)
#pragma unroll
      for (int j = 0; j < 4; ++j) asm volatile("v_max_f32 %0, 0, %0" : "+v"(acc[m][t][j]));
  bool bad = false;
  float gmax;
  {
    const char* gp = me + GM_OFF + (blockIdx.x & 7) * 128;
    const char* tm = me + TMO_OFF;
    for (;;) {
      u32x2 g; unsigned t;
      asm volatile("global_load_dwordx2 %0, %2, %3 sc0 sc1\n\tglobal_load_dword %1, %2, %4 sc0 sc1\n\ts_waitcnt vmcnt(0)"
                   : "=&v"(g), "=&v"(t) : "v"(0), "s"(gp), "s"(tm) : "memory");
      gmax = __builtin_bit_cast(float, rfl(g[0]));
      if (rfl(t) == call) { bad = true; break; }
      if (rfl(g[1]) == call) break;
      if (expired(t, call, t0, me)) { bad = true; break; }
      __builtin_amdgcn_s_sleep(1);
    }
  }
  float inv = gmax > 0.f ? 127.0f / gmax : 0.f;
  STAMP(16 + wave);
  char* ob = p.out + pn * 128 + 16 * (lane >> 4);
#pragma unroll
  for (int m = 0; m < NM; ++m) {
    int row = (m < 4 ? row0 + 64 * wave + 16 * m : row4 + 16 * wave) + (lane & 15);
#pragma unroll
    for (int hf = 0; hf < 2; ++hf) {
      u32x4 o;
#pragma unroll
      for (int j = 0; j < 4; ++j) {
        unsigned q[4];
#pragma unroll
        for (int c = 0; c < 4; ++c) q[c] = __builtin_bit_cast(unsigned, __builtin_fmaf(acc[m][4 * hf + c][j], inv, 8388608.0f));
        o[j] = perm((q[3] << 8) | q[2], (q[1] << 8) | q[0], 0x05040100u);
      }
      if (bad) o = u32x4{0x80808080u, 0x80808080u, 0x80808080u, 0x80808080u};
      *(u32x4*)(ob + (size_t)row * NLOC + 64 * hf) = o;
    }
  }
  STAMP(20 + wave);
}

__global__ __launch_bounds__(512, 2) void fused(Params p) {
  extern __shared__ char smem[];
  LAS char* lds = (LAS char*)smem;
  const int lane = threadIdx.x & 63;
  const int wave = __builtin_amdgcn_readfirstlane(threadIdx.x >> 6);
  const int b = blockIdx.x;
  const int rank = p.rank;
  char* me = p.ws[rank];
  const char* tmo = me + TMO_OFF;
  if (wave == 0) STAMP(0);

  unsigned* cntp = (unsigned*)(me + CNT_OFF + b * 64);
  unsigned cnt0, cnt1;
  asm volatile("global_load_dword %0, %1, off sc0 sc1" : "=&v"(cnt0) : "v"(cntp) : "memory");
#define GETCNT(N) asm volatile("s_waitcnt vmcnt(" #N ")\n\tv_readfirstlane_b32 %0, %1" : "=s"(cnt1) : "v"(cnt0) : "memory")
  const int wt0 = b - NCOMM + NGEMM * (7 - wave);
  const bool wup = b >= NCOMM && wave >= 8 - UPF;
  f32x4 xv[8];
  WRegs w0;
  const int ub = b < NPULL ? b : b < NCOMM ? b + NGEMM : b - NCONV;
  const int us = ub >> 1, uh = ub & 1;
  if (wave < 4) {
    const float* xp = p.x + (size_t)(256 * uh + 64 * wave + (lane >> 3)) * KD + 32 * us + 4 * (lane & 7);
#pragma unroll
    for (int i = 0; i < 8; ++i) xv[i] = __builtin_nontemporal_load((const f32x4*)(xp + (size_t)i * 8 * KD));
    GETCNT(8);
  } else if (wup) {
    w_load(w0, p, wt0 & 15, wt0 >> 4, lane);
    GETCNT(16);
  } else {
    if (wave == 4 && lane < 36) lds_st(SY + lane * 4, 0u);
    if (wave == 5 && lane < NR) *(LAS unsigned long long*)(lds + SY_PTR + lane * 8) = (unsigned long long)p.ws[lane];
    GETCNT(0);
  }
  const unsigned call = cnt1 + 1u;
  asm volatile("s_waitcnt lgkmcnt(0)" ::: "memory");
  __builtin_amdgcn_s_barrier();
  cfence();
  if (wave == 7 && lane == 0) __hip_atomic_store(cntp, call, __ATOMIC_RELAXED, __HIP_MEMORY_SCOPE_SYSTEM);
  if (wup) {
    w_store(w0, me, wt0 & 15, wt0 >> 4, lane);
    w_flag(me, wt0 & 15, wt0 >> 4, lane, call);
    if (wave == 7) STAMP(2);
  }
  if (wave < 4) {
    float mx = 0.f, sm = 0.f;
    u32x4 D[4];
    {
      LAS char* tw = lds + wave * XSTG + (lane >> 3) * 16 + ((lane & 7) >> 1) * 256 + (lane & 1) * 8;
#pragma unroll
      for (int i = 0; i < 8; ++i) {
        mx = fmaxf(fmaxf(mx, fmaxf(fabsf(xv[i][0]), fabsf(xv[i][1]))), fmaxf(fabsf(xv[i][2]), fabsf(xv[i][3])));
        sm += fabsf(xv[i][i & 3]);
        *(LAS u32x2*)(tw + (i >> 1) * 1024 + (i & 1) * 128) = u32x2{pk_bf16(xv[i][0], xv[i][1]), pk_bf16(xv[i][2], xv[i][3])};
      }
      if (wave == 0) STAMP(56);
      cfence();
#pragma unroll
      for (int f = 0; f < 4; ++f) D[f] = *(LAS u32x4*)(lds + wave * XSTG + f * 1024 + lane * 16);
    }
    char* xl = me + XB_OFF + (size_t)((rank * 2 + uh) * NSTEP + us) * XSTG + wave * 4096 + lane * 16;
#pragma unroll
    for (int f = 0; f < 4; ++f) st16(xl + f * 1024, D[f]);
    const unsigned emax = wave_umax((pk_bf16(mx, 0.f) >> 7) & 0xffu);
    const unsigned emean = (wave_fsum(sm * (3.32f / 512.f)) >> 23) & 0xffu;
    const unsigned etop = umin(emax, emean);
    const unsigned base = etop > 7u ? etop - 7u : 0u;
    const unsigned B2 = base * 0x00800080u;
    char* pk = me + PK_OFF + (size_t)ub * PKU;
    {
      unsigned S[8];
#pragma unroll
      for (int i = 0; i < 8; ++i) {
        const unsigned d0 = D[i >> 1][2 * (i & 1)], d1 = D[i >> 1][2 * (i & 1) + 1];
        S[i] = bfi(0x80808080u, perm(d1, d0, 0x07050301u), perm(d1, d0, 0x06040200u));
      }
      st16(pk + wave * 2048 + lane * 16, u32x4{S[0], S[1], S[2], S[3]});
      st16(pk + wave * 2048 + 1024 + lane * 16, u32x4{S[4], S[5], S[6], S[7]});
    }
    unsigned C[16], Ma = 0, Mb = 0;
#pragma unroll
    for (int i = 0; i < 16; ++i) {
      const unsigned t = pk_sub16(D[i >> 2][i & 3] & 0x7f807f80u, B2);
      C[i] = t & 0x03800380u;
      const unsigned y = pk_min16(t, 0x04000400u);
      if (i < 8) Ma = bfi(0x04000400u, y, Ma >> 1); else Mb = bfi(0x04000400u, y, Mb >> 1);
    }
    {
      u32x3 P;
#pragma unroll
      for (int g = 0; g < 3; ++g)
        P[g] = C[5 * g] | (C[5 * g + 1] << 3) | (C[5 * g + 2] << 6) | (((C[5 * g + 3] << 3) | C[5 * g + 4]) >> 6) | ((C[15] >> (7 + g)) & 0x00010001u);
      asm volatile("global_store_dwordx3 %0, %1, off sc0 sc1\n\ts_nop 1" :: "v"(pk + PK_C + wave * 768 + lane * 12), "v"(P) : "memory");
    }
    unsigned M = (Ma >> 3) | (Mb << 5), cnt = 0;
    bool raw = FORCE_RAW;
    const int eb = wave * XSTG + 4096;
    for (;;) {
      const unsigned long long act = __builtin_amdgcn_ballot_w64(M != 0);
      if (!act) break;
      const unsigned k = M ? __builtin_ctz(M) : 0u;
      const unsigned pos = (k & 12u) * 128u + lane * 8u + (k & 3u) * 2u + (k >> 4);
      const unsigned v = *(LAS unsigned short*)(lds + wave * XSTG + pos * 2);
      const unsigned hi = (unsigned)(((int)((v >> 7) & 0xffu) - (int)base) >> 3) + 28u;
      const unsigned sl = cnt + __builtin_amdgcn_mbcnt_hi((unsigned)(act >> 32), __builtin_amdgcn_mbcnt_lo((unsigned)act, 0u));
      const unsigned slc = M != 0 && sl < EXC ? sl : 127u - 2u;
      *(LAS unsigned short*)(lds + eb + ((wave & 1) && slc < EXC ? PK_R - 6 - 2 * slc : 4 + 2 * slc)) = (unsigned short)(pos | (hi << 11));
      if (__builtin_amdgcn_ballot_w64(M != 0 && hi > 31u)) raw = true;
      cnt += (unsigned)__builtin_popcountll(act);
      M &= M - 1;
      if (cnt > EXC) { raw = true; break; }
    }
    {
      const unsigned nd = ((raw ? 0u : cnt) + 1u) / 2u + 1u;
      const int o = (wave & 1) ? PK_R - 4 - 4 * lane : 4 * lane;
      cfence();
      unsigned v = *(LAS unsigned*)(lds + eb + ((unsigned)lane < nd ? o : 0));
      cfence();
      lds_st(SY_PACK + 4 * wave, 1u);
      if (lane == 0) v = base | ((raw ? 255u : cnt) << 8);
      if ((unsigned)lane < nd) st4(pk + PK_E + (wave >> 1) * PK_R + o, v);
    }
    if (wave == 0) STAMP(57);
    waitv<0>();
    if (lds_ticket(SY_ARR + 4) == 3) {
      if (lane <= NR) {
        char* f = lane == rank || lane == NR ? me + XF_OFF + (size_t)(((rank * 2 + uh) * 2 + (lane >> 3)) * NSTEP + us) * 4
                               : (char*)*(LAS unsigned long long*)(lds + SY_PTR + lane * 8) + SF_OFF + (size_t)(rank * 256 + ub) * 4;
        st4(f, call);
      }
      STAMP(1);
    }
  }
  const unsigned long long t0 = rt();

  if (b < NPULL) {
    const char* tkp = me + TK_OFF + (call & 1) * 128;
    if (b == 0 && wave == 7 && lane == 0) st4(me + TK_OFF + ((call + 1) & 1) * 128, 0u);
    if (wave == 7) {
      const char* sf = me + SF_OFF + lane * 16;
      for (;;) {
        u32x4 f[8]; unsigned t;
        asm volatile("global_load_dwordx4 %0, %9, off sc0 sc1\n\tglobal_load_dwordx4 %1, %9, off offset:1024 sc0 sc1\n\t"
                     "global_load_dwordx4 %2, %9, off offset:2048 sc0 sc1\n\tglobal_load_dwordx4 %3, %9, off offset:3072 sc0 sc1\n\t"
                     "global_load_dwordx4 %4, %10, off sc0 sc1\n\tglobal_load_dwordx4 %5, %10, off offset:1024 sc0 sc1\n\t"
                     "global_load_dwordx4 %6, %10, off offset:2048 sc0 sc1\n\tglobal_load_dwordx4 %7, %10, off offset:3072 sc0 sc1\n\t"
                     "global_load_dword %8, %11, off sc0 sc1\n\ts_waitcnt vmcnt(0)"
                     : "=&v"(f[0]), "=&v"(f[1]), "=&v"(f[2]), "=&v"(f[3]), "=&v"(f[4]), "=&v"(f[5]), "=&v"(f[6]), "=&v"(f[7]), "=&v"(t)
                     : "v"(sf), "v"(sf + 4096), "v"(tmo) : "memory");
        unsigned all = 256;
#pragma unroll
        for (int j = 0; j < NR; ++j) {
          const unsigned c = f[j][0] < call ? 0u : f[j][1] < call ? 1u : f[j][2] < call ? 2u : f[j][3] < call ? 3u : 4u;
          const unsigned long long part = __builtin_amdgcn_ballot_w64(c < 4u);
          unsigned r = 256;
          if (part && j != rank) { const unsigned l = __builtin_ctzll(part); r = 4 * l + __builtin_amdgcn_readlane(c, l); }
          lds_st(SY_SRC + 4 * j, r);
          all = umin(all, r);
        }
        if (all == 256) break;
        if (expired(t, call, t0, me)) {
#pragma unroll
          for (int j = 0; j < NR; ++j) lds_st(SY_SRC + 4 * j, ABORT);
          break;
        }
      }
    }
    const unsigned v0 = lane * 16, vc = lane * 12, vh = (lane < 48 ? lane : 47) * 4;
    LAS char* img = lds + wave * XSTG;
    const char* fl = me + TK_OFF + 512;
    unsigned T = wave * NPULL + b;
    for (bool first = true; wave < NWP; first = false) {
      if (T >= 7u * 512u) break;
      const unsigned hu = T / 7u, src = (rank + 1 + (T - hu * 7u)) & 7u, u = hu >> 1, r = hu & 1, s = u >> 1, h = u & 1, n = u >> 2;
      unsigned av = rfl(lds_ld(SY_SRC + 4 * src));
      if (av <= u) {
        waitv<0>();
        flag_st(fl, call);
        fl = me + TK_OFF + 512;
        do { __builtin_amdgcn_s_sleep(1); av = rfl(lds_ld(SY_SRC + 4 * src)); } while (av <= u);
      }
      if (av >= ABORT) break;
      if (first) for (unsigned i = 0; i < hu; ++i) __builtin_amdgcn_s_sleep(STAG);
      if (n < 8) STAMP(8 + n);
      u32x4 A[4]; u32x3 P[2]; unsigned E, tk;
      const char* sws = p.ws[src];
      const char* sb = sws + PK_OFF + (size_t)u * PKU;
      asm volatile("s_nop 4\n\t"
                   "global_load_dword %6, %10, %13 sc0 sc1\n\t"
                   "global_load_dwordx3 %4, %9, %12 sc0 sc1\n\t"
                   "global_load_dwordx3 %5, %9, %12 offset:768 sc0 sc1\n\t"
                   "global_load_dwordx4 %0, %8, %11 sc0 sc1\n\t"
                   "global_load_dwordx4 %1, %8, %11 offset:1024 sc0 sc1\n\t"
                   "global_load_dwordx4 %2, %8, %11 offset:2048 sc0 sc1\n\t"
                   "global_load_dwordx4 %3, %8, %11 offset:3072 sc0 sc1\n\t"
                   "s_mov_b64 exec, 1\n\ts_nop 3\n\t"
                   "global_atomic_add %7, %8, %16, %14 sc0\n\t"
                   "s_waitcnt vmcnt(8)\n\t"
                   "global_store_dword %8, %17, %15 sc0 sc1\n\t"
                   "s_mov_b64 exec, -1\n\t"
                   "s_waitcnt vmcnt(1)"
                   : "=&v"(A[0]), "=&v"(A[1]), "=&v"(A[2]), "=&v"(A[3]), "=&v"(P[0]), "=&v"(P[1]), "=&v"(E), "=&v"(tk)
                   : "v"(v0), "v"(vc), "v"(vh), "s"(sb + r * 4096), "s"(sb + PK_C + r * 1536), "s"(sb + PK_E + r * PK_R),
                     "s"(tkp), "s"(fl), "v"(1u), "v"(call) : "memory");
      const size_t xo = XB_OFF + (size_t)((src * 2 + h) * NSTEP + s) * XSTG + r * 8192;
      char* xd = me + xo + lane * 16;
      const unsigned hd[2] = {(unsigned)__builtin_amdgcn_readlane(E, 0), (unsigned)__builtin_amdgcn_readlane(E, 47)};
      const unsigned c0 = (hd[0] >> 8) & 0xffu, c1 = (hd[1] >> 8) & 0xffu;
      const unsigned n0 = c0 > EXC ? 0u : c0, n1 = c1 > EXC ? 0u : c1;
      const bool clash = ((n0 + 1u) & ~1u) + ((n1 + 1u) & ~1u) > EXC;
      const bool raw[2] = {clash || c0 > EXC, clash || c1 > EXC};
      const unsigned ce = clash ? 0u : n0, co = clash ? 0u : n1;
#pragma unroll
      for (int q = 0; q < 2; ++q) {
        const unsigned B2 = (hd[q] & 0xffu) * 0x00800080u;
#pragma unroll
        for (int f = 0; f < 4; ++f) {
          u32x4 d;
#pragma unroll
          for (int j = 0; j < 4; ++j) {
            const int i = 4 * f + j, g = i / 5, rr = i % 5;
            const unsigned c = i == 15 ? ((P[q][0] & 0x00010001u) << 7) | ((P[q][1] & 0x00010001u) << 8) | ((P[q][2] & 0x00010001u) << 9)
                                       : (rr < 3 ? P[q][g] >> (3 * rr) : P[q][g] << (3 * rr - 6)) & 0x03800380u;
            const unsigned sm = A[2 * q + (f >> 1)][2 * (f & 1) + (j >> 1)];
            d[j] = (perm(sm, sm, (j & 1) ? 0x03030202u : 0x01010000u) & 0x807f807fu) | (c + B2);
          }
          *(LAS u32x4*)(img + q * 4096 + f * 1024 + lane * 16) = d;
        }
      }
      cfence();
#pragma unroll
      for (int c = 0; c < 2; ++c) {
        const unsigned ent = (E >> (16 * c)) & 0xffffu, hw = 2u * lane + c;
        const bool ev = hw - 2u < ce, od = (PK_R / 2 - 3) - hw < co;
        const unsigned dl = (ev || od) && lane < 48 ? ((ent >> 11) - 28u) << (10u + (ent & 1u) * 16u) : 0u;
        const unsigned ad = wave * XSTG + (od ? 4096u : 0u) + (ent & 0x7feu) * 2u;
        asm volatile("ds_add_u32 %0, %1" :: "v"(ad), "v"(dl) : "memory");
      }
      cfence();
#pragma unroll
      for (int q = 0; q < 2; ++q) {
        u32x4 R0, R1, R2, R3;
        if (raw[q]) {
          const char* rb = sws + xo + q * 4096;
          asm volatile("s_nop 4\n\t"
                       "global_load_dwordx4 %0, %4, %5 sc0 sc1\n\t"
                       "global_load_dwordx4 %1, %4, %5 offset:1024 sc0 sc1\n\t"
                       "global_load_dwordx4 %2, %4, %5 offset:2048 sc0 sc1\n\t"
                       "global_load_dwordx4 %3, %4, %5 offset:3072 sc0 sc1\n\t"
                       "s_waitcnt vmcnt(0)"
                       : "=&v"(R0), "=&v"(R1), "=&v"(R2), "=&v"(R3) : "v"(v0), "s"(rb) : "memory");
        } else {
          R0 = *(LAS u32x4*)(img + q * 4096 + lane * 16); R1 = *(LAS u32x4*)(img + q * 4096 + 1024 + lane * 16);
          R2 = *(LAS u32x4*)(img + q * 4096 + 2048 + lane * 16); R3 = *(LAS u32x4*)(img + q * 4096 + 3072 + lane * 16);
        }
        st16(xd + q * 4096, R0); st16(xd + q * 4096 + 1024, R1); st16(xd + q * 4096 + 2048, R2); st16(xd + q * 4096 + 3072, R3);
      }
      fl = me + XF_OFF + (size_t)(((src * 2 + h) * 2 + r) * NSTEP + s) * 4;
      if (n < 8) STAMP(16 + n);
      if ((n & 7) == 7) STAMP(48 + n / 8);
      T = umin(rfl(tk), 7u * 512u) + NWP * NPULL;
    }
    if (wave < NWP) { waitv<0>(); flag_st(fl, call); }
    if (wave < NWP) { STAMP(24 + wave); lds_st(SY_TKT, 64u); }
    if (b >= 8) return;
    if (wave >= NWP) {
      while (rfl(lds_ld(SY_TKT)) < 64u) __builtin_amdgcn_s_sleep(32);
    }
    const int lg = lane < NGEMM / 4 ? lane : NGEMM / 4 - 1;
    const char* g0 = me + GR_OFF + (size_t)(((call & 1) * NR + wave) * 256) * 8 + lg * 32;
    for (;;) {
      u32x4 a0, a1; unsigned t;
      asm volatile("global_load_dwordx4 %0, %3, off sc0 sc1\n\tglobal_load_dwordx4 %1, %3, off offset:16 sc0 sc1\n\t"
                   "global_load_dword %2, %4, off sc0 sc1\n\ts_waitcnt vmcnt(0)"
                   : "=&v"(a0), "=&v"(a1), "=&v"(t) : "v"(g0), "v"(tmo) : "memory");
      const bool ok = a0[1] == call && a0[3] == call && a1[1] == call && a1[3] == call;
      if (__builtin_amdgcn_ballot_w64(ok) == ~0ull) {
        const unsigned best = wave_umax(umax(umax(a0[0], a0[2]), umax(a1[0], a1[2])));
        asm volatile("ds_max_u32 %0, %1" :: "v"(SY_GMAX), "v"(best) : "memory");
        if (lds_ticket(SY_ARR) == 7u) {
          const unsigned gm = lds_ld(SY_GMAX);
          if (lane == 0)
            __hip_atomic_store((unsigned long long*)(me + GM_OFF + b * 128), ((unsigned long long)call << 32) | gm,
                               __ATOMIC_RELAXED, __HIP_MEMORY_SCOPE_SYSTEM);
          STAMP(4);
        }
        break;
      }
      if (expired(t, call, t0, me)) break;
      __builtin_amdgcn_s_sleep(1);
    }
    return;
  }
  if (b < NCOMM) {
    for (;;) {
      const int t = NGEMM * UPF + (b - NPULL) + NCONV * (int)lds_ticket(SY_TKT);
      if (t >= 16 * NSTEP) break;
      if ((t & 255) < NCONV) STAMP(8 + (t >> 8));
      convert_w(p, me, t & 15, t >> 4, lane, call);
    }
    if (wave == 0) STAMP(2);
    return;
  }

  const int xcd = b & 7, idx = (b >> 3) - NCOMM / 8, xh = xcd & 1;
  const bool extra = idx < 24;
  const int di = extra ? idx >> 2 : 6;
  const int pn = (xcd >> 1) * 4 + (extra ? idx & 3 : 2 * xh + (idx & 1));
  const int src = (rank + 1 + di) & 7, h = extra ? xh : 0;
  const int esrc = di < 4 ? rank : (rank + 7) & 7, eh = di < 4 ? xh : extra ? 1 : 0, eq = di < 4 ? di : 2 * xh + (di & 1);
  if (wave < 4) {
    __builtin_amdgcn_s_setprio(PRIOC);
    if (extra) consumer<5>(p, me, lds, lane, wave, call, b - NCOMM, src * 512 + h * 256, esrc * 512 + eh * 256 + eq * 64, pn, t0);
    else consumer<4>(p, me, lds, lane, wave, call, b - NCOMM, src * 512 + h * 256, 0, pn, t0);
    return;
  }
  bool dead = false;
  if (wave < 6) {
    __builtin_amdgcn_s_setprio(PRIOL);
    const char* xs = me + XB_OFF + (size_t)((src * 2 + h) * NSTEP) * XSTG;
    const char* ls = me + XB_OFF + (size_t)((esrc * 2 + eh) * NSTEP) * XSTG + eq * 4096;
    const char* wsr = me + WB_OFF + (size_t)(pn * NSTEP) * WSTG;
    const unsigned voff = lane * 16;
    unsigned slot = (wave - 4) * STG;
    const int fw = SY_FULL + 4 * (wave - 4);
    const char* tok = me + IO_OFF + (size_t)(b * 2 + (wave - 4)) * 256;
    st4((char*)tok + lane * 4, 2 * lane + (wave - 4) + 1);
    waitv<0>();
    for (;;) {
      u32x4 d;
      asm volatile("ds_read_b128 %0, %1\n\ts_waitcnt lgkmcnt(0)" : "=&v"(d) : "v"(SY_PACK) : "memory");
      if (rfl(d[0] & d[1] & d[2] & d[3])) break;
      __builtin_amdgcn_s_sleep(2);
    }
    for (int s = wave - 4; s < NSTEP; s += 2) {
      unsigned rdy;
      for (;;) {
        u32x4 d; unsigned r;
        asm volatile("ds_read_b128 %0, %2\n\tds_read_b32 %1, %3\n\ts_waitcnt lgkmcnt(0)" : "=&v"(d), "=&v"(r) : "v"(SY_DONE), "v"(SY_READY) : "memory");
        rdy = rfl(r);
        unsigned dm = rfl(umin(umin(d[0], d[1]), umin(d[2], d[3])));
        if (rdy >= ABORT || (rdy > (unsigned)s && dm + NS > (unsigned)s)) break;
        __builtin_amdgcn_s_sleep(SLEEPL);
      }
      if (rdy >= ABORT) { dead = true; break; }
      const int sa = s;
      const char* xa = xs + (size_t)sa * XSTG;
#define PACE_
      glds4(voff, wsr + (size_t)sa * WSTG, slot + WOF); PACE_ glds4(voff, wsr + (size_t)sa * WSTG + 4096, slot + WOF + 4096); PACE_
      glds4(voff, xa, slot); PACE_ glds4(voff, xa + 4096, slot + 5120); PACE_ glds4(voff, xa + 8192, slot + 10240); PACE_ glds4(voff, xa + 12288, slot + 15360); PACE_
      if (extra) glds4s(voff, ls + (size_t)sa * XSTG, slot + 4096);
      asm volatile("s_mov_b32 m0, %2\n\ts_mov_b64 exec, 1\n\ts_nop 1\n\tglobal_load_lds_dword %0, %1 sc0 sc1\n\ts_mov_b64 exec, -1"
                   :: "v"(voff), "s"(tok + (s >> 1) * 4), "s"(fw) : "memory");
      slot += 2 * STG; if (slot >= NS * STG) slot -= NS * STG;
    }
    waitv<0>();
    if (dead) lds_st(fw, ABORT);
    __builtin_amdgcn_s_setprio(0);
  } else if (wave == 6) {
    __builtin_amdgcn_s_setprio(PRIOP);
    const char* pa = lane < 32 ? me + XF_OFF + (size_t)((src * 2 + h) * 2 * NSTEP) * 4 + lane * 16
                               : me + WF_OFF + (size_t)(pn * NSTEP) * 4 + (lane & 31) * 16;
    const char* pb = me + XF_OFF + (size_t)((lane < 32 ? (esrc * 2 + eh) * 2 + (eq >> 1) : (src * 2 + h) * 2 + 1) * NSTEP) * 4 + (lane & 31) * 16;
    unsigned ready = 0;
    while (!dead && ready < (unsigned)NSTEP) {
      if (ready > rfl(lds_ld(SY_DONE)) + NS + 2) { __builtin_amdgcn_s_sleep(SLEEPP); continue; }
      u32x4 f, g; unsigned t;
      asm volatile("global_load_dwordx4 %0, %3, off sc0 sc1\n\tglobal_load_dwordx4 %1, %4, off sc0 sc1\n\t"
                   "global_load_dword %2, %5, off sc0 sc1\n\ts_waitcnt vmcnt(0)"
                   : "=&v"(f), "=&v"(g), "=&v"(t) : "v"(pa), "v"(pb), "v"(tmo) : "memory");
#pragma unroll
      for (int i = 0; i < 4; ++i) f[i] = umin(f[i], g[i]);
      unsigned c = f[0] < call ? 0u : f[1] < call ? 1u : f[2] < call ? 2u : f[3] < call ? 3u : 4u;
      unsigned long long part = __builtin_amdgcn_ballot_w64(c < 4u);
      unsigned px = (unsigned)part, pw = (unsigned)(part >> 32);
      unsigned rx = NSTEP, rw = NSTEP;
      if (px) { unsigned l = __builtin_ctz(px); rx = 4 * l + __builtin_amdgcn_readlane(c, l); }
      if (pw) { unsigned l = __builtin_ctz(pw); rw = 4 * l + __builtin_amdgcn_readlane(c, 32 + l); }
      unsigned r = umin(rx, rw);
      if (r > ready) {
        ready = r; lds_st(SY_READY, ready); continue; }
      if (expired(t, call, t0, me)) { dead = true; break; }
      __builtin_amdgcn_s_sleep(1);
    }
    if (dead) lds_st(SY_READY, ABORT);
    STAMP(3);
  }
}

}

extern "C" void kernel_launch(int rank, int nranks, ncclComm_t comm, hipStream_t stream,
                              void* const* d_in, const int* in_sizes, int n_in,
                              void* d_out, int out_size, const int* in_dtypes, int out_dtype,
                              void* const* d_out_all, void* const* d_ws_all, size_t ws_bytes) {
  if (nranks != NR || (unsigned)rank >= (unsigned)NR || n_in < 2 || ws_bytes < WS_NEED ||
      in_sizes[0] != 512 * KD || out_size != 4096 * NLOC) {
    fprintf(stderr, "kernel_launch: unexpected problem shape\n");
    return;
  }
  HIP_CHECK(hipFuncSetAttribute((const void*)fused, hipFuncAttributeMaxDynamicSharedMemorySize, LDS_BYTES));
  Params p;
  p.x = (const float*)d_in[0]; p.w = (const float*)d_in[1]; p.out = (char*)d_out; p.rank = rank;
  for (int i = 0; i < NR; ++i) p.ws[i] = (char*)d_ws_all[i];
  fused<<<NWG, 512, LDS_BYTES, stream>>>(p);
  HIP_CHECK(hipGetLastError());
}
```

```cpp
#include <hip/hip_runtime.h>
#include <rccl/rccl.h>
#include <cstdio>
#include <cstdint>

#define HIP_CHECK(expr) do { hipError_t _e = (expr); if (_e != hipSuccess) { \
    fprintf(stderr, "HIP error %s at %s:%d\n", hipGetErrorString(_e), __FILE__, __LINE__); \
    return; } } while (0)

namespace {

typedef unsigned u32x4 __attribute__((ext_vector_type(4)));
typedef unsigned u32x3 __attribute__((ext_vector_type(3)));
typedef unsigned u32x2 __attribute__((ext_vector_type(2)));
typedef unsigned short u16x2 __attribute__((ext_vector_type(2)));
typedef float f32x4 __attribute__((ext_vector_type(4)));
typedef float f32x2 __attribute__((ext_vector_type(2)));
typedef __bf16 bf16x8 __attribute__((ext_vector_type(8)));
typedef __bf16 bf16x2 __attribute__((ext_vector_type(2)));
#define LAS __attribute__((address_space(3)))

#define FORCE_RAW 0
#define UPF 1
#define PRIOC 3
#define PRIOL 0
#define PRIOP 0
#define SLEEPL 1
#define SLEEPP 4
#define STAG 0
#define LSC " sc0 sc1"
#define GNOP 2
#define GREP 1
#define STR2(x) #x
#define STR(x) STR2(x)
#define GAP ".rept " STR(GREP) "\n\ts_nop " STR(GNOP) "\n\t.endr\n\t"
#define NPUSH 12
#define PSTAG 0
#define OSC ""
#define POLL1 1
#define XDLY 40
#define CPULL 0
#define TST0 450
#define TSTD 10
#define UPC 0
#define NSPL 3
#define SPREAD 0
#define NWP 6

constexpr int NR = 8;
constexpr int KD = 4096, ND = 16384, NLOC = 2048;
constexpr int NSTEP = 128;
constexpr int NPULL = 28, NCOMM = 48, NCONV = NCOMM - NPULL, NGEMM = 208, NWG = 256;
constexpr int XSTG = 16384, WSTG = 8192;
constexpr int PK_C = 8192, PK_E = 11264, PK_R = 192, PKU = PK_E + 2 * PK_R;
constexpr unsigned EXC = (PK_R - 8) / 2;
constexpr size_t XB_OFF = 0;
constexpr size_t WB_OFF = 32u << 20;
constexpr size_t PK_OFF = 48u << 20;
constexpr size_t LD_OFF = 52u << 20;
constexpr size_t FL_OFF = 64u << 20;
constexpr size_t CNT_OFF = FL_OFF;
constexpr size_t TMO_OFF = FL_OFF + 0x4000;
constexpr size_t SF_OFF = FL_OFF + 0x5000;
constexpr size_t XF_OFF = FL_OFF + 0x60000;
constexpr size_t WF_OFF = FL_OFF + 0xA000;
constexpr size_t GM_OFF = FL_OFF + 0xC000;
constexpr size_t TK_OFF = FL_OFF + 0xD000;
constexpr size_t GR_OFF = FL_OFF + 0x10000;
constexpr size_t ST_OFF = FL_OFF + 0x20000;
constexpr size_t IO_OFF = FL_OFF + 0x40000;
constexpr size_t PF_OFF = FL_OFF + 0x64000;
constexpr size_t WS_NEED = FL_OFF + 0x74000;
constexpr int NS = 5;
constexpr int STG = 28672;
constexpr int SY = NS * STG;
constexpr int SY_FULL = SY, SY_READY = SY + 8, SY_DONE = SY + 16, SY_ARR = SY + 32, SY_WMAX = SY + 48,
              SY_GMAX = SY + 64, SY_TKT = SY + 84, SY_PACK = SY + 96, SY_SRC = SY + 112, SY_PTR = SY + 144;
constexpr int SCR = SY + 512;
constexpr int LDS_BYTES = SCR + 8192;
constexpr unsigned long long DEADLINE = 50000000ull;
constexpr unsigned ABORT = 0x40000000u;
static_assert(NPUSH % 4 == 0 && NPUSH * 14 + NWP * NPULL + CPULL * NCONV <= 64 * 14, "first round: within the PF flags");

struct Params { const float* x; const float* w; char* out; char* ws[NR]; int rank; };

__device__ __forceinline__ unsigned long long rt() { return __builtin_amdgcn_s_memrealtime(); }
template <int N_> __device__ __forceinline__ void waitv() { asm volatile("s_waitcnt vmcnt(%0)" :: "i"(N_) : "memory"); }
__device__ __forceinline__ void cfence() { asm volatile("" ::: "memory"); }
__device__ __forceinline__ void st16(char* p, u32x4 v) {
  asm volatile("global_store_dwordx4 %0, %1, off sc0 sc1\n\ts_nop 1" :: "v"(p), "v"(v) : "memory");
}
__device__ __forceinline__ void st4(char* p, unsigned v) {
  asm volatile("global_store_dword %0, %1, off sc0 sc1" :: "v"(p), "v"(v) : "memory");
}
__device__ __forceinline__ void flag_st(const char* p, unsigned v) {
  asm volatile("s_mov_b64 exec, 1\n\ts_nop 3\n\tglobal_store_dword %0, %1, %2 sc0 sc1\n\ts_mov_b64 exec, -1" :: "v"(0), "v"(v), "s"(p) : "memory");
}
__device__ __forceinline__ unsigned lds_ticket(int addr) {
  unsigned v, r;
  asm volatile("s_mov_b64 exec, 1\n\tds_add_rtn_u32 %0, %2, %3\n\ts_mov_b64 exec, -1\n\ts_waitcnt lgkmcnt(0)\n\tv_readfirstlane_b32 %1, %0"
               : "=&v"(v), "=s"(r) : "v"(addr), "v"(1u) : "memory");
  return r;
}
__device__ __forceinline__ void poll16(u32x4& a, unsigned& t, const char* p, const char* tm) {
  asm volatile("global_load_dwordx4 %0, %2, off sc0 sc1\n\tglobal_load_dword %1, %3, off sc0 sc1\n\ts_waitcnt vmcnt(0)"
               : "=&v"(a), "=&v"(t) : "v"(p), "v"(tm) : "memory");
}
__device__ __forceinline__ void lds_st(int addr, unsigned v) {
  asm volatile("ds_write_b32 %0, %1" :: "v"(addr), "v"(v) : "memory");
}
__device__ __forceinline__ unsigned lds_ld(int addr) {
  unsigned v; asm volatile("ds_read_b32 %0, %1\n\ts_waitcnt lgkmcnt(0)" : "=&v"(v) : "v"(addr) : "memory"); return v;
}
__device__ __forceinline__ unsigned lds_ld1(int addr) {
  unsigned v, r;
  asm volatile("s_mov_b64 exec, 1\n\tds_read_b32 %0, %2\n\ts_waitcnt lgkmcnt(0)\n\tv_readfirstlane_b32 %1, %0\n\ts_mov_b64 exec, -1"
               : "=&v"(v), "=s"(r) : "v"(addr) : "memory");
  return r;
}
__device__ __forceinline__ unsigned rfl(unsigned v) { return __builtin_amdgcn_readfirstlane(v); }
__device__ __forceinline__ unsigned umax(unsigned a, unsigned b) { return a > b ? a : b; }
__device__ __forceinline__ unsigned umin(unsigned a, unsigned b) { return a < b ? a : b; }
__device__ __forceinline__ unsigned wave_umax(unsigned v) {
  v = umax(v, (unsigned)__builtin_amdgcn_update_dpp(0, (int)v, 0x111, 0xf, 0xf, false));
  v = umax(v, (unsigned)__builtin_amdgcn_update_dpp(0, (int)v, 0x112, 0xf, 0xf, false));
  v = umax(v, (unsigned)__builtin_amdgcn_update_dpp(0, (int)v, 0x114, 0xf, 0xf, false));
  v = umax(v, (unsigned)__builtin_amdgcn_update_dpp(0, (int)v, 0x118, 0xf, 0xf, false));
  unsigned a = __builtin_amdgcn_readlane(v, 15), b = __builtin_amdgcn_readlane(v, 31);
  unsigned c = __builtin_amdgcn_readlane(v, 47), d = __builtin_amdgcn_readlane(v, 63);
  return umax(umax(a, b), umax(c, d));
}
__device__ __forceinline__ unsigned wave_fsum(float v) {
#define DPPADD(ctrl, rm) v += __builtin_bit_cast(float, __builtin_amdgcn_update_dpp(0, __builtin_bit_cast(int, v), ctrl, rm, 0xf, false))
  DPPADD(0x111, 0xf); DPPADD(0x112, 0xf); DPPADD(0x114, 0xf); DPPADD(0x118, 0xf); DPPADD(0x142, 0xa); DPPADD(0x143, 0xc);
  return __builtin_amdgcn_readlane(__builtin_bit_cast(unsigned, v), 63);
}
__device__ __forceinline__ unsigned bfi(unsigned m, unsigned a, unsigned b) { return (a & m) | (b & ~m); }
__device__ __forceinline__ unsigned pk_sub16(unsigned a, unsigned b) {
  return __builtin_bit_cast(unsigned, (u16x2)(__builtin_bit_cast(u16x2, a) - __builtin_bit_cast(u16x2, b)));
}
__device__ __forceinline__ unsigned pk_min16(unsigned a, unsigned b) {
  return __builtin_bit_cast(unsigned, __builtin_elementwise_min(__builtin_bit_cast(u16x2, a), __builtin_bit_cast(u16x2, b)));
}
__device__ __forceinline__ unsigned pk_bf16(float lo, float hi) {
  f32x2 f = {lo, hi};
  return __builtin_bit_cast(unsigned, __builtin_convertvector(f, bf16x2));
}
__device__ __forceinline__ unsigned perm(unsigned hi, unsigned lo, unsigned sel) { return __builtin_amdgcn_perm(hi, lo, sel); }
__device__ __forceinline__ unsigned rotr(unsigned v, unsigned n) { return __builtin_amdgcn_alignbit(v, v, n); }
__device__ __forceinline__ void glds4(unsigned voff, const char* sbase, unsigned dst) {
  asm volatile("s_mov_b32 m0, %2\n\ts_nop 0\n\t"
               "global_load_lds_dwordx4 %0, %1" LSC "\n\t" GAP
               "global_load_lds_dwordx4 %0, %1 offset:1024" LSC "\n\t" GAP
               "global_load_lds_dwordx4 %0, %1 offset:2048" LSC "\n\t" GAP
               "global_load_lds_dwordx4 %0, %1 offset:3072" LSC "\n\t" GAP
               :: "v"(voff), "s"(sbase), "s"(dst) : "memory");
}
__device__ __forceinline__ void glds4s(unsigned voff, const char* sbase, unsigned dst) {
  asm volatile("s_mov_b32 m0, %2\n\ts_nop 0\n\t"
               "global_load_lds_dwordx4 %0, %1" LSC "\n\t" GAP
               "s_add_u32 m0, m0, 4096\n\ts_nop 0\n\t"
               "global_load_lds_dwordx4 %0, %1 offset:1024" LSC "\n\t" GAP
               "s_add_u32 m0, m0, 4096\n\ts_nop 0\n\t"
               "global_load_lds_dwordx4 %0, %1 offset:2048" LSC "\n\t" GAP
               "s_add_u32 m0, m0, 4096\n\ts_nop 0\n\t"
               "global_load_lds_dwordx4 %0, %1 offset:3072" LSC "\n\t" GAP
               :: "v"(voff), "s"(sbase), "s"(dst) : "memory", "scc");
}
#define STAMP(i) do {} while (0)

__device__ __forceinline__ bool expired(unsigned tmo_seen, unsigned call, unsigned long long t0, char* me) {
  if (rfl(tmo_seen) == call) return true;
  if (rt() - t0 > DEADLINE) {
    __hip_atomic_store((unsigned*)(me + TMO_OFF), call, __ATOMIC_RELAXED, __HIP_MEMORY_SCOPE_AGENT);
    return true;
  }
  return false;
}

struct WRegs { f32x4 v[2][8]; };
__device__ __forceinline__ void w_load(WRegs& r, const Params& p, int panel, int s, int lane) {
  const float* wp = p.w + (size_t)(32 * s + 8 * (lane >> 4)) * ND + p.rank * NLOC + panel * 128 + 4 * (lane & 15);
#pragma unroll
  for (int hf = 0; hf < 2; ++hf)
#pragma unroll
    for (int j = 0; j < 8; ++j) r.v[hf][j] = __builtin_nontemporal_load((const f32x4*)(wp + (size_t)j * ND + hf * 64));
}
__device__ __forceinline__ void w_store(const WRegs& r, char* me, int panel, int s, int lane) {
  char* dst = me + WB_OFF + (size_t)(panel * NSTEP + s) * WSTG + lane * 16;
#pragma unroll
  for (int hf = 0; hf < 2; ++hf)
#pragma unroll
    for (int c = 0; c < 4; ++c) {
      u32x4 o = {pk_bf16(r.v[hf][0][c], r.v[hf][1][c]), pk_bf16(r.v[hf][2][c], r.v[hf][3][c]),
                 pk_bf16(r.v[hf][4][c], r.v[hf][5][c]), pk_bf16(r.v[hf][6][c], r.v[hf][7][c])};
      st16(dst + (4 * hf + c) * 1024, o);
    }
}
__device__ __forceinline__ void w_flag(char* me, int panel, int s, int lane, unsigned call) {
  waitv<0>();
  flag_st(me + WF_OFF + (size_t)(panel * NSTEP + s) * 4, call);
}
__device__ __forceinline__ void convert_w(const Params& p, char* me, int panel, int s, int lane, unsigned call) {
  WRegs r;
  w_load(r, p, panel, s, lane);
  w_store(r, me, panel, s, lane);
  w_flag(me, panel, s, lane, call);
}

template <bool PUSH>
__device__ __forceinline__ void pack(const Params& p, const f32x4 (&xv)[8], char* me, LAS char* lds, int lane, int wave,
                                     unsigned call, int ub, int part) {
  const int rank = p.rank, us = ub >> 1, uh = ub & 1;
  float mx = 0.f, sm = 0.f;
  u32x4 D[4];
  {
    LAS char* tw = lds + wave * XSTG + (lane >> 3) * 16 + ((lane & 7) >> 1) * 256 + (lane & 1) * 8;
#pragma unroll
    for (int i = 0; i < 8; ++i) {
      mx = fmaxf(fmaxf(mx, fmaxf(fabsf(xv[i][0]), fabsf(xv[i][1]))), fmaxf(fabsf(xv[i][2]), fabsf(xv[i][3])));
      sm += fabsf(xv[i][i & 3]);
      *(LAS u32x2*)(tw + (i >> 1) * 1024 + (i & 1) * 128) = u32x2{pk_bf16(xv[i][0], xv[i][1]), pk_bf16(xv[i][2], xv[i][3])};
    }
    if (wave == 0 && !PUSH) STAMP(56);
    cfence();
#pragma unroll
    for (int f = 0; f < 4; ++f) D[f] = *(LAS u32x4*)(lds + wave * XSTG + f * 1024 + lane * 16);
  }
  char* xl = me + XB_OFF + (size_t)((rank * 2 + uh) * NSTEP + us) * XSTG + wave * 4096 + lane * 16;
#pragma unroll
  for (int f = 0; f < 4; ++f) if (!PUSH) st16(xl + f * 1024, D[f]);
  const unsigned emax = wave_umax((pk_bf16(mx, 0.f) >> 7) & 0xffu);
  const unsigned emean = (wave_fsum(sm * (3.32f / 512.f)) >> 23) & 0xffu;
  const unsigned etop = umin(emax, emean);
  const unsigned base = etop > 7u ? etop - 7u : 0u;
  const unsigned B2 = base * 0x00800080u;
  const int j0 = ub < NPUSH ? 1 + part * 7 / NSPL : 0, j1 = ub < NPUSH ? 1 + (part + 1) * 7 / NSPL : 1;
  auto pkd = [&](int j) -> char* {
    return j ? p.ws[(rank + 1 + (unsigned)(j + ub) % 7u) & 7u] + LD_OFF + (size_t)(rank * NPUSH + ub) * PKU : me + PK_OFF + (size_t)ub * PKU;
  };
  {
    unsigned S[8];
#pragma unroll
    for (int i = 0; i < 8; ++i) {
      const unsigned d0 = D[i >> 1][2 * (i & 1)], d1 = D[i >> 1][2 * (i & 1) + 1];
      S[i] = bfi(0x80808080u, perm(d1, d0, 0x07050301u), perm(d1, d0, 0x06040200u));
    }
    if (PSTAG && ub < NPUSH) for (int i = 0; i < ub; ++i) __builtin_amdgcn_s_sleep(PSTAG);
    for (int j = j0; j < j1; ++j) {
      char* pk = pkd(j);
      st16(pk + wave * 2048 + lane * 16, u32x4{S[0], S[1], S[2], S[3]});
      st16(pk + wave * 2048 + 1024 + lane * 16, u32x4{S[4], S[5], S[6], S[7]});
    }
  }
  unsigned C[16], Ma = 0, Mb = 0;
#pragma unroll
  for (int i = 0; i < 16; ++i) {
    const unsigned t = pk_sub16(D[i >> 2][i & 3] & 0x7f807f80u, B2);
    C[i] = t & 0x03800380u;
    const unsigned y = pk_min16(t, 0x04000400u);
    if (i < 8) Ma = bfi(0x04000400u, y, Ma >> 1); else Mb = bfi(0x04000400u, y, Mb >> 1);
  }
  {
    u32x3 P;
#pragma unroll
    for (int g = 0; g < 3; ++g)
      P[g] = C[5 * g] | (C[5 * g + 1] << 3) | (C[5 * g + 2] << 6) | (((C[5 * g + 3] << 3) | C[5 * g + 4]) >> 6) | ((C[15] >> (7 + g)) & 0x00010001u);
    for (int j = j0; j < j1; ++j)
      asm volatile("global_store_dwordx3 %0, %1, off sc0 sc1\n\ts_nop 1" :: "v"(pkd(j) + PK_C + wave * 768 + lane * 12), "v"(P) : "memory");
  }
  unsigned M = (Ma >> 3) | (Mb << 5), cnt = 0;
  bool raw = FORCE_RAW;
  const int eb = wave * XSTG + 4096;
  for (;;) {
    const unsigned long long act = __builtin_amdgcn_ballot_w64(M != 0);
    if (!act) break;
    const unsigned k = M ? __builtin_ctz(M) : 0u;
    const unsigned pos = (k & 12u) * 128u + lane * 8u + (k & 3u) * 2u + (k >> 4);
    const unsigned v = *(LAS unsigned short*)(lds + wave * XSTG + pos * 2);
    const unsigned hi = (unsigned)(((int)((v >> 7) & 0xffu) - (int)base) >> 3) + 28u;
    const unsigned sl = cnt + __builtin_amdgcn_mbcnt_hi((unsigned)(act >> 32), __builtin_amdgcn_mbcnt_lo((unsigned)act, 0u));
    const unsigned slc = M != 0 && sl < EXC ? sl : 127u - 2u;
    *(LAS unsigned short*)(lds + eb + ((wave & 1) && slc < EXC ? PK_R - 6 - 2 * slc : 4 + 2 * slc)) = (unsigned short)(pos | (hi << 11));
    if (__builtin_amdgcn_ballot_w64(M != 0 && hi > 31u)) raw = true;
    cnt += (unsigned)__builtin_popcountll(act);
    M &= M - 1;
    if (cnt > EXC) { raw = true; break; }
  }
  {
    const unsigned nd = ((raw ? 0u : cnt) + 1u) / 2u + 1u;
    const int o = (wave & 1) ? PK_R - 4 - 4 * lane : 4 * lane;
    cfence();
    unsigned v = *(LAS unsigned*)(lds + eb + ((unsigned)lane < nd ? o : 0));
    cfence();
    if (!PUSH) lds_st(SY_PACK + 4 * wave, 1u);
    if (lane == 0) v = base | ((raw ? 255u : cnt) << 8);
    if ((unsigned)lane < nd)
      for (int j = j0; j < j1; ++j) st4(pkd(j) + PK_E + (wave >> 1) * PK_R + o, v);
  }
  if (wave == 0 && !PUSH) STAMP(57);
  waitv<0>();
  if (lds_ticket(SY_ARR + (PUSH ? 12 : 4)) == 3) {
    const int j = lane - 16, pr = lane < NR ? lane : (rank + 1 + (unsigned)(j + ub) % 7u) & 7u;
    if ((!PUSH && lane <= NR) || (ub < NPUSH ? j >= j0 && j < j1 : ub < 64 && j >= 1 && j < NR)) {
      char* f = lane == rank || lane == NR ? me + XF_OFF + (size_t)(((rank * 2 + uh) * 2 + (lane >> 3)) * NSTEP + us) * 4
                             : (char*)*(LAS unsigned long long*)(lds + SY_PTR + pr * 8)
                               + (lane < NR ? SF_OFF + (size_t)(rank * 256 + ub) * 4 : PF_OFF + (size_t)(rank * 64 + ub) * 128);
      st4(f, call);
    }
    STAMP(PUSH ? 6 : 1);
  }
}

__device__ __forceinline__ void decode(const u32x4 (&A)[4], const u32x3 (&P)[2], unsigned E, LAS char* lds, int io,
                                       char* dst, const char* rawp, int lane) {
  char* xd = dst + lane * 16;
  LAS char* img = lds + io;
  const unsigned v0 = lane * 16;
  const unsigned hd[2] = {(unsigned)__builtin_amdgcn_readlane(E, 0), (unsigned)__builtin_amdgcn_readlane(E, 47)};
  const unsigned c0 = (hd[0] >> 8) & 0xffu, c1 = (hd[1] >> 8) & 0xffu;
  const unsigned n0 = c0 > EXC ? 0u : c0, n1 = c1 > EXC ? 0u : c1;
  const bool clash = ((n0 + 1u) & ~1u) + ((n1 + 1u) & ~1u) > EXC;
  const bool raw[2] = {clash || c0 > EXC, clash || c1 > EXC};
  const unsigned ce = clash ? 0u : n0, co = clash ? 0u : n1;
#pragma unroll
  for (int q = 0; q < 2; ++q) {
    const unsigned B2 = (hd[q] & 0xffu) * 0x00800080u;
#pragma unroll
    for (int f = 0; f < 4; ++f) {
      u32x4 d;
#pragma unroll
      for (int j = 0; j < 4; ++j) {
        const int i = 4 * f + j, g = i / 5, rr = i % 5;
        const unsigned c = i == 15 ? ((P[q][0] & 0x00010001u) << 7) | ((P[q][1] & 0x00010001u) << 8) | ((P[q][2] & 0x00010001u) << 9)
                                   : (rr < 3 ? P[q][g] >> (3 * rr) : P[q][g] << (3 * rr - 6)) & 0x03800380u;
        const unsigned sm = A[2 * q + (f >> 1)][2 * (f & 1) + (j >> 1)];
        d[j] = (perm(sm, sm, (j & 1) ? 0x03030202u : 0x01010000u) & 0x807f807fu) | (c + B2);
      }
      *(LAS u32x4*)(img + q * 4096 + f * 1024 + lane * 16) = d;
    }
  }
  cfence();
#pragma unroll
  for (int c = 0; c < 2; ++c) {
    const unsigned ent = (E >> (16 * c)) & 0xffffu, hw = 2u * lane + c;
    const bool ev = hw - 2u < ce, od = (PK_R / 2 - 3) - hw < co;
    const unsigned dl = (ev || od) && lane < 48 ? ((ent >> 11) - 28u) << (10u + (ent & 1u) * 16u) : 0u;
    const unsigned ad = io + (od ? 4096u : 0u) + (ent & 0x7feu) * 2u;
    asm volatile("ds_add_u32 %0, %1" :: "v"(ad), "v"(dl) : "memory");
  }
  cfence();
#pragma unroll
  for (int q = 0; q < 2; ++q) {
    u32x4 R0, R1, R2, R3;
    if (raw[q]) {
      const char* rb = rawp + q * 4096;
      asm volatile("s_nop 4\n\t"
                   "global_load_dwordx4 %0, %4, %5 sc0 sc1\n\t"
                   "global_load_dwordx4 %1, %4, %5 offset:1024 sc0 sc1\n\t"
                   "global_load_dwordx4 %2, %4, %5 offset:2048 sc0 sc1\n\t"
                   "global_load_dwordx4 %3, %4, %5 offset:3072 sc0 sc1\n\t"
                   "s_waitcnt vmcnt(0)"
                   : "=&v"(R0), "=&v"(R1), "=&v"(R2), "=&v"(R3) : "v"(v0), "s"(rb) : "memory");
    } else {
      R0 = *(LAS u32x4*)(img + q * 4096 + lane * 16); R1 = *(LAS u32x4*)(img + q * 4096 + 1024 + lane * 16);
      R2 = *(LAS u32x4*)(img + q * 4096 + 2048 + lane * 16); R3 = *(LAS u32x4*)(img + q * 4096 + 3072 + lane * 16);
    }
    st16(xd + q * 4096, R0); st16(xd + q * 4096 + 1024, R1); st16(xd + q * 4096 + 2048, R2); st16(xd + q * 4096 + 3072, R3);
  }
}

#define MFMA(m, t) asm volatile("v_mfma_f32_16x16x32_bf16 %0, %1, %2, %0" : "+v"(acc[m][t]) : "v"(Wf[t]), "v"(Xc[m]))
#define LDR(dst, addr, off) asm volatile("ds_read_b128 %0, %1 offset:%2" : "=v"(dst) : "v"(addr), "i"(off) : "memory")
template <int N_> __device__ __forceinline__ void waitl() { asm volatile("s_waitcnt lgkmcnt(%0)" :: "i"(N_) : "memory"); }
constexpr int WOF = 20480;
#define MAX2(m, t) asm volatile("v_max3_f32 %0, %2, %3, %0\n\tv_max3_f32 %1, %4, %5, %1" : "+v"(mx0), "+v"(mx1) \
                                : "v"(acc[m][t][0]), "v"(acc[m][t][1]), "v"(acc[m][t][2]), "v"(acc[m][t][3]))
template <int NM, int MODE, int PAR>
__device__ __forceinline__ void mstep(f32x4 (&acc)[NM][8], bf16x8 (&Wf)[8], bf16x8 (&Xc)[NM], bf16x8 (&Xn)[NM],
                                      unsigned& va, unsigned& vx, unsigned va0, unsigned vx0, unsigned vsy, unsigned& vdone,
                                      unsigned& slot, unsigned& fpre, int s, float& mx0, float& mx1, unsigned& nslow) {
  constexpr bool PF = MODE < 2;
  unsigned sf = 0, nslot = 0;
  constexpr int NX2 = 2 + (NM == 5) + (MODE == 0);
  waitl<5>();
  MFMA(0, 0);
  LDR(Wf[7], va, WOF + 7168);
  MFMA(1, 0);
  if (PF) asm volatile("ds_write_b32 %0, %1 offset:16" :: "v"(vsy), "v"(vdone) : "memory");
  MFMA(2, 0);
  if (PF) asm volatile("v_readfirstlane_b32 %0, %1" : "=s"(sf) : "v"(fpre) : "memory");
  MFMA(3, 0);
  if (PF) asm volatile("v_add_u32 %0, %1, %2" : "=v"(va) : "s"(slot), "v"(va0));
  if (NM == 5) MFMA(NM - 1, 0);
  if (PF) asm volatile("v_add_u32 %0, %1, %2" : "=v"(vx) : "s"(slot), "v"(vx0));
  if (PF && __builtin_expect(sf <= (unsigned)(s + 1), 0)) {
    do __builtin_amdgcn_s_sleep(1); while (lds_ld1(SY_FULL + 4 * (1 - PAR)) <= (unsigned)(s + 1));
  }
#pragma unroll
  for (int t = 1; t < 8; ++t) {
    if (t == 2) waitl<PF ? 10 : 5>();
    if (t == 3) waitl<PF ? 9 + NX2 : 4>();
    if (t == 4) waitl<PF ? 9 + NX2 : 3>();
    if (t == 5) waitl<PF ? 9 + NX2 : 2>();
    if (t == 6) waitl<PF ? 9 + NX2 : 1>();
    if (t == 7) waitl<PF ? 9 + NX2 : 0>();
    MFMA(0, t);
    if (PF) LDR(Wf[t - 1], va, WOF + (t - 1) * 1024); else MAX2(0, t - 1);
    MFMA(1, t);
    if (PF && t == 1) LDR(Xn[0], vx, 0);
    if (PF && t == 2) LDR(Xn[3], vx, 3 * 1024);
    if (MODE == 0 && t == 5) asm volatile("s_add_u32 %0, %1, %2" : "=s"(nslot) : "s"(slot), "i"(STG) : "scc");
    if (MODE == 0 && t == 6) asm volatile("s_cmp_lg_u32 %1, %2\n\ts_cselect_b32 %0, %1, 0" : "=s"(slot) : "s"(nslot), "i"(NS * STG) : "scc");
    if (!PF) MAX2(1, t - 1);
    MFMA(2, t);
    if (PF && t == 1) LDR(Xn[1], vx, 1024);
    if (PF && t == 2 && NM == 5) LDR(Xn[NM - 1], vx, 4 * 1024);
    if (PF && t == 4) asm volatile("v_add_u32 %0, 1, %0" : "+v"(vdone));
    if (!PF) MAX2(2, t - 1);
    MFMA(3, t);
    if (PF && t == 1) LDR(Xn[2], vx, 2 * 1024);
    if (MODE == 0 && t == 2) asm volatile("ds_read_b32 %0, %1 offset:%2" : "=v"(fpre) : "v"(SY_FULL), "i"(4 * PAR) : "memory");
    if (!PF) MAX2(3, t - 1);
    if (NM == 5) { MFMA(NM - 1, t); if (!PF) MAX2(NM - 1, t - 1); }
  }
  if (!PF) {
    asm volatile("s_nop 7\n\ts_nop 7" ::: "memory");
#pragma unroll
    for (int m = 0; m < NM; ++m) MAX2(m, 7);
  }
}

template <int NM>
__device__ __forceinline__ void consumer(const Params& p, char* me, LAS char* lds, int lane, int wave, unsigned call,
                                         int gidx, int row0, int row4, int pn, unsigned long long t0) {
  f32x4 acc[NM][8];
#pragma unroll
  for (int m = 0; m < NM; ++m)
#pragma unroll
    for (int t = 0; t < 8; ++t) { acc[m][t] = f32x4{0.f, 0.f, 0.f, 0.f}; asm volatile("" : "+v"(acc[m][t])); }
  bf16x8 Wf[8], Xa[NM], Xb[NM];
  unsigned slot = STG, fpre, vdone = 1;
  float mx0 = 0.f, mx1 = 0.f;
  unsigned nslow = 0;
  const unsigned va0 = lane * 16, vx0 = lane * 16 + wave * 5120, vsy = SY + 4 * wave;
  unsigned va = va0, vx = vx0;
  while (lds_ld1(SY_FULL) == 0) __builtin_amdgcn_s_sleep(1);
  STAMP(8 + wave);
#pragma unroll
  for (int t = 0; t < 7; ++t) LDR(Wf[t], va, WOF + t * 1024);
#pragma unroll
  for (int t = 0; t < NM; ++t) LDR(Xa[t], vx, t * 1024);
  asm volatile("ds_read_b32 %0, %1 offset:4\n\ts_waitcnt lgkmcnt(0)" : "=v"(fpre) : "v"(SY_FULL) : "memory");
  for (int s = 0; s < NSTEP - 2; s += 2) {
    mstep<NM, 0, 0>(acc, Wf, Xa, Xb, va, vx, va0, vx0, vsy, vdone, slot, fpre, s, mx0, mx1, nslow);
    mstep<NM, 0, 1>(acc, Wf, Xb, Xa, va, vx, va0, vx0, vsy, vdone, slot, fpre, s + 1, mx0, mx1, nslow);
  }
  mstep<NM, 1, 0>(acc, Wf, Xa, Xb, va, vx, va0, vx0, vsy, vdone, slot, fpre, NSTEP - 2, mx0, mx1, nslow);
  mstep<NM, 2, 1>(acc, Wf, Xb, Xa, va, vx, va0, vx0, vsy, vdone, slot, fpre, NSTEP - 1, mx0, mx1, nslow);
  STAMP(12 + wave);
  unsigned wm = wave_umax(umax(__builtin_bit_cast(unsigned, mx0), __builtin_bit_cast(unsigned, mx1)));
  asm volatile("ds_max_u32 %0, %1" :: "v"(SY_WMAX), "v"(wm) : "memory");
  if (lds_ticket(SY_ARR + 8) == 3) {
    unsigned m4 = lds_ld(SY_WMAX);
    if (lane < NR) {
      char* g = (char*)*(LAS unsigned long long*)(lds + SY_PTR + lane * 8) + GR_OFF + (size_t)(((call & 1) * NR + p.rank) * 256 + gidx) * 8;
      asm volatile("global_store_dwordx2 %0, %1, off sc0 sc1\n\ts_nop 1" :: "v"(g), "v"(u32x2{m4, call}) : "memory");
    }
  }
#pragma unroll
  for (int m = 0; m < NM; ++m)
#pragma unroll
    for (int t = 0; t < 8; ++t)
#pragma unroll
      for (int j = 0; j < 4; ++j) asm volatile("v_max_f32 %0, 0, %0" : "+v"(acc[m][t][j]));
  bool bad = false;
  float gmax;
  {
    const char* gp = me + GM_OFF + (blockIdx.x & 7) * 128;
    const char* tm = me + TMO_OFF;
    for (;;) {
      u32x2 g; unsigned t;
      asm volatile("global_load_dwordx2 %0, %2, %3 sc0 sc1\n\tglobal_load_dword %1, %2, %4 sc0 sc1\n\ts_waitcnt vmcnt(0)"
                   : "=&v"(g), "=&v"(t) : "v"(0), "s"(gp), "s"(tm) : "memory");
      gmax = __builtin_bit_cast(float, rfl(g[0]));
      if (rfl(t) == call) { bad = true; break; }
      if (rfl(g[1]) == call) break;
      if (expired(t, call, t0, me)) { bad = true; break; }
      __builtin_amdgcn_s_sleep(1);
    }
  }
  const float inv = bad ? 0.f : gmax > 0.f ? 127.0f / gmax : 0.f, mag = bad ? 8388736.0f : 8388608.0f;
  const f32x2 inv2 = {inv, inv}, mag2 = {mag, mag};
  STAMP(16 + wave);
  char* ob = p.out + pn * 128 + 16 * (lane >> 4);
#pragma unroll
  for (int m = 0; m < NM; ++m) {
    int row = (m < 4 ? row0 + 64 * wave + 16 * m : row4 + 16 * wave) + (lane & 15);
#pragma unroll
    for (int hf = 0; hf < 2; ++hf) {
      u32x4 o;
      u32x2 q[4][2];
#pragma unroll
      for (int c = 0; c < 4; ++c)
#pragma unroll
        for (int jj = 0; jj < 2; ++jj)
          q[c][jj] = __builtin_bit_cast(u32x2, __builtin_elementwise_fma(f32x2{acc[m][4 * hf + c][2 * jj], acc[m][4 * hf + c][2 * jj + 1]}, inv2, mag2));
#pragma unroll
      for (int j = 0; j < 4; ++j)
        o[j] = perm(perm(q[3][j >> 1][j & 1], q[2][j >> 1][j & 1], 0x0c0c0400u), perm(q[1][j >> 1][j & 1], q[0][j >> 1][j & 1], 0x0c0c0400u), 0x05040100u);
      asm volatile("global_store_dwordx4 %0, %1, off" OSC "\n\ts_nop 1" :: "v"(ob + (size_t)row * NLOC + 64 * hf), "v"(o) : "memory");
    }
  }
  STAMP(20 + wave);
}

__global__ __launch_bounds__(512, 2) void fused(Params p) {
  extern __shared__ char smem[];
  LAS char* lds = (LAS char*)smem;
  const int lane = threadIdx.x & 63;
  const int wave = __builtin_amdgcn_readfirstlane(threadIdx.x >> 6);
  const int b = blockIdx.x;
  const int rank = p.rank;
  char* me = p.ws[rank];
  const char* tmo = me + TMO_OFF;
  if (wave == 0) STAMP(0);
  const unsigned long long ts = rt();

  unsigned* cntp = (unsigned*)(me + CNT_OFF + b * 64);
  unsigned cnt0, cnt1;
  asm volatile("global_load_dword %0, %1, off sc0 sc1" : "=&v"(cnt0) : "v"(cntp) : "memory");
#define GETCNT(N) asm volatile("s_waitcnt vmcnt(" #N ")\n\tv_readfirstlane_b32 %0, %1" : "=s"(cnt1) : "v"(cnt0) : "memory")
  const int wt0 = b - NCOMM + NGEMM * (7 - wave);
  const bool wup = b >= NCOMM && wave >= 8 - UPF;
  f32x4 xv[8], xw[8];
  WRegs w0;
  const int rb = NWG - 1 - b;
  const int ub = rb < NPUSH ? rb : b < NPULL ? b + NPUSH : b < NCOMM ? b + NGEMM : b - NCONV + NPUSH;
  const bool pusher = rb >= NPUSH && rb < NPUSH * NSPL;
  const int pu = NPUSH ? rb % (NPUSH + !NPUSH) : 0, pj = NPUSH ? rb / (NPUSH + !NPUSH) : 0;
  if (XDLY && rb >= NPUSH * NSPL && b >= NPULL) __builtin_amdgcn_s_sleep(XDLY);
  if (wave < 4) {
    const float* xp = p.x + (size_t)(64 * wave + (lane >> 3)) * KD + 4 * (lane & 7);
    if (pusher) {
#pragma unroll
      for (int i = 0; i < 8; ++i) xw[i] = *(const f32x4*)(xp + (size_t)(256 * (pu & 1) + i * 8) * KD + 32 * (pu >> 1));
    }
#pragma unroll
    for (int i = 0; i < 8; ++i) xv[i] = __builtin_nontemporal_load((const f32x4*)(xp + (size_t)(256 * (ub & 1) + i * 8) * KD + 32 * (ub >> 1)));
    if (pusher) GETCNT(16); else GETCNT(8);
  } else if (wup) {
    w_load(w0, p, wt0 & 15, wt0 >> 4, lane);
    GETCNT(16);
  } else {
    if (wave == 4 && lane < 36) lds_st(SY + lane * 4, 0u);
    if (wave == 5 && lane < NR) *(LAS unsigned long long*)(lds + SY_PTR + lane * 8) = (unsigned long long)p.ws[lane];
    GETCNT(0);
  }
  const unsigned call = cnt1 + 1u;
  asm volatile("s_waitcnt lgkmcnt(0)" ::: "memory");
  __builtin_amdgcn_s_barrier();
  cfence();
  if (wave == 7 && lane == 0) __hip_atomic_store(cntp, call, __ATOMIC_RELAXED, __HIP_MEMORY_SCOPE_SYSTEM);
  if (wup) {
    w_store(w0, me, wt0 & 15, wt0 >> 4, lane);
    w_flag(me, wt0 & 15, wt0 >> 4, lane, call);
    if (wave == 7) STAMP(2);
  }
  if (wave < 4) {
    if (pusher) pack<true>(p, xw, me, lds, lane, wave, call, pu, pj);
    pack<false>(p, xv, me, lds, lane, wave, call, ub, 0);
  }
  __builtin_amdgcn_s_waitcnt(0x0f70);
  const unsigned long long t0 = rt();

  if (b < NCOMM) {
    const bool conv = b >= NPULL;
    const char* tkp = me + TK_OFF + (conv ? 640 : (call & 1) * 128);
    if (b == 0 && wave == 7 && lane == 0) st4(me + TK_OFF + ((call + 1) & 1) * 128, 0u);
    if (wave == 7 && !conv) {
      const char* sf = me + SF_OFF + lane * 16;
      for (;;) {
        u32x4 f[8]; unsigned t;
        asm volatile("global_load_dwordx4 %0, %9, off sc0 sc1\n\tglobal_load_dwordx4 %1, %9, off offset:1024 sc0 sc1\n\t"
                     "global_load_dwordx4 %2, %9, off offset:2048 sc0 sc1\n\tglobal_load_dwordx4 %3, %9, off offset:3072 sc0 sc1\n\t"
                     "global_load_dwordx4 %4, %10, off sc0 sc1\n\tglobal_load_dwordx4 %5, %10, off offset:1024 sc0 sc1\n\t"
                     "global_load_dwordx4 %6, %10, off offset:2048 sc0 sc1\n\tglobal_load_dwordx4 %7, %10, off offset:3072 sc0 sc1\n\t"
                     "global_load_dword %8, %11, off sc0 sc1\n\ts_waitcnt vmcnt(0)"
                     : "=&v"(f[0]), "=&v"(f[1]), "=&v"(f[2]), "=&v"(f[3]), "=&v"(f[4]), "=&v"(f[5]), "=&v"(f[6]), "=&v"(f[7]), "=&v"(t)
                     : "v"(sf), "v"(sf + 4096), "v"(tmo) : "memory");
        unsigned all = 256;
#pragma unroll
        for (int j = 0; j < NR; ++j) {
          const unsigned c = f[j][0] < call ? 0u : f[j][1] < call ? 1u : f[j][2] < call ? 2u : f[j][3] < call ? 3u : 4u;
          const unsigned long long part = __builtin_amdgcn_ballot_w64(c < 4u && lane >= NPUSH / 4);
          unsigned r = 256;
          if (part && j != rank) { const unsigned l = __builtin_ctzll(part); r = 4 * l + __builtin_amdgcn_readlane(c, l); }
          lds_st(SY_SRC + 4 * j, r);
          all = umin(all, r);
        }
        if (all == 256) break;
        if (expired(t, call, t0, me)) {
#pragma unroll
          for (int j = 0; j < NR; ++j) lds_st(SY_SRC + 4 * j, ABORT);
          break;
        }
      }
    }
    const unsigned v0 = lane * 16, vc = lane * 12, vh = (lane < 48 ? lane : 47) * 4;
    const char* fl = me + TK_OFF + 512;
    unsigned T = NPUSH * 14 + (conv ? NWP * NPULL + wave * NCONV + (b - NPULL) : wave * NPULL + b);
    const int nw = conv ? CPULL : NWP;
    for (bool first = true; wave < nw; first = false) {
      if (T >= 7u * 512u) break;
      const unsigned hu = T / 7u, src = (rank + 1 + (T - hu * 7u)) & 7u, u = hu >> 1, r = hu & 1, s = u >> 1, h = u & 1, n = u >> 2;
      unsigned av = lds_ld1(SY_SRC + 4 * src);
      if (first) {
        while (rt() - ts < (unsigned long long)(TST0 + TSTD * (hu - 2 * NPUSH))) __builtin_amdgcn_s_sleep(1);
        const char* fp = me + PF_OFF + (size_t)(src * 64 + u) * 128;
        for (;;) {
          unsigned f, t;
          asm volatile("global_load_dword %0, %2, %3 sc0 sc1\n\tglobal_load_dword %1, %2, %4 sc0 sc1\n\ts_waitcnt vmcnt(0)"
                       : "=&v"(f), "=&v"(t) : "v"(0), "s"(fp), "s"(tmo) : "memory");
          av = u + 1;
          if (rfl(f) >= call) break;
          av = ABORT;
          if (expired(t, call, t0, me)) break;
          __builtin_amdgcn_s_sleep(1);
        }
      }
      if (av <= u) {
        waitv<0>();
        flag_st(fl, call);
        fl = me + TK_OFF + 512;
        do { __builtin_amdgcn_s_sleep(1); av = lds_ld1(SY_SRC + 4 * src); } while (av <= u);
      }
      if (av >= ABORT) break;
      if (STAG && first) for (unsigned i = 2 * NPUSH; i < hu; ++i) __builtin_amdgcn_s_sleep(STAG);
      if (n < 8 && !conv) STAMP(8 + n);
      u32x4 A[4]; u32x3 P[2]; unsigned E, tk;
      const char* sws = p.ws[src];
      const char* sb = sws + PK_OFF + (size_t)u * PKU;
      asm volatile("s_nop 4\n\t"
                   "global_load_dword %6, %10, %13 sc0 sc1\n\t"
                   "global_load_dwordx3 %4, %9, %12 sc0 sc1\n\t"
                   "global_load_dwordx3 %5, %9, %12 offset:768 sc0 sc1\n\t"
                   "global_load_dwordx4 %0, %8, %11 sc0 sc1\n\t"
                   "global_load_dwordx4 %1, %8, %11 offset:1024 sc0 sc1\n\t"
                   "global_load_dwordx4 %2, %8, %11 offset:2048 sc0 sc1\n\t"
                   "global_load_dwordx4 %3, %8, %11 offset:3072 sc0 sc1\n\t"
                   "s_mov_b64 exec, 1\n\ts_nop 3\n\t"
                   "global_atomic_add %7, %8, %16, %14 sc0\n\t"
                   "s_waitcnt vmcnt(8)\n\t"
                   "global_store_dword %8, %17, %15 sc0 sc1\n\t"
                   "s_mov_b64 exec, -1\n\t"
                   "s_waitcnt vmcnt(1)"
                   : "=&v"(A[0]), "=&v"(A[1]), "=&v"(A[2]), "=&v"(A[3]), "=&v"(P[0]), "=&v"(P[1]), "=&v"(E), "=&v"(tk)
                   : "v"(v0), "v"(vc), "v"(vh), "s"(sb + r * 4096), "s"(sb + PK_C + r * 1536), "s"(sb + PK_E + r * PK_R),
                     "s"(tkp), "s"(fl), "v"(1u), "v"(call) : "memory");
      const size_t xo = XB_OFF + (size_t)((src * 2 + h) * NSTEP + s) * XSTG + r * 8192;
      decode(A, P, E, lds, wave * XSTG, me + xo, sws + xo, lane);
      fl = me + XF_OFF + (size_t)(((src * 2 + h) * 2 + r) * NSTEP + s) * 4;
      if (n < 8) STAMP(16 + n);
      if ((n & 7) == 7) STAMP(48 + n / 8);
      if (conv) break;
      T = umin(rfl(tk), 7u * 512u) + NWP * NPULL + CPULL * NCONV + NPUSH * 14;
    }
    if (wave < nw) { waitv<0>(); flag_st(fl, call); }
    if (conv) {
      for (;;) {
        const int t = NGEMM * (UPF + UPC) + (b - NPULL) + NCONV * (int)lds_ticket(SY_TKT);
        if (t >= 16 * NSTEP) break;
        if ((t & 255) < NCONV) STAMP(8 + (t >> 8));
        convert_w(p, me, t & 15, t >> 4, lane, call);
      }
      if (wave == 0) STAMP(2);
      return;
    }
    if (wave < NWP) { STAMP(24 + wave); lds_st(SY_TKT, 64u); }
    if (b >= 8) return;
    if (wave >= NWP) {
      while (lds_ld1(SY_TKT) < 64u) __builtin_amdgcn_s_sleep(32);
    }
    const int lg = lane < NGEMM / 4 ? lane : NGEMM / 4 - 1;
    const char* g0 = me + GR_OFF + (size_t)(((call & 1) * NR + wave) * 256) * 8 + lg * 32;
    for (;;) {
      u32x4 a0, a1; unsigned t;
      asm volatile("global_load_dwordx4 %0, %3, off sc0 sc1\n\tglobal_load_dwordx4 %1, %3, off offset:16 sc0 sc1\n\t"
                   "global_load_dword %2, %4, off sc0 sc1\n\ts_waitcnt vmcnt(0)"
                   : "=&v"(a0), "=&v"(a1), "=&v"(t) : "v"(g0), "v"(tmo) : "memory");
      const bool ok = a0[1] == call && a0[3] == call && a1[1] == call && a1[3] == call;
      if (__builtin_amdgcn_ballot_w64(ok) == ~0ull) {
        const unsigned best = wave_umax(umax(umax(a0[0], a0[2]), umax(a1[0], a1[2])));
        asm volatile("ds_max_u32 %0, %1" :: "v"(SY_GMAX), "v"(best) : "memory");
        if (lds_ticket(SY_ARR) == 7u) {
          const unsigned gm = lds_ld(SY_GMAX);
          if (lane == 0)
            __hip_atomic_store((unsigned long long*)(me + GM_OFF + b * 128), ((unsigned long long)call << 32) | gm,
                               __ATOMIC_RELAXED, __HIP_MEMORY_SCOPE_SYSTEM);
          STAMP(4);
        }
        break;
      }
      if (expired(t, call, t0, me)) break;
      __builtin_amdgcn_s_sleep(1);
    }
    return;
  }

  const int xcd = b & 7, idx = (b >> 3) - NCOMM / 8, xh = xcd & 1;
  const bool extra = idx < 24;
  const int di = extra ? idx >> 2 : 6;
  const int pn = (xcd >> 1) * 4 + (extra ? idx & 3 : 2 * xh + (idx & 1));
  const int src = (rank + 1 + di) & 7, h = extra ? xh : 0;
  const int esrc = di < 4 ? rank : (rank + 7) & 7, eh = di < 4 ? xh : extra ? 1 : 0, eq = di < 4 ? di : 2 * xh + (di & 1);
  if (wave < 4) {
    if (wave < UPC) {
      const int t = NGEMM * (UPF + wave) + b - NCOMM;
      convert_w(p, me, t & 15, t >> 4, lane, call);
      if (wave == UPC - 1) STAMP(7);
    }
    __builtin_amdgcn_s_setprio(PRIOC);
    if (extra) consumer<5>(p, me, lds, lane, wave, call, b - NCOMM, src * 512 + h * 256, esrc * 512 + eh * 256 + eq * 64, pn, t0);
    else consumer<4>(p, me, lds, lane, wave, call, b - NCOMM, src * 512 + h * 256, 0, pn, t0);
    return;
  }
  bool dead = false;
  if (wave < 6) {
    __builtin_amdgcn_s_setprio(PRIOL);
    const char* xs = me + XB_OFF + (size_t)((src * 2 + h) * NSTEP) * XSTG;
    const char* ls = me + XB_OFF + (size_t)((esrc * 2 + eh) * NSTEP) * XSTG + eq * 4096;
    const char* wsr = me + WB_OFF + (size_t)(pn * NSTEP) * WSTG;
    const unsigned voff = lane * 16, pad = lane < 4 ? SY_DONE + 4 * lane : SY_READY;
    unsigned slot = (wave - 4) * STG;
    const int fw = SY_FULL + 4 * (wave - 4);
    const char* tok = me + IO_OFF + (size_t)(b * 2 + (wave - 4)) * 256;
    st4((char*)tok + lane * 4, 2 * lane + (wave - 4) + 1);
    waitv<0>();
    for (;;) {
      u32x4 d;
      asm volatile("ds_read_b128 %0, %1\n\ts_waitcnt lgkmcnt(0)" : "=&v"(d) : "v"(SY_PACK) : "memory");
      if (rfl(d[0] & d[1] & d[2] & d[3])) break;
      __builtin_amdgcn_s_sleep(2);
    }
    for (int s = wave - 4; s < NSTEP; s += 2) {
      unsigned rdy;
      for (;;) {
        unsigned r, dm, d1, d2, d3;
        asm volatile("s_mov_b64 exec, 31\n\tds_read_b32 %0, %6\n\ts_mov_b64 exec, -1\n\ts_waitcnt lgkmcnt(0)\n\t"
                     "v_readlane_b32 %1, %0, 4\n\tv_readlane_b32 %2, %0, 0\n\tv_readlane_b32 %3, %0, 1\n\t"
                     "v_readlane_b32 %4, %0, 2\n\tv_readlane_b32 %5, %0, 3\n\ts_nop 0\n\t"
                     "s_min_u32 %2, %2, %3\n\ts_min_u32 %4, %4, %5\n\ts_min_u32 %2, %2, %4"
                     : "=&v"(r), "=s"(rdy), "=s"(dm), "=s"(d1), "=s"(d2), "=s"(d3) : "v"(pad) : "memory", "scc");
        if (rdy >= ABORT || (rdy > (unsigned)s && dm + NS > (unsigned)s)) break;
        __builtin_amdgcn_s_sleep(SLEEPL);
      }
      if (rdy >= ABORT) { dead = true; break; }
      const int sa = s;
      const char* xa = xs + (size_t)sa * XSTG;
#define PACE_
      glds4(voff, wsr + (size_t)sa * WSTG, slot + WOF); PACE_ glds4(voff, wsr + (size_t)sa * WSTG + 4096, slot + WOF + 4096); PACE_
      glds4(voff, xa, slot); PACE_ glds4(voff, xa + 4096, slot + 5120); PACE_ glds4(voff, xa + 8192, slot + 10240); PACE_ glds4(voff, xa + 12288, slot + 15360); PACE_
      if (extra) glds4s(voff, ls + (size_t)sa * XSTG, slot + 4096);
      asm volatile("s_mov_b32 m0, %2\n\ts_mov_b64 exec, 1\n\ts_nop 1\n\tglobal_load_lds_dword %0, %1 sc0 sc1\n\ts_mov_b64 exec, -1"
                   :: "v"(voff), "s"(tok + (s >> 1) * 4), "s"(fw) : "memory");
      slot += 2 * STG; if (slot >= NS * STG) slot -= NS * STG;
    }
    waitv<0>();
    if (dead) lds_st(fw, ABORT);
    __builtin_amdgcn_s_setprio(0);
  } else if (wave == 6) {
    __builtin_amdgcn_s_setprio(PRIOP);
    const char* pa = lane < 32 ? me + XF_OFF + (size_t)((src * 2 + h) * 2 * NSTEP) * 4 + lane * 16
                               : me + WF_OFF + (size_t)(pn * NSTEP) * 4 + (lane & 31) * 16;
    const char* pb = me + XF_OFF + (size_t)((lane < 32 ? (esrc * 2 + eh) * 2 + (eq >> 1) : (src * 2 + h) * 2 + 1) * NSTEP) * 4 + (lane & 31) * 16;
    unsigned ready = 0;
    while (!dead && ready < (unsigned)NSTEP) {
      if (ready > lds_ld1(SY_DONE) + NS + 2) { __builtin_amdgcn_s_sleep(SLEEPP); continue; }
      u32x4 f, g; unsigned t;
      asm volatile("global_load_dwordx4 %0, %3, off sc0 sc1\n\tglobal_load_dwordx4 %1, %4, off sc0 sc1\n\t"
                   "global_load_dword %2, %5, off sc0 sc1\n\ts_waitcnt vmcnt(0)"
                   : "=&v"(f), "=&v"(g), "=&v"(t) : "v"(pa), "v"(pb), "v"(tmo) : "memory");
#pragma unroll
      for (int i = 0; i < 4; ++i) f[i] = umin(f[i], g[i]);
      unsigned c = f[0] < call ? 0u : f[1] < call ? 1u : f[2] < call ? 2u : f[3] < call ? 3u : 4u;
      unsigned long long part = __builtin_amdgcn_ballot_w64(c < 4u);
      unsigned px = (unsigned)part, pw = (unsigned)(part >> 32);
      unsigned rx = NSTEP, rw = NSTEP;
      if (px) { unsigned l = __builtin_ctz(px); rx = 4 * l + __builtin_amdgcn_readlane(c, l); }
      if (pw) { unsigned l = __builtin_ctz(pw); rw = 4 * l + __builtin_amdgcn_readlane(c, 32 + l); }
      unsigned r = umin(rx, rw);
      if (r > ready) {
        ready = r; lds_st(SY_READY, ready); continue; }
      if (expired(t, call, t0, me)) { dead = true; break; }
      __builtin_amdgcn_s_sleep(1);
    }
    if (dead) lds_st(SY_READY, ABORT);
    STAMP(3);
  } else {
    for (unsigned T = b - NCOMM; T < NPUSH * 14u; T += NGEMM) {
      const unsigned hu = T / 7u, sr = (rank + 1 + (T - hu * 7u)) & 7u, u = hu >> 1, r = hu & 1;
      auto wait_flag = [&](const char* fp) {
        for (;;) {
          unsigned f, t;
          asm volatile("global_load_dword %0, %2, %3 sc0 sc1\n\tglobal_load_dword %1, %2, %4 sc0 sc1\n\ts_waitcnt vmcnt(0)"
                       : "=&v"(f), "=&v"(t) : "v"(0), "s"(fp), "s"(tmo) : "memory");
          if (rfl(f) >= call) break;
          if (expired(t, call, t0, me)) { dead = true; break; }
          __builtin_amdgcn_s_sleep(1);
        }
      };
      wait_flag(me + PF_OFF + (size_t)(sr * 64 + u) * 128);
      if (dead) break;
      STAMP(4);
      u32x4 A[4]; u32x3 P[2]; unsigned E;
      const char* sb = me + LD_OFF + (size_t)(sr * NPUSH + u) * PKU;
      asm volatile("s_nop 4\n\t"
                   "global_load_dword %6, %9, %12 sc0 sc1\n\t"
                   "global_load_dwordx3 %4, %8, %11 sc0 sc1\n\t"
                   "global_load_dwordx3 %5, %8, %11 offset:768 sc0 sc1\n\t"
                   "global_load_dwordx4 %0, %7, %10 sc0 sc1\n\t"
                   "global_load_dwordx4 %1, %7, %10 offset:1024 sc0 sc1\n\t"
                   "global_load_dwordx4 %2, %7, %10 offset:2048 sc0 sc1\n\t"
                   "global_load_dwordx4 %3, %7, %10 offset:3072 sc0 sc1\n\t"
                   "s_waitcnt vmcnt(0)"
                   : "=&v"(A[0]), "=&v"(A[1]), "=&v"(A[2]), "=&v"(A[3]), "=&v"(P[0]), "=&v"(P[1]), "=&v"(E)
                   : "v"(lane * 16), "v"(lane * 12), "v"((lane < 48 ? lane : 47) * 4), "s"(sb + r * 4096), "s"(sb + PK_C + r * 1536),
                     "s"(sb + PK_E + r * PK_R) : "memory");
      const size_t xo = XB_OFF + (size_t)((sr * 2 + (u & 1)) * NSTEP + (u >> 1)) * XSTG + r * 8192;
      const unsigned c0 = (rfl(E) >> 8) & 0xffu, c1 = ((unsigned)__builtin_amdgcn_readlane(E, 47) >> 8) & 0xffu;
      if (c0 + c1 + 2u > EXC) wait_flag(me + SF_OFF + (size_t)(sr * 256 + u) * 4);
      if (dead) break;
      decode(A, P, E, lds, SCR, me + xo, p.ws[sr] + xo, lane);
      waitv<0>();
      flag_st(me + XF_OFF + (size_t)(((sr * 2 + (u & 1)) * 2 + r) * NSTEP + (u >> 1)) * 4, call);
      STAMP(5);
    }
  }
}

}

extern "C" void kernel_launch(int rank, int nranks, ncclComm_t comm, hipStream_t stream,
                              void* const* d_in, const int* in_sizes, int n_in,
                              void* d_out, int out_size, const int* in_dtypes, int out_dtype,
                              void* const* d_out_all, void* const* d_ws_all, size_t ws_bytes) {
  if (nranks != NR || (unsigned)rank >= (unsigned)NR || n_in < 2 || ws_bytes < WS_NEED ||
      in_sizes[0] != 512 * KD || out_size != 4096 * NLOC) {
    fprintf(stderr, "kernel_launch: unexpected problem shape\n");
    return;
  }
  HIP_CHECK(hipFuncSetAttribute((const void*)fused, hipFuncAttributeMaxDynamicSharedMemorySize, LDS_BYTES));
  Params p;
  p.x = (const float*)d_in[0]; p.w = (const float*)d_in[1]; p.out = (char*)d_out; p.rank = rank;
  for (int i = 0; i < NR; ++i) p.ws[i] = (char*)d_ws_all[i];
  fused<<<NWG, 512, LDS_BYTES, stream>>>(p);
  HIP_CHECK(hipGetLastError());
}
```

```cpp
#include <hip/hip_runtime.h>
#include <rccl/rccl.h>
#include <cstdio>
#include <cstdint>

#define HIP_CHECK(expr) do { hipError_t _e = (expr); if (_e != hipSuccess) { \
    fprintf(stderr, "HIP error %s at %s:%d\n", hipGetErrorString(_e), __FILE__, __LINE__); \
    return; } } while (0)

namespace {

typedef unsigned u32x4 __attribute__((ext_vector_type(4)));
typedef unsigned u32x3 __attribute__((ext_vector_type(3)));
typedef unsigned u32x2 __attribute__((ext_vector_type(2)));
typedef unsigned short u16x2 __attribute__((ext_vector_type(2)));
typedef float f32x4 __attribute__((ext_vector_type(4)));
typedef float f32x2 __attribute__((ext_vector_type(2)));
typedef __bf16 bf16x8 __attribute__((ext_vector_type(8)));
typedef __bf16 bf16x2 __attribute__((ext_vector_type(2)));
#define LAS __attribute__((address_space(3)))

#define FORCE_RAW 0
#define UPF 1
#define PRIOC 3
#define PRIOL 0
#define PRIOP 0
#define SLEEPL 1
#define SLEEPP 4
#define STAG 0
#define LSC " sc0 sc1"
#define GNOP 2
#define GREP 1
#define STR2(x) #x
#define STR(x) STR2(x)
#define GAP ".rept " STR(GREP) "\n\ts_nop " STR(GNOP) "\n\t.endr\n\t"
#define NPUSH 12
#define PSTAG 0
#define SPEC 800
#define GSCN 0
#define GSC ""
#define GLC " sc1"
#define ASLP 0
#define EPOLL 8
#define FULLN 1
#define OSCN 1
#define OSC " sc0 sc1"
#define POLL1 1
#define XDLY 40
#define CPULL 0
#define TST0 450
#define TSTD 10
#define UPC 0
#define NSPL 3
#define SPREAD 0
#define NWP 6

constexpr int NR = 8;
constexpr int KD = 4096, ND = 16384, NLOC = 2048;
constexpr int NSTEP = 128;
constexpr int NPULL = 28, NCOMM = 48, NCONV = NCOMM - NPULL, NGEMM = 208, NWG = 256;
constexpr int XSTG = 16384, WSTG = 8192;
constexpr int PK_C = 8192, PK_E = 11264, PK_R = 192, PKU = PK_E + 2 * PK_R;
constexpr unsigned EXC = (PK_R - 8) / 2;
constexpr size_t XB_OFF = 0;
constexpr size_t WB_OFF = 32u << 20;
constexpr size_t PK_OFF = 48u << 20;
constexpr size_t LD_OFF = 52u << 20;
constexpr size_t FL_OFF = 64u << 20;
constexpr size_t CNT_OFF = FL_OFF;
constexpr size_t TMO_OFF = FL_OFF + 0x4000;
constexpr size_t SF_OFF = FL_OFF + 0x5000;
constexpr size_t XF_OFF = FL_OFF + 0x60000;
constexpr size_t WF_OFF = FL_OFF + 0xA000;
constexpr size_t GM_OFF = FL_OFF + 0xC000;
constexpr size_t TK_OFF = FL_OFF + 0xD000;
constexpr size_t GR_OFF = FL_OFF + 0x10000;
constexpr size_t ST_OFF = FL_OFF + 0x20000;
constexpr size_t IO_OFF = FL_OFF + 0x40000;
constexpr size_t PF_OFF = FL_OFF + 0x64000;
constexpr size_t WS_NEED = FL_OFF + 0x74000;
constexpr int NS = 5;
constexpr int STG = 28672;
constexpr int SY = NS * STG;
constexpr int SY_FULL = SY, SY_READY = SY + 8, SY_DONE = SY + 16, SY_ARR = SY + 32, SY_WMAX = SY + 48,
              SY_GMAX = SY + 64, SY_TKT = SY + 84, SY_PACK = SY + 96, SY_SRC = SY + 112, SY_PTR = SY + 144;
constexpr int SCR = SY + 512;
constexpr int LDS_BYTES = SCR + 8192;
constexpr unsigned long long DEADLINE = 50000000ull;
constexpr unsigned ABORT = 0x40000000u;
static_assert(NPUSH % 4 == 0 && NPUSH * 14 + NWP * NPULL + CPULL * NCONV <= 64 * 14, "first round: within the PF flags");

struct Params { const float* x; const float* w; char* out; char* ws[NR]; int rank; };

__device__ __forceinline__ unsigned long long rt() { return __builtin_amdgcn_s_memrealtime(); }
template <int N_> __device__ __forceinline__ void waitv() { asm volatile("s_waitcnt vmcnt(%0)" :: "i"(N_) : "memory"); }
__device__ __forceinline__ void cfence() { asm volatile("" ::: "memory"); }
__device__ __forceinline__ void st16(char* p, u32x4 v) {
  asm volatile("global_store_dwordx4 %0, %1, off sc0 sc1\n\ts_nop 1" :: "v"(p), "v"(v) : "memory");
}
__device__ __forceinline__ void st4(char* p, unsigned v) {
  asm volatile("global_store_dword %0, %1, off sc0 sc1" :: "v"(p), "v"(v) : "memory");
}
__device__ __forceinline__ void flag_st(const char* p, unsigned v) {
  asm volatile("s_mov_b64 exec, 1\n\ts_nop 3\n\tglobal_store_dword %0, %1, %2 sc0 sc1\n\ts_mov_b64 exec, -1" :: "v"(0), "v"(v), "s"(p) : "memory");
}
__device__ __forceinline__ unsigned lds_ticket(int addr) {
  unsigned v, r;
  asm volatile("s_mov_b64 exec, 1\n\tds_add_rtn_u32 %0, %2, %3\n\ts_mov_b64 exec, -1\n\ts_waitcnt lgkmcnt(0)\n\tv_readfirstlane_b32 %1, %0"
               : "=&v"(v), "=s"(r) : "v"(addr), "v"(1u) : "memory");
  return r;
}
__device__ __forceinline__ void poll16(u32x4& a, unsigned& t, const char* p, const char* tm) {
  asm volatile("global_load_dwordx4 %0, %2, off sc0 sc1\n\tglobal_load_dword %1, %3, off sc0 sc1\n\ts_waitcnt vmcnt(0)"
               : "=&v"(a), "=&v"(t) : "v"(p), "v"(tm) : "memory");
}
__device__ __forceinline__ void lds_st(int addr, unsigned v) {
  asm volatile("ds_write_b32 %0, %1" :: "v"(addr), "v"(v) : "memory");
}
__device__ __forceinline__ unsigned lds_ld(int addr) {
  unsigned v; asm volatile("ds_read_b32 %0, %1\n\ts_waitcnt lgkmcnt(0)" : "=&v"(v) : "v"(addr) : "memory"); return v;
}
__device__ __forceinline__ unsigned lds_ld1(int addr) {
  unsigned v, r;
  asm volatile("s_mov_b64 exec, 1\n\tds_read_b32 %0, %2\n\ts_waitcnt lgkmcnt(0)\n\tv_readfirstlane_b32 %1, %0\n\ts_mov_b64 exec, -1"
               : "=&v"(v), "=s"(r) : "v"(addr) : "memory");
  return r;
}
__device__ __forceinline__ unsigned rfl(unsigned v) { return __builtin_amdgcn_readfirstlane(v); }
__device__ __forceinline__ unsigned umax(unsigned a, unsigned b) { return a > b ? a : b; }
__device__ __forceinline__ unsigned umin(unsigned a, unsigned b) { return a < b ? a : b; }
__device__ __forceinline__ unsigned wave_umax(unsigned v) {
  v = umax(v, (unsigned)__builtin_amdgcn_update_dpp(0, (int)v, 0x111, 0xf, 0xf, false));
  v = umax(v, (unsigned)__builtin_amdgcn_update_dpp(0, (int)v, 0x112, 0xf, 0xf, false));
  v = umax(v, (unsigned)__builtin_amdgcn_update_dpp(0, (int)v, 0x114, 0xf, 0xf, false));
  v = umax(v, (unsigned)__builtin_amdgcn_update_dpp(0, (int)v, 0x118, 0xf, 0xf, false));
  unsigned a = __builtin_amdgcn_readlane(v, 15), b = __builtin_amdgcn_readlane(v, 31);
  unsigned c = __builtin_amdgcn_readlane(v, 47), d = __builtin_amdgcn_readlane(v, 63);
  return umax(umax(a, b), umax(c, d));
}
__device__ __forceinline__ unsigned wave_fsum(float v) {
#define DPPADD(ctrl, rm) v += __builtin_bit_cast(float, __builtin_amdgcn_update_dpp(0, __builtin_bit_cast(int, v), ctrl, rm, 0xf, false))
  DPPADD(0x111, 0xf); DPPADD(0x112, 0xf); DPPADD(0x114, 0xf); DPPADD(0x118, 0xf); DPPADD(0x142, 0xa); DPPADD(0x143, 0xc);
  return __builtin_amdgcn_readlane(__builtin_bit_cast(unsigned, v), 63);
}
__device__ __forceinline__ unsigned bfi(unsigned m, unsigned a, unsigned b) { return (a & m) | (b & ~m); }
__device__ __forceinline__ unsigned pk_sub16(unsigned a, unsigned b) {
  return __builtin_bit_cast(unsigned, (u16x2)(__builtin_bit_cast(u16x2, a) - __builtin_bit_cast(u16x2, b)));
}
__device__ __forceinline__ unsigned pk_min16(unsigned a, unsigned b) {
  return __builtin_bit_cast(unsigned, __builtin_elementwise_min(__builtin_bit_cast(u16x2, a), __builtin_bit_cast(u16x2, b)));
}
__device__ __forceinline__ unsigned pk_bf16(float lo, float hi) {
  f32x2 f = {lo, hi};
  return __builtin_bit_cast(unsigned, __builtin_convertvector(f, bf16x2));
}
__device__ __forceinline__ unsigned perm(unsigned hi, unsigned lo, unsigned sel) { return __builtin_amdgcn_perm(hi, lo, sel); }
__device__ __forceinline__ unsigned rotr(unsigned v, unsigned n) { return __builtin_amdgcn_alignbit(v, v, n); }
__device__ __forceinline__ void glds4(unsigned voff, const char* sbase, unsigned dst) {
  asm volatile("s_mov_b32 m0, %2\n\ts_nop 0\n\t"
               "global_load_lds_dwordx4 %0, %1" LSC "\n\t" GAP
               "global_load_lds_dwordx4 %0, %1 offset:1024" LSC "\n\t" GAP
               "global_load_lds_dwordx4 %0, %1 offset:2048" LSC "\n\t" GAP
               "global_load_lds_dwordx4 %0, %1 offset:3072" LSC "\n\t" GAP
               :: "v"(voff), "s"(sbase), "s"(dst) : "memory");
}
__device__ __forceinline__ void glds4s(unsigned voff, const char* sbase, unsigned dst) {
  asm volatile("s_mov_b32 m0, %2\n\ts_nop 0\n\t"
               "global_load_lds_dwordx4 %0, %1" LSC "\n\t" GAP
               "s_add_u32 m0, m0, 4096\n\ts_nop 0\n\t"
               "global_load_lds_dwordx4 %0, %1 offset:1024" LSC "\n\t" GAP
               "s_add_u32 m0, m0, 4096\n\ts_nop 0\n\t"
               "global_load_lds_dwordx4 %0, %1 offset:2048" LSC "\n\t" GAP
               "s_add_u32 m0, m0, 4096\n\ts_nop 0\n\t"
               "global_load_lds_dwordx4 %0, %1 offset:3072" LSC "\n\t" GAP
               :: "v"(voff), "s"(sbase), "s"(dst) : "memory", "scc");
}
#define STAMP(i) do {} while (0)

__device__ __forceinline__ bool expired(unsigned tmo_seen, unsigned call, unsigned long long t0, char* me) {
  if (rfl(tmo_seen) == call) return true;
  if (rt() - t0 > DEADLINE) {
    __hip_atomic_store((unsigned*)(me + TMO_OFF), call, __ATOMIC_RELAXED, __HIP_MEMORY_SCOPE_AGENT);
    return true;
  }
  return false;
}

struct WRegs { f32x4 v[2][8]; };
__device__ __forceinline__ void w_load(WRegs& r, const Params& p, int panel, int s, int lane) {
  const float* wp = p.w + (size_t)(32 * s + 8 * (lane >> 4)) * ND + p.rank * NLOC + panel * 128 + 4 * (lane & 15);
#pragma unroll
  for (int hf = 0; hf < 2; ++hf)
#pragma unroll
    for (int j = 0; j < 8; ++j) r.v[hf][j] = __builtin_nontemporal_load((const f32x4*)(wp + (size_t)j * ND + hf * 64));
}
__device__ __forceinline__ void w_store(const WRegs& r, char* me, int panel, int s, int lane) {
  char* dst = me + WB_OFF + (size_t)(panel * NSTEP + s) * WSTG + lane * 16;
#pragma unroll
  for (int hf = 0; hf < 2; ++hf)
#pragma unroll
    for (int c = 0; c < 4; ++c) {
      u32x4 o = {pk_bf16(r.v[hf][0][c], r.v[hf][1][c]), pk_bf16(r.v[hf][2][c], r.v[hf][3][c]),
                 pk_bf16(r.v[hf][4][c], r.v[hf][5][c]), pk_bf16(r.v[hf][6][c], r.v[hf][7][c])};
      st16(dst + (4 * hf + c) * 1024, o);
    }
}
__device__ __forceinline__ void w_flag(char* me, int panel, int s, int lane, unsigned call) {
  waitv<0>();
  flag_st(me + WF_OFF + (size_t)(panel * NSTEP + s) * 4, call);
}
__device__ __forceinline__ void convert_w(const Params& p, char* me, int panel, int s, int lane, unsigned call) {
  WRegs r;
  w_load(r, p, panel, s, lane);
  w_store(r, me, panel, s, lane);
  w_flag(me, panel, s, lane, call);
}

template <bool PUSH>
__device__ __forceinline__ void pack(const Params& p, const f32x4 (&xv)[8], char* me, LAS char* lds, int lane, int wave,
                                     unsigned call, int ub, int part) {
  const int rank = p.rank, us = ub >> 1, uh = ub & 1;
  float mx = 0.f, sm = 0.f;
  u32x4 D[4];
  {
    LAS char* tw = lds + wave * XSTG + (lane >> 3) * 16 + ((lane & 7) >> 1) * 256 + (lane & 1) * 8;
#pragma unroll
    for (int i = 0; i < 8; ++i) {
      mx = fmaxf(fmaxf(mx, fmaxf(fabsf(xv[i][0]), fabsf(xv[i][1]))), fmaxf(fabsf(xv[i][2]), fabsf(xv[i][3])));
      sm += fabsf(xv[i][i & 3]);
      *(LAS u32x2*)(tw + (i >> 1) * 1024 + (i & 1) * 128) = u32x2{pk_bf16(xv[i][0], xv[i][1]), pk_bf16(xv[i][2], xv[i][3])};
    }
    if (wave == 0 && !PUSH) STAMP(56);
    cfence();
#pragma unroll
    for (int f = 0; f < 4; ++f) D[f] = *(LAS u32x4*)(lds + wave * XSTG + f * 1024 + lane * 16);
  }
  char* xl = me + XB_OFF + (size_t)((rank * 2 + uh) * NSTEP + us) * XSTG + wave * 4096 + lane * 16;
#pragma unroll
  for (int f = 0; f < 4; ++f) if (!PUSH) st16(xl + f * 1024, D[f]);
  const unsigned emax = wave_umax((pk_bf16(mx, 0.f) >> 7) & 0xffu);
  const unsigned emean = (wave_fsum(sm * (3.32f / 512.f)) >> 23) & 0xffu;
  const unsigned etop = umin(emax, emean);
  const unsigned base = etop > 7u ? etop - 7u : 0u;
  const unsigned B2 = base * 0x00800080u;
  const int j0 = ub < NPUSH ? 1 + part * 7 / NSPL : 0, j1 = ub < NPUSH ? 1 + (part + 1) * 7 / NSPL : 1;
  auto pkd = [&](int j) -> char* {
    return j ? p.ws[(rank + 1 + (unsigned)(j + ub) % 7u) & 7u] + LD_OFF + (size_t)(rank * NPUSH + ub) * PKU : me + PK_OFF + (size_t)ub * PKU;
  };
  {
    unsigned S[8];
#pragma unroll
    for (int i = 0; i < 8; ++i) {
      const unsigned d0 = D[i >> 1][2 * (i & 1)], d1 = D[i >> 1][2 * (i & 1) + 1];
      S[i] = bfi(0x80808080u, perm(d1, d0, 0x07050301u), perm(d1, d0, 0x06040200u));
    }
    if (PSTAG && ub < NPUSH) for (int i = 0; i < ub; ++i) __builtin_amdgcn_s_sleep(PSTAG);
    for (int j = j0; j < j1; ++j) {
      char* pk = pkd(j);
      st16(pk + wave * 2048 + lane * 16, u32x4{S[0], S[1], S[2], S[3]});
      st16(pk + wave * 2048 + 1024 + lane * 16, u32x4{S[4], S[5], S[6], S[7]});
    }
  }
  unsigned C[16], Ma = 0, Mb = 0;
#pragma unroll
  for (int i = 0; i < 16; ++i) {
    const unsigned t = pk_sub16(D[i >> 2][i & 3] & 0x7f807f80u, B2);
    C[i] = t & 0x03800380u;
    const unsigned y = pk_min16(t, 0x04000400u);
    if (i < 8) Ma = bfi(0x04000400u, y, Ma >> 1); else Mb = bfi(0x04000400u, y, Mb >> 1);
  }
  {
    u32x3 P;
#pragma unroll
    for (int g = 0; g < 3; ++g)
      P[g] = C[5 * g] | (C[5 * g + 1] << 3) | (C[5 * g + 2] << 6) | (((C[5 * g + 3] << 3) | C[5 * g + 4]) >> 6) | ((C[15] >> (7 + g)) & 0x00010001u);
    for (int j = j0; j < j1; ++j)
      asm volatile("global_store_dwordx3 %0, %1, off sc0 sc1\n\ts_nop 1" :: "v"(pkd(j) + PK_C + wave * 768 + lane * 12), "v"(P) : "memory");
  }
  unsigned M = (Ma >> 3) | (Mb << 5), cnt = 0;
  bool raw = FORCE_RAW;
  const int eb = wave * XSTG + 4096;
  for (;;) {
    const unsigned long long act = __builtin_amdgcn_ballot_w64(M != 0);
    if (!act) break;
    const unsigned k = M ? __builtin_ctz(M) : 0u;
    const unsigned pos = (k & 12u) * 128u + lane * 8u + (k & 3u) * 2u + (k >> 4);
    const unsigned v = *(LAS unsigned short*)(lds + wave * XSTG + pos * 2);
    const unsigned hi = (unsigned)(((int)((v >> 7) & 0xffu) - (int)base) >> 3) + 28u;
    const unsigned sl = cnt + __builtin_amdgcn_mbcnt_hi((unsigned)(act >> 32), __builtin_amdgcn_mbcnt_lo((unsigned)act, 0u));
    const unsigned slc = M != 0 && sl < EXC ? sl : 127u - 2u;
    *(LAS unsigned short*)(lds + eb + ((wave & 1) && slc < EXC ? PK_R - 6 - 2 * slc : 4 + 2 * slc)) = (unsigned short)(pos | (hi << 11));
    if (__builtin_amdgcn_ballot_w64(M != 0 && hi > 31u)) raw = true;
    cnt += (unsigned)__builtin_popcountll(act);
    M &= M - 1;
    if (cnt > EXC) { raw = true; break; }
  }
  {
    const unsigned nd = ((raw ? 0u : cnt) + 1u) / 2u + 1u;
    const int o = (wave & 1) ? PK_R - 4 - 4 * lane : 4 * lane;
    cfence();
    unsigned v = *(LAS unsigned*)(lds + eb + ((unsigned)lane < nd ? o : 0));
    cfence();
    if (!PUSH) lds_st(SY_PACK + 4 * wave, 1u);
    if (lane == 0) v = base | ((raw ? 255u : cnt) << 8);
    if ((unsigned)lane < nd)
      for (int j = j0; j < j1; ++j) st4(pkd(j) + PK_E + (wave >> 1) * PK_R + o, v);
  }
  if (wave == 0 && !PUSH) STAMP(57);
  waitv<0>();
  if (lds_ticket(SY_ARR + (PUSH ? 12 : 4)) == 3) {
    const int j = lane - 16, pr = lane < NR ? lane : (rank + 1 + (unsigned)(j + ub) % 7u) & 7u;
    if ((!PUSH && lane <= NR) || (ub < NPUSH ? j >= j0 && j < j1 : ub < 64 && j >= 1 && j < NR)) {
      char* f = lane == rank || lane == NR ? me + XF_OFF + (size_t)(((rank * 2 + uh) * 2 + (lane >> 3)) * NSTEP + us) * 4
                             : (char*)*(LAS unsigned long long*)(lds + SY_PTR + pr * 8)
                               + (lane < NR ? SF_OFF + (size_t)(rank * 256 + ub) * 4 : PF_OFF + (size_t)(rank * 64 + ub) * 128);
      st4(f, call);
    }
    STAMP(PUSH ? 6 : 1);
  }
}

__device__ __forceinline__ void decode(const u32x4 (&A)[4], const u32x3 (&P)[2], unsigned E, LAS char* lds, int io,
                                       char* dst, const char* rawp, int lane) {
  char* xd = dst + lane * 16;
  LAS char* img = lds + io;
  const unsigned v0 = lane * 16;
  const unsigned hd[2] = {(unsigned)__builtin_amdgcn_readlane(E, 0), (unsigned)__builtin_amdgcn_readlane(E, 47)};
  const unsigned c0 = (hd[0] >> 8) & 0xffu, c1 = (hd[1] >> 8) & 0xffu;
  const unsigned n0 = c0 > EXC ? 0u : c0, n1 = c1 > EXC ? 0u : c1;
  const bool clash = ((n0 + 1u) & ~1u) + ((n1 + 1u) & ~1u) > EXC;
  const bool raw[2] = {clash || c0 > EXC, clash || c1 > EXC};
  const unsigned ce = clash ? 0u : n0, co = clash ? 0u : n1;
#pragma unroll
  for (int q = 0; q < 2; ++q) {
    const unsigned B2 = (hd[q] & 0xffu) * 0x00800080u;
#pragma unroll
    for (int f = 0; f < 4; ++f) {
      u32x4 d;
#pragma unroll
      for (int j = 0; j < 4; ++j) {
        const int i = 4 * f + j, g = i / 5, rr = i % 5;
        const unsigned c = i == 15 ? ((P[q][0] & 0x00010001u) << 7) | ((P[q][1] & 0x00010001u) << 8) | ((P[q][2] & 0x00010001u) << 9)
                                   : (rr < 3 ? P[q][g] >> (3 * rr) : P[q][g] << (3 * rr - 6)) & 0x03800380u;
        const unsigned sm = A[2 * q + (f >> 1)][2 * (f & 1) + (j >> 1)];
        d[j] = (perm(sm, sm, (j & 1) ? 0x03030202u : 0x01010000u) & 0x807f807fu) | (c + B2);
      }
      *(LAS u32x4*)(img + q * 4096 + f * 1024 + lane * 16) = d;
    }
  }
  cfence();
#pragma unroll
  for (int c = 0; c < 2; ++c) {
    const unsigned ent = (E >> (16 * c)) & 0xffffu, hw = 2u * lane + c;
    const bool ev = hw - 2u < ce, od = (PK_R / 2 - 3) - hw < co;
    const unsigned dl = (ev || od) && lane < 48 ? ((ent >> 11) - 28u) << (10u + (ent & 1u) * 16u) : 0u;
    const unsigned ad = io + (od ? 4096u : 0u) + (ent & 0x7feu) * 2u;
    asm volatile("ds_add_u32 %0, %1" :: "v"(ad), "v"(dl) : "memory");
  }
  cfence();
#pragma unroll
  for (int q = 0; q < 2; ++q) {
    u32x4 R0, R1, R2, R3;
    if (raw[q]) {
      const char* rb = rawp + q * 4096;
      asm volatile("s_nop 4\n\t"
                   "global_load_dwordx4 %0, %4, %5 sc0 sc1\n\t"
                   "global_load_dwordx4 %1, %4, %5 offset:1024 sc0 sc1\n\t"
                   "global_load_dwordx4 %2, %4, %5 offset:2048 sc0 sc1\n\t"
                   "global_load_dwordx4 %3, %4, %5 offset:3072 sc0 sc1\n\t"
                   "s_waitcnt vmcnt(0)"
                   : "=&v"(R0), "=&v"(R1), "=&v"(R2), "=&v"(R3) : "v"(v0), "s"(rb) : "memory");
    } else {
      R0 = *(LAS u32x4*)(img + q * 4096 + lane * 16); R1 = *(LAS u32x4*)(img + q * 4096 + 1024 + lane * 16);
      R2 = *(LAS u32x4*)(img + q * 4096 + 2048 + lane * 16); R3 = *(LAS u32x4*)(img + q * 4096 + 3072 + lane * 16);
    }
    st16(xd + q * 4096, R0); st16(xd + q * 4096 + 1024, R1); st16(xd + q * 4096 + 2048, R2); st16(xd + q * 4096 + 3072, R3);
  }
}

#define MFMA(m, t) asm volatile("v_mfma_f32_16x16x32_bf16 %0, %1, %2, %0" : "+v"(acc[m][t]) : "v"(Wf[t]), "v"(Xc[m]))
#define LDR(dst, addr, off) asm volatile("ds_read_b128 %0, %1 offset:%2" : "=v"(dst) : "v"(addr), "i"(off) : "memory")
template <int N_> __device__ __forceinline__ void waitl() { asm volatile("s_waitcnt lgkmcnt(%0)" :: "i"(N_) : "memory"); }
constexpr int WOF = 20480;
#define MAX2(m, t) asm volatile("v_max3_f32 %0, %2, %3, %0\n\tv_max3_f32 %1, %4, %5, %1" : "+v"(mx0), "+v"(mx1) \
                                : "v"(acc[m][t][0]), "v"(acc[m][t][1]), "v"(acc[m][t][2]), "v"(acc[m][t][3]))
template <int NM, int MODE, int PAR>
__device__ __forceinline__ void mstep(f32x4 (&acc)[NM][8], bf16x8 (&Wf)[8], bf16x8 (&Xc)[NM], bf16x8 (&Xn)[NM],
                                      unsigned& va, unsigned& vx, unsigned va0, unsigned vx0, unsigned vsy, unsigned& vdone,
                                      unsigned& slot, unsigned& fpre, int s, float& mx0, float& mx1, unsigned& nslow) {
  constexpr bool PF = MODE < 2;
  unsigned sf = 0, nslot = 0;
  constexpr int NX2 = 2 + (NM == 5) + (MODE == 0);
  waitl<5>();
  MFMA(0, 0);
  LDR(Wf[7], va, WOF + 7168);
  MFMA(1, 0);
  if (PF) asm volatile("ds_write_b32 %0, %1 offset:16" :: "v"(vsy), "v"(vdone) : "memory");
  MFMA(2, 0);
  if (PF) asm volatile("v_readfirstlane_b32 %0, %1" : "=s"(sf) : "v"(fpre) : "memory");
  MFMA(3, 0);
  if (PF) asm volatile("v_add_u32 %0, %1, %2" : "=v"(va) : "s"(slot), "v"(va0));
  if (NM == 5) MFMA(NM - 1, 0);
  if (PF) asm volatile("v_add_u32 %0, %1, %2" : "=v"(vx) : "s"(slot), "v"(vx0));
  if (PF && __builtin_expect(sf <= (unsigned)(s + 1), 0)) {
    do __builtin_amdgcn_s_sleep(1); while (lds_ld1(SY_FULL + 4 * (1 - PAR)) <= (unsigned)(s + 1));
  }
#pragma unroll
  for (int t = 1; t < 8; ++t) {
    if (t == 2) waitl<PF ? 10 : 5>();
    if (t == 3) waitl<PF ? 9 + NX2 : 4>();
    if (t == 4) waitl<PF ? 9 + NX2 : 3>();
    if (t == 5) waitl<PF ? 9 + NX2 : 2>();
    if (t == 6) waitl<PF ? 9 + NX2 : 1>();
    if (t == 7) waitl<PF ? 9 + NX2 : 0>();
    MFMA(0, t);
    if (PF) LDR(Wf[t - 1], va, WOF + (t - 1) * 1024); else MAX2(0, t - 1);
    MFMA(1, t);
    if (PF && t == 1) LDR(Xn[0], vx, 0);
    if (PF && t == 2) LDR(Xn[3], vx, 3 * 1024);
    if (MODE == 0 && t == 5) asm volatile("s_add_u32 %0, %1, %2" : "=s"(nslot) : "s"(slot), "i"(STG) : "scc");
    if (MODE == 0 && t == 6) asm volatile("s_cmp_lg_u32 %1, %2\n\ts_cselect_b32 %0, %1, 0" : "=s"(slot) : "s"(nslot), "i"(NS * STG) : "scc");
    if (!PF) MAX2(1, t - 1);
    MFMA(2, t);
    if (PF && t == 1) LDR(Xn[1], vx, 1024);
    if (PF && t == 2 && NM == 5) LDR(Xn[NM - 1], vx, 4 * 1024);
    if (PF && t == 4) asm volatile("v_add_u32 %0, 1, %0" : "+v"(vdone));
    if (!PF) MAX2(2, t - 1);
    MFMA(3, t);
    if (PF && t == 1) LDR(Xn[2], vx, 2 * 1024);
    if (MODE == 0 && t == 2) asm volatile("ds_read_b32 %0, %1 offset:%2" : "=v"(fpre) : "v"(SY_FULL), "i"(4 * PAR) : "memory");
    if (!PF) MAX2(3, t - 1);
    if (NM == 5) { MFMA(NM - 1, t); if (!PF) MAX2(NM - 1, t - 1); }
  }
  if (!PF) {
    asm volatile("s_nop 7\n\ts_nop 7" ::: "memory");
#pragma unroll
    for (int m = 0; m < NM; ++m) MAX2(m, 7);
  }
}

template <int NM>
__device__ __forceinline__ void consumer(const Params& p, char* me, LAS char* lds, int lane, int wave, unsigned call,
                                         int gidx, int row0, int row4, int pn, unsigned long long t0) {
  f32x4 acc[NM][8];
#pragma unroll
  for (int m = 0; m < NM; ++m)
#pragma unroll
    for (int t = 0; t < 8; ++t) { acc[m][t] = f32x4{0.f, 0.f, 0.f, 0.f}; asm volatile("" : "+v"(acc[m][t])); }
  bf16x8 Wf[8], Xa[NM], Xb[NM];
  unsigned slot = STG, fpre, vdone = 1;
  float mx0 = 0.f, mx1 = 0.f;
  unsigned nslow = 0;
  const unsigned va0 = lane * 16, vx0 = lane * 16 + wave * 5120, vsy = SY + 4 * wave;
  unsigned va = va0, vx = vx0;
  while (lds_ld1(SY_FULL) == 0) __builtin_amdgcn_s_sleep(1);
  STAMP(8 + wave);
#pragma unroll
  for (int t = 0; t < 7; ++t) LDR(Wf[t], va, WOF + t * 1024);
#pragma unroll
  for (int t = 0; t < NM; ++t) LDR(Xa[t], vx, t * 1024);
  asm volatile("ds_read_b32 %0, %1 offset:4\n\ts_waitcnt lgkmcnt(0)" : "=v"(fpre) : "v"(SY_FULL) : "memory");
  for (int s = 0; s < NSTEP - 2; s += 2) {
    mstep<NM, 0, 0>(acc, Wf, Xa, Xb, va, vx, va0, vx0, vsy, vdone, slot, fpre, s, mx0, mx1, nslow);
    mstep<NM, 0, 1>(acc, Wf, Xb, Xa, va, vx, va0, vx0, vsy, vdone, slot, fpre, s + 1, mx0, mx1, nslow);
  }
  mstep<NM, 1, 0>(acc, Wf, Xa, Xb, va, vx, va0, vx0, vsy, vdone, slot, fpre, NSTEP - 2, mx0, mx1, nslow);
  mstep<NM, 2, 1>(acc, Wf, Xb, Xa, va, vx, va0, vx0, vsy, vdone, slot, fpre, NSTEP - 1, mx0, mx1, nslow);
  STAMP(12 + wave);
  unsigned wm = wave_umax(umax(__builtin_bit_cast(unsigned, mx0), __builtin_bit_cast(unsigned, mx1)));
  {
    unsigned tk, m4; unsigned long long pp;
    asm volatile("s_mov_b64 exec, 1\n\tds_max_u32 %3, %4\n\tds_add_rtn_u32 %0, %3, %5 offset:8\n\tds_read_b32 %1, %3\n\t"
                 "s_mov_b64 exec, -1\n\tds_read_b64 %2, %6\n\ts_waitcnt lgkmcnt(0)"
                 : "=&v"(tk), "=&v"(m4), "=&v"(pp) : "v"(SY_WMAX), "v"(wm), "v"(1u), "v"(SY_PTR + (lane & 7) * 8) : "memory");
    if (rfl(tk) == 3u) {
      const char* g = (const char*)pp + GR_OFF + (size_t)(((call & 1) * NR + p.rank) * 256 + gidx) * 8;
      asm volatile("s_mov_b64 exec, 0xff\n\ts_nop 3\n\tglobal_store_dwordx2 %0, %1, off sc0 sc1\n\ts_mov_b64 exec, 1\n\t"
                   "ds_write_b32 %2, %3 offset:4\n\ts_mov_b64 exec, -1\n\ts_nop 1"
                   :: "v"(g), "v"(u32x2{rfl(m4), call}), "v"(SY_WMAX), "v"(1u) : "memory");
    }
  }
  char* ob = p.out + pn * 128 + 16 * (lane >> 4) + (FULLN ? (lane & 7) * NLOC + (lane & 8) * 8 : (lane & 15) * NLOC);
  unsigned used = 0xffffffffu;
  for (;;) {
    u32x4 g;
    asm volatile("s_mov_b64 exec, 1\n\tds_read2_b64 %0, %1 offset1:2\n\ts_mov_b64 exec, -1\n\ts_waitcnt lgkmcnt(0)" : "=&v"(g) : "v"(SY_WMAX) : "memory");
    const unsigned st = rfl(g[3]);
    const bool bad = st == 3u, fin = st >= 2u;
    const unsigned gb = bad ? 0xfffffffeu : fin ? rfl(g[2]) : st == 1u && rfl(g[1]) ? umax(rfl(g[2]), rfl(g[0])) : used;
    if (gb != used) {
      used = gb;
      const float gmax = __builtin_bit_cast(float, gb);
      const float inv = bad ? 0.f : gmax > 0.f ? 127.0f / gmax : 0.f, mag = bad ? 128.f : 0.f;
      const f32x2 inv2 = {inv, inv}, mag2 = {mag, mag};
      if (fin) STAMP(16 + wave);
      STAMP(48 + wave);
#pragma unroll
      for (int m = 0; m < NM; ++m) {
        u32x4 o[2];
#pragma unroll
        for (int hf = 0; hf < 2; ++hf) {
          f32x2 q[4][2];
#pragma unroll
          for (int c = 0; c < 4; ++c)
#pragma unroll
            for (int jj = 0; jj < 2; ++jj)
              q[c][jj] = __builtin_elementwise_fma(f32x2{acc[m][4 * hf + c][2 * jj], acc[m][4 * hf + c][2 * jj + 1]}, inv2, mag2);
#pragma unroll
          for (int j = 0; j < 4; ++j) {
            unsigned w = 0;
#pragma unroll
            for (int c = 0; c < 4; ++c) w = __builtin_amdgcn_cvt_pk_u8_f32(q[c][j >> 1][j & 1], c, w);
            o[hf][j] = w;
          }
        }
#pragma unroll
        for (int j = 0; j < 4; ++j) {
          const unsigned lo = o[0][j], hi = o[1][j];
          o[0][j] = __builtin_amdgcn_update_dpp(lo, hi, 0x128, 0xf, 0xc, false);
          o[1][j] = __builtin_amdgcn_update_dpp(hi, lo, 0x128, 0xf, 0x3, false);
        }
        char* dst = ob + (size_t)(m < 4 ? row0 + 64 * wave + 16 * m : row4 + 16 * wave) * NLOC;
        asm volatile("global_store_dwordx4 %0, %1, off" OSC "\n\ts_nop 1" :: "v"(dst), "v"(o[0]) : "memory");
        asm volatile("global_store_dwordx4 %0, %1, off" OSC "\n\ts_nop 1" :: "v"(dst + 8 * NLOC), "v"(o[1]) : "memory");
      }
      STAMP(52 + wave);
    }
    if (fin) break;
    __builtin_amdgcn_s_sleep(1);
  }
  STAMP(20 + wave);
}

__global__ __launch_bounds__(512, 2) void fused(Params p) {
  extern __shared__ char smem[];
  LAS char* lds = (LAS char*)smem;
  const int lane = threadIdx.x & 63;
  const int wave = __builtin_amdgcn_readfirstlane(threadIdx.x >> 6);
  const int b = blockIdx.x;
  const int rank = p.rank;
  char* me = p.ws[rank];
  const char* tmo = me + TMO_OFF;
  if (wave == 0) STAMP(0);
  const unsigned long long ts = rt();

  unsigned* cntp = (unsigned*)(me + CNT_OFF + b * 64);
  unsigned cnt0, cnt1;
  asm volatile("global_load_dword %0, %1, off sc0 sc1" : "=&v"(cnt0) : "v"(cntp) : "memory");
#define GETCNT(N) asm volatile("s_waitcnt vmcnt(" #N ")\n\tv_readfirstlane_b32 %0, %1" : "=s"(cnt1) : "v"(cnt0) : "memory")
  const int wt0 = b - NCOMM + NGEMM * (7 - wave);
  const bool wup = b >= NCOMM && wave >= 8 - UPF;
  f32x4 xv[8], xw[8];
  WRegs w0;
  const int rb = NWG - 1 - b;
  const int ub = rb < NPUSH ? rb : b < NPULL ? b + NPUSH : b < NCOMM ? b + NGEMM : b - NCONV + NPUSH;
  const bool pusher = rb >= NPUSH && rb < NPUSH * NSPL;
  const int pu = NPUSH ? rb % (NPUSH + !NPUSH) : 0, pj = NPUSH ? rb / (NPUSH + !NPUSH) : 0;
  if (XDLY && rb >= NPUSH * NSPL && b >= NPULL) __builtin_amdgcn_s_sleep(XDLY);
  if (wave < 4) {
    const float* xp = p.x + (size_t)(64 * wave + (lane >> 3)) * KD + 4 * (lane & 7);
    if (pusher) {
#pragma unroll
      for (int i = 0; i < 8; ++i) xw[i] = *(const f32x4*)(xp + (size_t)(256 * (pu & 1) + i * 8) * KD + 32 * (pu >> 1));
    }
#pragma unroll
    for (int i = 0; i < 8; ++i) xv[i] = __builtin_nontemporal_load((const f32x4*)(xp + (size_t)(256 * (ub & 1) + i * 8) * KD + 32 * (ub >> 1)));
    if (pusher) GETCNT(16); else GETCNT(8);
  } else if (wup) {
    w_load(w0, p, wt0 & 15, wt0 >> 4, lane);
    GETCNT(16);
  } else {
    if (wave == 4 && lane < 36) lds_st(SY + lane * 4, 0u);
    if (wave == 5 && lane < NR) *(LAS unsigned long long*)(lds + SY_PTR + lane * 8) = (unsigned long long)p.ws[lane];
    GETCNT(0);
  }
  const unsigned call = cnt1 + 1u;
  asm volatile("s_waitcnt lgkmcnt(0)" ::: "memory");
  __builtin_amdgcn_s_barrier();
  cfence();
  if (wave == 7 && lane == 0) __hip_atomic_store(cntp, call, __ATOMIC_RELAXED, __HIP_MEMORY_SCOPE_SYSTEM);
  if (wup) {
    w_store(w0, me, wt0 & 15, wt0 >> 4, lane);
    w_flag(me, wt0 & 15, wt0 >> 4, lane, call);
    if (wave == 7) STAMP(2);
  }
  if (wave < 4) {
    if (pusher) pack<true>(p, xw, me, lds, lane, wave, call, pu, pj);
    pack<false>(p, xv, me, lds, lane, wave, call, ub, 0);
  }
  __builtin_amdgcn_s_waitcnt(0x0f70);
  const unsigned long long t0 = rt();

  if (b < NCOMM) {
    const bool conv = b >= NPULL;
    const char* tkp = me + TK_OFF + (conv ? 640 : (call & 1) * 128);
    if (b == 0 && wave == 7 && lane == 0) st4(me + TK_OFF + ((call + 1) & 1) * 128, 0u);
    if (wave == 7 && !conv) {
      const char* sf = me + SF_OFF + lane * 16;
      for (;;) {
        u32x4 f[8]; unsigned t;
        asm volatile("global_load_dwordx4 %0, %9, off sc0 sc1\n\tglobal_load_dwordx4 %1, %9, off offset:1024 sc0 sc1\n\t"
                     "global_load_dwordx4 %2, %9, off offset:2048 sc0 sc1\n\tglobal_load_dwordx4 %3, %9, off offset:3072 sc0 sc1\n\t"
                     "global_load_dwordx4 %4, %10, off sc0 sc1\n\tglobal_load_dwordx4 %5, %10, off offset:1024 sc0 sc1\n\t"
                     "global_load_dwordx4 %6, %10, off offset:2048 sc0 sc1\n\tglobal_load_dwordx4 %7, %10, off offset:3072 sc0 sc1\n\t"
                     "global_load_dword %8, %11, off sc0 sc1\n\ts_waitcnt vmcnt(0)"
                     : "=&v"(f[0]), "=&v"(f[1]), "=&v"(f[2]), "=&v"(f[3]), "=&v"(f[4]), "=&v"(f[5]), "=&v"(f[6]), "=&v"(f[7]), "=&v"(t)
                     : "v"(sf), "v"(sf + 4096), "v"(tmo) : "memory");
        unsigned all = 256;
#pragma unroll
        for (int j = 0; j < NR; ++j) {
          const unsigned c = f[j][0] < call ? 0u : f[j][1] < call ? 1u : f[j][2] < call ? 2u : f[j][3] < call ? 3u : 4u;
          const unsigned long long part = __builtin_amdgcn_ballot_w64(c < 4u && lane >= NPUSH / 4);
          unsigned r = 256;
          if (part && j != rank) { const unsigned l = __builtin_ctzll(part); r = 4 * l + __builtin_amdgcn_readlane(c, l); }
          lds_st(SY_SRC + 4 * j, r);
          all = umin(all, r);
        }
        if (all == 256) break;
        if (expired(t, call, t0, me)) {
#pragma unroll
          for (int j = 0; j < NR; ++j) lds_st(SY_SRC + 4 * j, ABORT);
          break;
        }
      }
    }
    const unsigned v0 = lane * 16, vc = lane * 12, vh = (lane < 48 ? lane : 47) * 4;
    const char* fl = me + TK_OFF + 512;
    unsigned T = NPUSH * 14 + (conv ? NWP * NPULL + wave * NCONV + (b - NPULL) : wave * NPULL + b);
    const int nw = conv ? CPULL : NWP;
    for (bool first = true; wave < nw; first = false) {
      if (T >= 7u * 512u) break;
      const unsigned hu = T / 7u, src = (rank + 1 + (T - hu * 7u)) & 7u, u = hu >> 1, r = hu & 1, s = u >> 1, h = u & 1, n = u >> 2;
      unsigned av = lds_ld1(SY_SRC + 4 * src);
      if (first) {
        while (rt() - ts < (unsigned long long)(TST0 + TSTD * (hu - 2 * NPUSH))) __builtin_amdgcn_s_sleep(1);
        const char* fp = me + PF_OFF + (size_t)(src * 64 + u) * 128;
        for (;;) {
          unsigned f, t;
          asm volatile("global_load_dword %0, %2, %3 sc0 sc1\n\tglobal_load_dword %1, %2, %4 sc0 sc1\n\ts_waitcnt vmcnt(0)"
                       : "=&v"(f), "=&v"(t) : "v"(0), "s"(fp), "s"(tmo) : "memory");
          av = u + 1;
          if (rfl(f) >= call) break;
          av = ABORT;
          if (expired(t, call, t0, me)) break;
          __builtin_amdgcn_s_sleep(1);
        }
      }
      if (av <= u) {
        waitv<0>();
        flag_st(fl, call);
        fl = me + TK_OFF + 512;
        do { __builtin_amdgcn_s_sleep(1); av = lds_ld1(SY_SRC + 4 * src); } while (av <= u);
      }
      if (av >= ABORT) break;
      if (STAG && first) for (unsigned i = 2 * NPUSH; i < hu; ++i) __builtin_amdgcn_s_sleep(STAG);
      if (n < 8 && !conv) STAMP(8 + n);
      u32x4 A[4]; u32x3 P[2]; unsigned E, tk;
      const char* sws = p.ws[src];
      const char* sb = sws + PK_OFF + (size_t)u * PKU;
      asm volatile("s_nop 4\n\t"
                   "global_load_dword %6, %10, %13 sc0 sc1\n\t"
                   "global_load_dwordx3 %4, %9, %12 sc0 sc1\n\t"
                   "global_load_dwordx3 %5, %9, %12 offset:768 sc0 sc1\n\t"
                   "global_load_dwordx4 %0, %8, %11 sc0 sc1\n\t"
                   "global_load_dwordx4 %1, %8, %11 offset:1024 sc0 sc1\n\t"
                   "global_load_dwordx4 %2, %8, %11 offset:2048 sc0 sc1\n\t"
                   "global_load_dwordx4 %3, %8, %11 offset:3072 sc0 sc1\n\t"
                   "s_mov_b64 exec, 1\n\ts_nop 3\n\t"
                   "global_atomic_add %7, %8, %16, %14 sc0\n\t"
                   "s_waitcnt vmcnt(8)\n\t"
                   "global_store_dword %8, %17, %15 sc0 sc1\n\t"
                   "s_mov_b64 exec, -1\n\t"
                   "s_waitcnt vmcnt(1)"
                   : "=&v"(A[0]), "=&v"(A[1]), "=&v"(A[2]), "=&v"(A[3]), "=&v"(P[0]), "=&v"(P[1]), "=&v"(E), "=&v"(tk)
                   : "v"(v0), "v"(vc), "v"(vh), "s"(sb + r * 4096), "s"(sb + PK_C + r * 1536), "s"(sb + PK_E + r * PK_R),
                     "s"(tkp), "s"(fl), "v"(1u), "v"(call) : "memory");
      const size_t xo = XB_OFF + (size_t)((src * 2 + h) * NSTEP + s) * XSTG + r * 8192;
      decode(A, P, E, lds, wave * XSTG, me + xo, sws + xo, lane);
      fl = me + XF_OFF + (size_t)(((src * 2 + h) * 2 + r) * NSTEP + s) * 4;
      if (n < 8) STAMP(16 + n);
      if ((n & 7) == 7) STAMP(48 + n / 8);
      if (conv) break;
      T = umin(rfl(tk), 7u * 512u) + NWP * NPULL + CPULL * NCONV + NPUSH * 14;
    }
    if (wave < nw) { waitv<0>(); flag_st(fl, call); }
    if (conv) {
      for (;;) {
        const int t = NGEMM * (UPF + UPC) + (b - NPULL) + NCONV * (int)lds_ticket(SY_TKT);
        if (t >= 16 * NSTEP) break;
        if ((t & 255) < NCONV) STAMP(8 + (t >> 8));
        convert_w(p, me, t & 15, t >> 4, lane, call);
      }
      if (wave == 0) STAMP(2);
      return;
    }
    if (wave < NWP) { STAMP(24 + wave); lds_st(SY_TKT, 64u); }
    if (b >= 8) return;
    if (wave >= NWP) {
      while (lds_ld1(SY_TKT) < 64u) __builtin_amdgcn_s_sleep(32);
    }
    const int lg = lane < NGEMM / 4 ? lane : NGEMM / 4 - 1;
    const char* g0 = me + GR_OFF + (size_t)(((call & 1) * NR + wave) * 256) * 8 + lg * 32;
    unsigned seen = 0, told = 0xffffffffu;
    const unsigned long long tp = rt(), real = (1ull << (NGEMM / 4)) - 1;
    const char* gp = me + GM_OFF + b * 128;
    for (;;) {
      u32x4 a0, a1;
      asm volatile("global_load_dwordx4 %0, %2, off sc0 sc1\n\tglobal_load_dwordx4 %1, %2, off offset:16 sc0 sc1\n\ts_waitcnt vmcnt(0)"
                   : "=&v"(a0), "=&v"(a1) : "v"(g0) : "memory");
      const unsigned cnt = __builtin_popcountll(__builtin_amdgcn_ballot_w64(a0[1] == call) & real) + __builtin_popcountll(__builtin_amdgcn_ballot_w64(a0[3] == call) & real)
                         + __builtin_popcountll(__builtin_amdgcn_ballot_w64(a1[1] == call) & real) + __builtin_popcountll(__builtin_amdgcn_ballot_w64(a1[3] == call) & real);
      if (cnt != seen && (SPEC || cnt == (unsigned)NGEMM)) {
        const unsigned part = wave_umax(umax(umax(a0[1] == call ? a0[0] : 0u, a0[3] == call ? a0[2] : 0u),
                                             umax(a1[1] == call ? a1[0] : 0u, a1[3] == call ? a1[2] : 0u)));
        unsigned v0; u32x2 gt;
        asm volatile("s_mov_b64 exec, 1\n\tds_max_u32 %2, %3\n\tds_add_rtn_u32 %0, %2, %4 offset:4\n\tds_read_b64 %1, %2\n\t"
                     "s_mov_b64 exec, -1\n\ts_waitcnt lgkmcnt(0)"
                     : "=&v"(v0), "=&v"(gt) : "v"(SY_GMAX), "v"(part), "v"(cnt - seen) : "memory");
        const unsigned gm = rfl(gt[0]), tot = rfl(v0) + cnt - seen;
        seen = cnt;
        if (tot == (unsigned)(NR * NGEMM)) {
          asm volatile("s_mov_b64 exec, 1\n\ts_nop 3\n\tglobal_store_dwordx2 %0, %1, %2" GSC "\n\t"
                       "global_store_dwordx2 %0, %1, %2 offset:1024 sc0 sc1\n\ts_mov_b64 exec, -1\n\ts_nop 1"
                       :: "v"(0), "v"(u32x2{gm, call}), "s"(gp) : "memory");
          STAMP(4);
        } else if (SPEC && tot >= SPEC && gm != told) {
          told = gm;
          asm volatile("s_mov_b64 exec, 1\n\ts_nop 3\n\tglobal_store_dwordx2 %0, %1, %2 offset:8" GSC "\n\ts_mov_b64 exec, -1\n\ts_nop 1"
                       :: "v"(0), "v"(u32x2{gm, call}), "s"(gp) : "memory");
        }
        if (cnt == (unsigned)NGEMM) break;
        continue;
      }
      if (rt() - tp > 3000u) {
        unsigned t;
        asm volatile("global_load_dword %0, %1, %2 sc0 sc1\n\ts_waitcnt vmcnt(0)" : "=&v"(t) : "v"(0), "s"(tmo) : "memory");
        if (expired(t, call, t0, me)) break;
      }
      if (ASLP) __builtin_amdgcn_s_sleep(ASLP);
    }
    return;
  }

  const int xcd = b & 7, idx = (b >> 3) - NCOMM / 8, xh = xcd & 1;
  const bool extra = idx < 24;
  const int di = extra ? idx >> 2 : 6;
  const int pn = (xcd >> 1) * 4 + (extra ? idx & 3 : 2 * xh + (idx & 1));
  const int src = (rank + 1 + di) & 7, h = extra ? xh : 0;
  const int esrc = di < 4 ? rank : (rank + 7) & 7, eh = di < 4 ? xh : extra ? 1 : 0, eq = di < 4 ? di : 2 * xh + (di & 1);
  if (wave < 4) {
    if (wave < UPC) {
      const int t = NGEMM * (UPF + wave) + b - NCOMM;
      convert_w(p, me, t & 15, t >> 4, lane, call);
      if (wave == UPC - 1) STAMP(7);
    }
    __builtin_amdgcn_s_setprio(PRIOC);
    if (extra) consumer<5>(p, me, lds, lane, wave, call, b - NCOMM, src * 512 + h * 256, esrc * 512 + eh * 256 + eq * 64, pn, t0);
    else consumer<4>(p, me, lds, lane, wave, call, b - NCOMM, src * 512 + h * 256, 0, pn, t0);
    return;
  }
  bool dead = false;
  if (wave < 6) {
    __builtin_amdgcn_s_setprio(PRIOL);
    const char* xs = me + XB_OFF + (size_t)((src * 2 + h) * NSTEP) * XSTG;
    const char* ls = me + XB_OFF + (size_t)((esrc * 2 + eh) * NSTEP) * XSTG + eq * 4096;
    const char* wsr = me + WB_OFF + (size_t)(pn * NSTEP) * WSTG;
    const unsigned voff = lane * 16, pad = lane < 4 ? SY_DONE + 4 * lane : SY_READY;
    unsigned slot = (wave - 4) * STG;
    const int fw = SY_FULL + 4 * (wave - 4);
    const char* tok = me + IO_OFF + (size_t)(b * 2 + (wave - 4)) * 256;
    st4((char*)tok + lane * 4, 2 * lane + (wave - 4) + 1);
    waitv<0>();
    for (;;) {
      u32x4 d;
      asm volatile("ds_read_b128 %0, %1\n\ts_waitcnt lgkmcnt(0)" : "=&v"(d) : "v"(SY_PACK) : "memory");
      if (rfl(d[0] & d[1] & d[2] & d[3])) break;
      __builtin_amdgcn_s_sleep(2);
    }
    for (int s = wave - 4; s < NSTEP; s += 2) {
      unsigned rdy;
      for (;;) {
        unsigned r, dm, d1, d2, d3;
        asm volatile("s_mov_b64 exec, 31\n\tds_read_b32 %0, %6\n\ts_mov_b64 exec, -1\n\ts_waitcnt lgkmcnt(0)\n\t"
                     "v_readlane_b32 %1, %0, 4\n\tv_readlane_b32 %2, %0, 0\n\tv_readlane_b32 %3, %0, 1\n\t"
                     "v_readlane_b32 %4, %0, 2\n\tv_readlane_b32 %5, %0, 3\n\ts_nop 0\n\t"
                     "s_min_u32 %2, %2, %3\n\ts_min_u32 %4, %4, %5\n\ts_min_u32 %2, %2, %4"
                     : "=&v"(r), "=s"(rdy), "=s"(dm), "=s"(d1), "=s"(d2), "=s"(d3) : "v"(pad) : "memory", "scc");
        if (rdy >= ABORT || (rdy > (unsigned)s && dm + NS > (unsigned)s)) break;
        __builtin_amdgcn_s_sleep(SLEEPL);
      }
      if (rdy >= ABORT) { dead = true; break; }
      const int sa = s;
      const char* xa = xs + (size_t)sa * XSTG;
#define PACE_
      glds4(voff, wsr + (size_t)sa * WSTG, slot + WOF); PACE_ glds4(voff, wsr + (size_t)sa * WSTG + 4096, slot + WOF + 4096); PACE_
      glds4(voff, xa, slot); PACE_ glds4(voff, xa + 4096, slot + 5120); PACE_ glds4(voff, xa + 8192, slot + 10240); PACE_ glds4(voff, xa + 12288, slot + 15360); PACE_
      if (extra) glds4s(voff, ls + (size_t)sa * XSTG, slot + 4096);
      asm volatile("s_mov_b32 m0, %2\n\ts_mov_b64 exec, 1\n\ts_nop 1\n\tglobal_load_lds_dword %0, %1 sc0 sc1\n\ts_mov_b64 exec, -1"
                   :: "v"(voff), "s"(tok + (s >> 1) * 4), "s"(fw) : "memory");
      slot += 2 * STG; if (slot >= NS * STG) slot -= NS * STG;
    }
    waitv<0>();
    if (dead) lds_st(fw, ABORT);
    __builtin_amdgcn_s_setprio(0);
  } else if (wave == 6) {
    __builtin_amdgcn_s_setprio(PRIOP);
    const char* pa = lane < 32 ? me + XF_OFF + (size_t)((src * 2 + h) * 2 * NSTEP) * 4 + lane * 16
                               : me + WF_OFF + (size_t)(pn * NSTEP) * 4 + (lane & 31) * 16;
    const char* pb = me + XF_OFF + (size_t)((lane < 32 ? (esrc * 2 + eh) * 2 + (eq >> 1) : (src * 2 + h) * 2 + 1) * NSTEP) * 4 + (lane & 31) * 16;
    unsigned ready = 0;
    while (!dead && ready < (unsigned)NSTEP) {
      if (ready > lds_ld1(SY_DONE) + NS + 2) { __builtin_amdgcn_s_sleep(SLEEPP); continue; }
      u32x4 f, g; unsigned t;
      asm volatile("global_load_dwordx4 %0, %3, off sc0 sc1\n\tglobal_load_dwordx4 %1, %4, off sc0 sc1\n\t"
                   "global_load_dword %2, %5, off sc0 sc1\n\ts_waitcnt vmcnt(0)"
                   : "=&v"(f), "=&v"(g), "=&v"(t) : "v"(pa), "v"(pb), "v"(tmo) : "memory");
#pragma unroll
      for (int i = 0; i < 4; ++i) f[i] = umin(f[i], g[i]);
      unsigned c = f[0] < call ? 0u : f[1] < call ? 1u : f[2] < call ? 2u : f[3] < call ? 3u : 4u;
      unsigned long long part = __builtin_amdgcn_ballot_w64(c < 4u);
      unsigned px = (unsigned)part, pw = (unsigned)(part >> 32);
      unsigned rx = NSTEP, rw = NSTEP;
      if (px) { unsigned l = __builtin_ctz(px); rx = 4 * l + __builtin_amdgcn_readlane(c, l); }
      if (pw) { unsigned l = __builtin_ctz(pw); rw = 4 * l + __builtin_amdgcn_readlane(c, 32 + l); }
      unsigned r = umin(rx, rw);
      if (r > ready) {
        ready = r; lds_st(SY_READY, ready); continue; }
      if (expired(t, call, t0, me)) { dead = true; break; }
      __builtin_amdgcn_s_sleep(1);
    }
    if (dead) lds_st(SY_READY, ABORT);
    STAMP(3);
    while (!dead && lds_ld1(SY_DONE) < (unsigned)(NSTEP - EPOLL)) __builtin_amdgcn_s_sleep(4);
    const char* gp = me + GM_OFF + xcd * 128;
    unsigned told = 0xffffffffu;
    const unsigned long long tp = rt();
    for (;;) {
      u32x4 g;
      asm volatile("global_load_dwordx4 %0, %1, %2" GLC "\n\ts_waitcnt vmcnt(0)" : "=&v"(g) : "v"(0), "s"(gp) : "memory");
      bool bad = false, fin = rfl(g[1]) == call;
      if (!fin && rt() - tp > 2000u) {
        u32x2 h2; unsigned t;
        asm volatile("global_load_dwordx2 %0, %2, %3 offset:1024 sc0 sc1\n\tglobal_load_dword %1, %2, %4 sc0 sc1\n\ts_waitcnt vmcnt(0)"
                     : "=&v"(h2), "=&v"(t) : "v"(0), "s"(gp), "s"(tmo) : "memory");
        bad = expired(t, call, t0, me);
        fin = bad || rfl(h2[1]) == call;
        g[0] = h2[0];
      }
      const unsigned v = fin ? rfl(g[0]) : SPEC && rfl(g[3]) == call ? rfl(g[2]) : told;
      if (fin || v != told) {
        if (fin) STAMP(58); else STAMP(59);
        told = v;
        asm volatile("ds_write_b64 %0, %1" :: "v"(SY_GMAX), "v"(u32x2{v, bad ? 3u : fin ? 2u : 1u}) : "memory");
      }
      if (fin) break;
      __builtin_amdgcn_s_sleep(1);
    }
  } else {
    for (unsigned T = b - NCOMM; T < NPUSH * 14u; T += NGEMM) {
      const unsigned hu = T / 7u, sr = (rank + 1 + (T - hu * 7u)) & 7u, u = hu >> 1, r = hu & 1;
      auto wait_flag = [&](const char* fp) {
        for (;;) {
          unsigned f, t;
          asm volatile("global_load_dword %0, %2, %3 sc0 sc1\n\tglobal_load_dword %1, %2, %4 sc0 sc1\n\ts_waitcnt vmcnt(0)"
                       : "=&v"(f), "=&v"(t) : "v"(0), "s"(fp), "s"(tmo) : "memory");
          if (rfl(f) >= call) break;
          if (expired(t, call, t0, me)) { dead = true; break; }
          __builtin_amdgcn_s_sleep(1);
        }
      };
      wait_flag(me + PF_OFF + (size_t)(sr * 64 + u) * 128);
      if (dead) break;
      STAMP(4);
      u32x4 A[4]; u32x3 P[2]; unsigned E;
      const char* sb = me + LD_OFF + (size_t)(sr * NPUSH + u) * PKU;
      asm volatile("s_nop 4\n\t"
                   "global_load_dword %6, %9, %12 sc0 sc1\n\t"
                   "global_load_dwordx3 %4, %8, %11 sc0 sc1\n\t"
                   "global_load_dwordx3 %5, %8, %11 offset:768 sc0 sc1\n\t"
                   "global_load_dwordx4 %0, %7, %10 sc0 sc1\n\t"
                   "global_load_dwordx4 %1, %7, %10 offset:1024 sc0 sc1\n\t"
                   "global_load_dwordx4 %2, %7, %10 offset:2048 sc0 sc1\n\t"
                   "global_load_dwordx4 %3, %7, %10 offset:3072 sc0 sc1\n\t"
                   "s_waitcnt vmcnt(0)"
                   : "=&v"(A[0]), "=&v"(A[1]), "=&v"(A[2]), "=&v"(A[3]), "=&v"(P[0]), "=&v"(P[1]), "=&v"(E)
                   : "v"(lane * 16), "v"(lane * 12), "v"((lane < 48 ? lane : 47) * 4), "s"(sb + r * 4096), "s"(sb + PK_C + r * 1536),
                     "s"(sb + PK_E + r * PK_R) : "memory");
      const size_t xo = XB_OFF + (size_t)((sr * 2 + (u & 1)) * NSTEP + (u >> 1)) * XSTG + r * 8192;
      const unsigned c0 = (rfl(E) >> 8) & 0xffu, c1 = ((unsigned)__builtin_amdgcn_readlane(E, 47) >> 8) & 0xffu;
      if (c0 + c1 + 2u > EXC) wait_flag(me + SF_OFF + (size_t)(sr * 256 + u) * 4);
      if (dead) break;
      decode(A, P, E, lds, SCR, me + xo, p.ws[sr] + xo, lane);
      waitv<0>();
      flag_st(me + XF_OFF + (size_t)(((sr * 2 + (u & 1)) * 2 + r) * NSTEP + (u >> 1)) * 4, call);
      STAMP(5);
    }
  }
}

}

extern "C" void kernel_launch(int rank, int nranks, ncclComm_t comm, hipStream_t stream,
                              void* const* d_in, const int* in_sizes, int n_in,
                              void* d_out, int out_size, const int* in_dtypes, int out_dtype,
                              void* const* d_out_all, void* const* d_ws_all, size_t ws_bytes) {
  if (nranks != NR || (unsigned)rank >= (unsigned)NR || n_in < 2 || ws_bytes < WS_NEED ||
      in_sizes[0] != 512 * KD || out_size != 4096 * NLOC) {
    fprintf(stderr, "kernel_launch: unexpected problem shape\n");
    return;
  }
  HIP_CHECK(hipFuncSetAttribute((const void*)fused, hipFuncAttributeMaxDynamicSharedMemorySize, LDS_BYTES));
  Params p;
  p.x = (const float*)d_in[0]; p.w = (const float*)d_in[1]; p.out = (char*)d_out; p.rank = rank;
  for (int i = 0; i < NR; ++i) p.ws[i] = (char*)d_ws_all[i];
  fused<<<NWG, 512, LDS_BYTES, stream>>>(p);
  HIP_CHECK(hipGetLastError());
}
```

```cpp
#include <hip/hip_runtime.h>
#include <rccl/rccl.h>
#include <cstdio>
#include <cstdint>

#define HIP_CHECK(expr) do { hipError_t _e = (expr); if (_e != hipSuccess) { \
    fprintf(stderr, "HIP error %s at %s:%d\n", hipGetErrorString(_e), __FILE__, __LINE__); \
    return; } } while (0)

namespace {

typedef unsigned u32x4 __attribute__((ext_vector_type(4)));
typedef unsigned u32x3 __attribute__((ext_vector_type(3)));
typedef unsigned u32x2 __attribute__((ext_vector_type(2)));
typedef unsigned short u16x2 __attribute__((ext_vector_type(2)));
typedef float f32x4 __attribute__((ext_vector_type(4)));
typedef float f32x2 __attribute__((ext_vector_type(2)));
typedef __bf16 bf16x8 __attribute__((ext_vector_type(8)));
typedef __bf16 bf16x2 __attribute__((ext_vector_type(2)));
#define LAS __attribute__((address_space(3)))

#define FORCE_RAW 0
#define UPF 1
#define PRIOC 3
#define PRIOL 0
#define PRIOP 0
#define SLEEPL 1
#define SLEEPP 4
#define STAG 0
#define LSC " sc0 sc1"
#define GNOP 2
#define GREP 1
#define STR2(x) #x
#define STR(x) STR2(x)
#define GAP ".rept " STR(GREP) "\n\ts_nop " STR(GNOP) "\n\t.endr\n\t"
#define NPUSH 12
#define PSTAG 0
#define SPEC 800
#define GSCN 0
#define GSC ""
#define GLC " sc1"
#define ASLP 0
#define WH 1
#define WH0 1050
#define WHD 20
#define EPOLL 8
#define FULLN 1
#define OSCN 1
#define OSC " sc0 sc1"
#define POLL1 1
#define XDLY 40
#define CPULL 0
#define TST0 450
#define TSTD 10
#define UPC 0
#define NSPL 3
#define SPREAD 0
#define NWP 6

constexpr int NR = 8;
constexpr int KD = 4096, ND = 16384, NLOC = 2048;
constexpr int NSTEP = 128;
constexpr int NPULL = 28, NCOMM = 48, NCONV = NCOMM - NPULL, NGEMM = 208, NWG = 256;
constexpr int XSTG = 16384, WSTG = 8192;
constexpr int PK_C = 8192, PK_E = 11264, PK_R = 192, PKU = PK_E + 2 * PK_R;
constexpr unsigned EXC = (PK_R - 8) / 2;
constexpr size_t XB_OFF = 0;
constexpr size_t WB_OFF = 32u << 20;
constexpr size_t PK_OFF = 48u << 20;
constexpr size_t LD_OFF = 52u << 20;
constexpr size_t FL_OFF = 64u << 20;
constexpr size_t CNT_OFF = FL_OFF;
constexpr size_t TMO_OFF = FL_OFF + 0x4000;
constexpr size_t SF_OFF = FL_OFF + 0x5000;
constexpr size_t XF_OFF = FL_OFF + 0x60000;
constexpr size_t WF_OFF = FL_OFF + 0xA000;
constexpr size_t GM_OFF = FL_OFF + 0xC000;
constexpr size_t TK_OFF = FL_OFF + 0xD000;
constexpr size_t GR_OFF = FL_OFF + 0x10000;
constexpr size_t ST_OFF = FL_OFF + 0x20000;
constexpr size_t IO_OFF = FL_OFF + 0x40000;
constexpr size_t PF_OFF = FL_OFF + 0x64000;
constexpr size_t WS_NEED = FL_OFF + 0x74000;
constexpr int NS = 5;
constexpr int STG = 28672;
constexpr int SY = NS * STG;
constexpr int SY_FULL = SY, SY_READY = SY + 8, SY_DONE = SY + 16, SY_ARR = SY + 32, SY_WMAX = SY + 48,
              SY_GMAX = SY + 64, SY_TKT = SY + 84, SY_PACK = SY + 96, SY_SRC = SY + 112, SY_PTR = SY + 144;
constexpr int SCR = SY + 512;
constexpr int LDS_BYTES = SCR + 8192;
constexpr unsigned long long DEADLINE = 50000000ull;
constexpr unsigned ABORT = 0x40000000u;
static_assert(NPUSH % 4 == 0 && NPUSH * 14 + NWP * NPULL + CPULL * NCONV <= 64 * 14, "first round: within the PF flags");

struct Params { const float* x; const float* w; char* out; char* ws[NR]; int rank; };

__device__ __forceinline__ unsigned long long rt() { return __builtin_amdgcn_s_memrealtime(); }
template <int N_> __device__ __forceinline__ void waitv() { asm volatile("s_waitcnt vmcnt(%0)" :: "i"(N_) : "memory"); }
__device__ __forceinline__ void cfence() { asm volatile("" ::: "memory"); }
__device__ __forceinline__ void st16(char* p, u32x4 v) {
  asm volatile("global_store_dwordx4 %0, %1, off sc0 sc1\n\ts_nop 1" :: "v"(p), "v"(v) : "memory");
}
__device__ __forceinline__ void st4(char* p, unsigned v) {
  asm volatile("global_store_dword %0, %1, off sc0 sc1" :: "v"(p), "v"(v) : "memory");
}
__device__ __forceinline__ void flag_st(const char* p, unsigned v) {
  asm volatile("s_mov_b64 exec, 1\n\ts_nop 3\n\tglobal_store_dword %0, %1, %2 sc0 sc1\n\ts_mov_b64 exec, -1" :: "v"(0), "v"(v), "s"(p) : "memory");
}
__device__ __forceinline__ unsigned lds_ticket(int addr) {
  unsigned v, r;
  asm volatile("s_mov_b64 exec, 1\n\tds_add_rtn_u32 %0, %2, %3\n\ts_mov_b64 exec, -1\n\ts_waitcnt lgkmcnt(0)\n\tv_readfirstlane_b32 %1, %0"
               : "=&v"(v), "=s"(r) : "v"(addr), "v"(1u) : "memory");
  return r;
}
__device__ __forceinline__ void poll16(u32x4& a, unsigned& t, const char* p, const char* tm) {
  asm volatile("global_load_dwordx4 %0, %2, off sc0 sc1\n\tglobal_load_dword %1, %3, off sc0 sc1\n\ts_waitcnt vmcnt(0)"
               : "=&v"(a), "=&v"(t) : "v"(p), "v"(tm) : "memory");
}
__device__ __forceinline__ void lds_st(int addr, unsigned v) {
  asm volatile("ds_write_b32 %0, %1" :: "v"(addr), "v"(v) : "memory");
}
__device__ __forceinline__ unsigned lds_ld(int addr) {
  unsigned v; asm volatile("ds_read_b32 %0, %1\n\ts_waitcnt lgkmcnt(0)" : "=&v"(v) : "v"(addr) : "memory"); return v;
}
__device__ __forceinline__ unsigned lds_ld1(int addr) {
  unsigned v, r;
  asm volatile("s_mov_b64 exec, 1\n\tds_read_b32 %0, %2\n\ts_waitcnt lgkmcnt(0)\n\tv_readfirstlane_b32 %1, %0\n\ts_mov_b64 exec, -1"
               : "=&v"(v), "=s"(r) : "v"(addr) : "memory");
  return r;
}
__device__ __forceinline__ unsigned rfl(unsigned v) { return __builtin_amdgcn_readfirstlane(v); }
__device__ __forceinline__ unsigned umax(unsigned a, unsigned b) { return a > b ? a : b; }
__device__ __forceinline__ unsigned umin(unsigned a, unsigned b) { return a < b ? a : b; }
__device__ __forceinline__ unsigned wave_umax(unsigned v) {
  v = umax(v, (unsigned)__builtin_amdgcn_update_dpp(0, (int)v, 0x111, 0xf, 0xf, false));
  v = umax(v, (unsigned)__builtin_amdgcn_update_dpp(0, (int)v, 0x112, 0xf, 0xf, false));
  v = umax(v, (unsigned)__builtin_amdgcn_update_dpp(0, (int)v, 0x114, 0xf, 0xf, false));
  v = umax(v, (unsigned)__builtin_amdgcn_update_dpp(0, (int)v, 0x118, 0xf, 0xf, false));
  unsigned a = __builtin_amdgcn_readlane(v, 15), b = __builtin_amdgcn_readlane(v, 31);
  unsigned c = __builtin_amdgcn_readlane(v, 47), d = __builtin_amdgcn_readlane(v, 63);
  return umax(umax(a, b), umax(c, d));
}
__device__ __forceinline__ unsigned wave_fsum(float v) {
#define DPPADD(ctrl, rm) v += __builtin_bit_cast(float, __builtin_amdgcn_update_dpp(0, __builtin_bit_cast(int, v), ctrl, rm, 0xf, false))
  DPPADD(0x111, 0xf); DPPADD(0x112, 0xf); DPPADD(0x114, 0xf); DPPADD(0x118, 0xf); DPPADD(0x142, 0xa); DPPADD(0x143, 0xc);
  return __builtin_amdgcn_readlane(__builtin_bit_cast(unsigned, v), 63);
}
__device__ __forceinline__ unsigned bfi(unsigned m, unsigned a, unsigned b) { return (a & m) | (b & ~m); }
__device__ __forceinline__ unsigned pk_sub16(unsigned a, unsigned b) {
  return __builtin_bit_cast(unsigned, (u16x2)(__builtin_bit_cast(u16x2, a) - __builtin_bit_cast(u16x2, b)));
}
__device__ __forceinline__ unsigned pk_min16(unsigned a, unsigned b) {
  return __builtin_bit_cast(unsigned, __builtin_elementwise_min(__builtin_bit_cast(u16x2, a), __builtin_bit_cast(u16x2, b)));
}
__device__ __forceinline__ unsigned pk_bf16(float lo, float hi) {
  f32x2 f = {lo, hi};
  return __builtin_bit_cast(unsigned, __builtin_convertvector(f, bf16x2));
}
__device__ __forceinline__ unsigned perm(unsigned hi, unsigned lo, unsigned sel) { return __builtin_amdgcn_perm(hi, lo, sel); }
__device__ __forceinline__ unsigned rotr(unsigned v, unsigned n) { return __builtin_amdgcn_alignbit(v, v, n); }
__device__ __forceinline__ void glds4(unsigned voff, const char* sbase, unsigned dst) {
  asm volatile("s_mov_b32 m0, %2\n\ts_nop 0\n\t"
               "global_load_lds_dwordx4 %0, %1" LSC "\n\t" GAP
               "global_load_lds_dwordx4 %0, %1 offset:1024" LSC "\n\t" GAP
               "global_load_lds_dwordx4 %0, %1 offset:2048" LSC "\n\t" GAP
               "global_load_lds_dwordx4 %0, %1 offset:3072" LSC "\n\t" GAP
               :: "v"(voff), "s"(sbase), "s"(dst) : "memory");
}
__device__ __forceinline__ void glds4s(unsigned voff, const char* sbase, unsigned dst) {
  asm volatile("s_mov_b32 m0, %2\n\ts_nop 0\n\t"
               "global_load_lds_dwordx4 %0, %1" LSC "\n\t" GAP
               "s_add_u32 m0, m0, 4096\n\ts_nop 0\n\t"
               "global_load_lds_dwordx4 %0, %1 offset:1024" LSC "\n\t" GAP
               "s_add_u32 m0, m0, 4096\n\ts_nop 0\n\t"
               "global_load_lds_dwordx4 %0, %1 offset:2048" LSC "\n\t" GAP
               "s_add_u32 m0, m0, 4096\n\ts_nop 0\n\t"
               "global_load_lds_dwordx4 %0, %1 offset:3072" LSC "\n\t" GAP
               :: "v"(voff), "s"(sbase), "s"(dst) : "memory", "scc");
}
#define STAMP(i) do {} while (0)

__device__ __forceinline__ bool expired(unsigned tmo_seen, unsigned call, unsigned long long t0, char* me) {
  if (rfl(tmo_seen) == call) return true;
  if (rt() - t0 > DEADLINE) {
    __hip_atomic_store((unsigned*)(me + TMO_OFF), call, __ATOMIC_RELAXED, __HIP_MEMORY_SCOPE_AGENT);
    return true;
  }
  return false;
}

struct WRegs { f32x4 v[2][8]; };
__device__ __forceinline__ void w_load(WRegs& r, const Params& p, int panel, int s, int lane) {
  const float* wp = p.w + (size_t)(32 * s + 8 * (lane >> 4)) * ND + p.rank * NLOC + panel * 128 + 4 * (lane & 15);
#pragma unroll
  for (int hf = 0; hf < 2; ++hf)
#pragma unroll
    for (int j = 0; j < 8; ++j) r.v[hf][j] = __builtin_nontemporal_load((const f32x4*)(wp + (size_t)j * ND + hf * 64));
}
__device__ __forceinline__ void w_store(const WRegs& r, char* me, int panel, int s, int lane) {
  char* dst = me + WB_OFF + (size_t)(panel * NSTEP + s) * WSTG + lane * 16;
#pragma unroll
  for (int hf = 0; hf < 2; ++hf)
#pragma unroll
    for (int c = 0; c < 4; ++c) {
      u32x4 o = {pk_bf16(r.v[hf][0][c], r.v[hf][1][c]), pk_bf16(r.v[hf][2][c], r.v[hf][3][c]),
                 pk_bf16(r.v[hf][4][c], r.v[hf][5][c]), pk_bf16(r.v[hf][6][c], r.v[hf][7][c])};
      st16(dst + (4 * hf + c) * 1024, o);
    }
}
__device__ __forceinline__ void w_flag(char* me, int panel, int s, int lane, unsigned call) {
  waitv<0>();
  flag_st(me + WF_OFF + (size_t)(panel * NSTEP + s) * 4, call);
}
__device__ __forceinline__ void convert_w(const Params& p, char* me, int panel, int s, int lane, unsigned call) {
  WRegs r;
  w_load(r, p, panel, s, lane);
  w_store(r, me, panel, s, lane);
  w_flag(me, panel, s, lane, call);
}

template <bool PUSH>
__device__ __forceinline__ void pack(const Params& p, const f32x4 (&xv)[8], char* me, LAS char* lds, int lane, int wave,
                                     unsigned call, int ub, int part) {
  const int rank = p.rank, us = ub >> 1, uh = ub & 1;
  float mx = 0.f, sm = 0.f;
  u32x4 D[4];
  {
    LAS char* tw = lds + wave * XSTG + (lane >> 3) * 16 + ((lane & 7) >> 1) * 256 + (lane & 1) * 8;
#pragma unroll
    for (int i = 0; i < 8; ++i) {
      mx = fmaxf(fmaxf(mx, fmaxf(fabsf(xv[i][0]), fabsf(xv[i][1]))), fmaxf(fabsf(xv[i][2]), fabsf(xv[i][3])));
      sm += fabsf(xv[i][i & 3]);
      *(LAS u32x2*)(tw + (i >> 1) * 1024 + (i & 1) * 128) = u32x2{pk_bf16(xv[i][0], xv[i][1]), pk_bf16(xv[i][2], xv[i][3])};
    }
    if (wave == 0 && !PUSH) STAMP(56);
    cfence();
#pragma unroll
    for (int f = 0; f < 4; ++f) D[f] = *(LAS u32x4*)(lds + wave * XSTG + f * 1024 + lane * 16);
  }
  char* xl = me + XB_OFF + (size_t)((rank * 2 + uh) * NSTEP + us) * XSTG + wave * 4096 + lane * 16;
#pragma unroll
  for (int f = 0; f < 4; ++f) if (!PUSH) st16(xl + f * 1024, D[f]);
  const unsigned emax = wave_umax((pk_bf16(mx, 0.f) >> 7) & 0xffu);
  const unsigned emean = (wave_fsum(sm * (3.32f / 512.f)) >> 23) & 0xffu;
  const unsigned etop = umin(emax, emean);
  const unsigned base = etop > 7u ? etop - 7u : 0u;
  const unsigned B2 = base * 0x00800080u;
  const int j0 = ub < NPUSH ? 1 + part * 7 / NSPL : 0, j1 = ub < NPUSH ? 1 + (part + 1) * 7 / NSPL : 1;
  auto pkd = [&](int j) -> char* {
    return j ? p.ws[(rank + 1 + (unsigned)(j + ub) % 7u) & 7u] + LD_OFF + (size_t)(rank * NPUSH + ub) * PKU : me + PK_OFF + (size_t)ub * PKU;
  };
  {
    unsigned S[8];
#pragma unroll
    for (int i = 0; i < 8; ++i) {
      const unsigned d0 = D[i >> 1][2 * (i & 1)], d1 = D[i >> 1][2 * (i & 1) + 1];
      S[i] = bfi(0x80808080u, perm(d1, d0, 0x07050301u), perm(d1, d0, 0x06040200u));
    }
    if (PSTAG && ub < NPUSH) for (int i = 0; i < ub; ++i) __builtin_amdgcn_s_sleep(PSTAG);
    for (int j = j0; j < j1; ++j) {
      char* pk = pkd(j);
      st16(pk + wave * 2048 + lane * 16, u32x4{S[0], S[1], S[2], S[3]});
      st16(pk + wave * 2048 + 1024 + lane * 16, u32x4{S[4], S[5], S[6], S[7]});
    }
  }
  unsigned C[16], Ma = 0, Mb = 0;
#pragma unroll
  for (int i = 0; i < 16; ++i) {
    const unsigned t = pk_sub16(D[i >> 2][i & 3] & 0x7f807f80u, B2);
    C[i] = t & 0x03800380u;
    const unsigned y = pk_min16(t, 0x04000400u);
    if (i < 8) Ma = bfi(0x04000400u, y, Ma >> 1); else Mb = bfi(0x04000400u, y, Mb >> 1);
  }
  {
    u32x3 P;
#pragma unroll
    for (int g = 0; g < 3; ++g)
      P[g] = C[5 * g] | (C[5 * g + 1] << 3) | (C[5 * g + 2] << 6) | (((C[5 * g + 3] << 3) | C[5 * g + 4]) >> 6) | ((C[15] >> (7 + g)) & 0x00010001u);
    for (int j = j0; j < j1; ++j)
      asm volatile("global_store_dwordx3 %0, %1, off sc0 sc1\n\ts_nop 1" :: "v"(pkd(j) + PK_C + wave * 768 + lane * 12), "v"(P) : "memory");
  }
  unsigned M = (Ma >> 3) | (Mb << 5), cnt = 0;
  bool raw = FORCE_RAW;
  const int eb = wave * XSTG + 4096;
  for (;;) {
    const unsigned long long act = __builtin_amdgcn_ballot_w64(M != 0);
    if (!act) break;
    const unsigned k = M ? __builtin_ctz(M) : 0u;
    const unsigned pos = (k & 12u) * 128u + lane * 8u + (k & 3u) * 2u + (k >> 4);
    const unsigned v = *(LAS unsigned short*)(lds + wave * XSTG + pos * 2);
    const unsigned hi = (unsigned)(((int)((v >> 7) & 0xffu) - (int)base) >> 3) + 28u;
    const unsigned sl = cnt + __builtin_amdgcn_mbcnt_hi((unsigned)(act >> 32), __builtin_amdgcn_mbcnt_lo((unsigned)act, 0u));
    const unsigned slc = M != 0 && sl < EXC ? sl : 127u - 2u;
    *(LAS unsigned short*)(lds + eb + ((wave & 1) && slc < EXC ? PK_R - 6 - 2 * slc : 4 + 2 * slc)) = (unsigned short)(pos | (hi << 11));
    if (__builtin_amdgcn_ballot_w64(M != 0 && hi > 31u)) raw = true;
    cnt += (unsigned)__builtin_popcountll(act);
    M &= M - 1;
    if (cnt > EXC) { raw = true; break; }
  }
  {
    const unsigned nd = ((raw ? 0u : cnt) + 1u) / 2u + 1u;
    const int o = (wave & 1) ? PK_R - 4 - 4 * lane : 4 * lane;
    cfence();
    unsigned v = *(LAS unsigned*)(lds + eb + ((unsigned)lane < nd ? o : 0));
    cfence();
    if (!PUSH) lds_st(SY_PACK + 4 * wave, 1u);
    if (lane == 0) v = base | ((raw ? 255u : cnt) << 8);
    if ((unsigned)lane < nd)
      for (int j = j0; j < j1; ++j) st4(pkd(j) + PK_E + (wave >> 1) * PK_R + o, v);
  }
  if (wave == 0 && !PUSH) STAMP(57);
  waitv<0>();
  if (lds_ticket(SY_ARR + (PUSH ? 12 : 4)) == 3) {
    const int j = lane - 16, pr = lane < NR ? lane : (rank + 1 + (unsigned)(j + ub) % 7u) & 7u;
    if ((!PUSH && lane <= NR) || (ub < NPUSH ? j >= j0 && j < j1 : ub < 64 && j >= 1 && j < NR)) {
      char* f = lane == rank || lane == NR ? me + XF_OFF + (size_t)(((rank * 2 + uh) * 2 + (lane >> 3)) * NSTEP + us) * 4
                             : (char*)*(LAS unsigned long long*)(lds + SY_PTR + pr * 8)
                               + (lane < NR ? SF_OFF + (size_t)(rank * 256 + ub) * 4 : PF_OFF + (size_t)(rank * 64 + ub) * 128);
      st4(f, call);
    }
    STAMP(PUSH ? 6 : 1);
  }
}

__device__ __forceinline__ void decode(const u32x4 (&A)[4], const u32x3 (&P)[2], unsigned E, LAS char* lds, int io,
                                       char* dst, const char* rawp, int lane) {
  char* xd = dst + lane * 16;
  LAS char* img = lds + io;
  const unsigned v0 = lane * 16;
  const unsigned hd[2] = {(unsigned)__builtin_amdgcn_readlane(E, 0), (unsigned)__builtin_amdgcn_readlane(E, 47)};
  const unsigned c0 = (hd[0] >> 8) & 0xffu, c1 = (hd[1] >> 8) & 0xffu;
  const unsigned n0 = c0 > EXC ? 0u : c0, n1 = c1 > EXC ? 0u : c1;
  const bool clash = ((n0 + 1u) & ~1u) + ((n1 + 1u) & ~1u) > EXC;
  const bool raw[2] = {clash || c0 > EXC, clash || c1 > EXC};
  const unsigned ce = clash ? 0u : n0, co = clash ? 0u : n1;
#pragma unroll
  for (int q = 0; q < 2; ++q) {
    const unsigned B2 = (hd[q] & 0xffu) * 0x00800080u;
#pragma unroll
    for (int f = 0; f < 4; ++f) {
      u32x4 d;
#pragma unroll
      for (int j = 0; j < 4; ++j) {
        const int i = 4 * f + j, g = i / 5, rr = i % 5;
        const unsigned c = i == 15 ? ((P[q][0] & 0x00010001u) << 7) | ((P[q][1] & 0x00010001u) << 8) | ((P[q][2] & 0x00010001u) << 9)
                                   : (rr < 3 ? P[q][g] >> (3 * rr) : P[q][g] << (3 * rr - 6)) & 0x03800380u;
        const unsigned sm = A[2 * q + (f >> 1)][2 * (f & 1) + (j >> 1)];
        d[j] = (perm(sm, sm, (j & 1) ? 0x03030202u : 0x01010000u) & 0x807f807fu) | (c + B2);
      }
      *(LAS u32x4*)(img + q * 4096 + f * 1024 + lane * 16) = d;
    }
  }
  cfence();
#pragma unroll
  for (int c = 0; c < 2; ++c) {
    const unsigned ent = (E >> (16 * c)) & 0xffffu, hw = 2u * lane + c;
    const bool ev = hw - 2u < ce, od = (PK_R / 2 - 3) - hw < co;
    const unsigned dl = (ev || od) && lane < 48 ? ((ent >> 11) - 28u) << (10u + (ent & 1u) * 16u) : 0u;
    const unsigned ad = io + (od ? 4096u : 0u) + (ent & 0x7feu) * 2u;
    asm volatile("ds_add_u32 %0, %1" :: "v"(ad), "v"(dl) : "memory");
  }
  cfence();
#pragma unroll
  for (int q = 0; q < 2; ++q) {
    u32x4 R0, R1, R2, R3;
    if (raw[q]) {
      const char* rb = rawp + q * 4096;
      asm volatile("s_nop 4\n\t"
                   "global_load_dwordx4 %0, %4, %5 sc0 sc1\n\t"
                   "global_load_dwordx4 %1, %4, %5 offset:1024 sc0 sc1\n\t"
                   "global_load_dwordx4 %2, %4, %5 offset:2048 sc0 sc1\n\t"
                   "global_load_dwordx4 %3, %4, %5 offset:3072 sc0 sc1\n\t"
                   "s_waitcnt vmcnt(0)"
                   : "=&v"(R0), "=&v"(R1), "=&v"(R2), "=&v"(R3) : "v"(v0), "s"(rb) : "memory");
    } else {
      R0 = *(LAS u32x4*)(img + q * 4096 + lane * 16); R1 = *(LAS u32x4*)(img + q * 4096 + 1024 + lane * 16);
      R2 = *(LAS u32x4*)(img + q * 4096 + 2048 + lane * 16); R3 = *(LAS u32x4*)(img + q * 4096 + 3072 + lane * 16);
    }
    st16(xd + q * 4096, R0); st16(xd + q * 4096 + 1024, R1); st16(xd + q * 4096 + 2048, R2); st16(xd + q * 4096 + 3072, R3);
  }
}

#define MFMA(m, t) asm volatile("v_mfma_f32_16x16x32_bf16 %0, %1, %2, %0" : "+v"(acc[m][t]) : "v"(Wf[t]), "v"(Xc[m]))
#define LDR(dst, addr, off) asm volatile("ds_read_b128 %0, %1 offset:%2" : "=v"(dst) : "v"(addr), "i"(off) : "memory")
template <int N_> __device__ __forceinline__ void waitl() { asm volatile("s_waitcnt lgkmcnt(%0)" :: "i"(N_) : "memory"); }
constexpr int WOF = 20480;
#define MAX2(m, t) asm volatile("v_max3_f32 %0, %2, %3, %0\n\tv_max3_f32 %1, %4, %5, %1" : "+v"(mx0), "+v"(mx1) \
                                : "v"(acc[m][t][0]), "v"(acc[m][t][1]), "v"(acc[m][t][2]), "v"(acc[m][t][3]))
template <int NM, int MODE, int PAR>
__device__ __forceinline__ void mstep(f32x4 (&acc)[NM][8], bf16x8 (&Wf)[8], bf16x8 (&Xc)[NM], bf16x8 (&Xn)[NM],
                                      unsigned& va, unsigned& vx, unsigned va0, unsigned vx0, unsigned vsy, unsigned& vdone,
                                      unsigned& slot, unsigned& fpre, int s, float& mx0, float& mx1, unsigned& nslow) {
  constexpr bool PF = MODE < 2;
  unsigned sf = 0, nslot = 0;
  constexpr int NX2 = 2 + (NM == 5) + (MODE == 0);
  waitl<5>();
  MFMA(0, 0);
  LDR(Wf[7], va, WOF + 7168);
  MFMA(1, 0);
  if (PF) asm volatile("ds_write_b32 %0, %1 offset:16" :: "v"(vsy), "v"(vdone) : "memory");
  MFMA(2, 0);
  if (PF) asm volatile("v_readfirstlane_b32 %0, %1" : "=s"(sf) : "v"(fpre) : "memory");
  MFMA(3, 0);
  if (PF) asm volatile("v_add_u32 %0, %1, %2" : "=v"(va) : "s"(slot), "v"(va0));
  if (NM == 5) MFMA(NM - 1, 0);
  if (PF) asm volatile("v_add_u32 %0, %1, %2" : "=v"(vx) : "s"(slot), "v"(vx0));
  if (PF && __builtin_expect(sf <= (unsigned)(s + 1), 0)) {
    do __builtin_amdgcn_s_sleep(1); while (lds_ld1(SY_FULL + 4 * (1 - PAR)) <= (unsigned)(s + 1));
  }
#pragma unroll
  for (int t = 1; t < 8; ++t) {
    if (t == 2) waitl<PF ? 10 : 5>();
    if (t == 3) waitl<PF ? 9 + NX2 : 4>();
    if (t == 4) waitl<PF ? 9 + NX2 : 3>();
    if (t == 5) waitl<PF ? 9 + NX2 : 2>();
    if (t == 6) waitl<PF ? 9 + NX2 : 1>();
    if (t == 7) waitl<PF ? 9 + NX2 : 0>();
    MFMA(0, t);
    if (PF) LDR(Wf[t - 1], va, WOF + (t - 1) * 1024); else MAX2(0, t - 1);
    MFMA(1, t);
    if (PF && t == 1) LDR(Xn[0], vx, 0);
    if (PF && t == 2) LDR(Xn[3], vx, 3 * 1024);
    if (MODE == 0 && t == 5) asm volatile("s_add_u32 %0, %1, %2" : "=s"(nslot) : "s"(slot), "i"(STG) : "scc");
    if (MODE == 0 && t == 6) asm volatile("s_cmp_lg_u32 %1, %2\n\ts_cselect_b32 %0, %1, 0" : "=s"(slot) : "s"(nslot), "i"(NS * STG) : "scc");
    if (!PF) MAX2(1, t - 1);
    MFMA(2, t);
    if (PF && t == 1) LDR(Xn[1], vx, 1024);
    if (PF && t == 2 && NM == 5) LDR(Xn[NM - 1], vx, 4 * 1024);
    if (PF && t == 4) asm volatile("v_add_u32 %0, 1, %0" : "+v"(vdone));
    if (!PF) MAX2(2, t - 1);
    MFMA(3, t);
    if (PF && t == 1) LDR(Xn[2], vx, 2 * 1024);
    if (MODE == 0 && t == 2) asm volatile("ds_read_b32 %0, %1 offset:%2" : "=v"(fpre) : "v"(SY_FULL), "i"(4 * PAR) : "memory");
    if (!PF) MAX2(3, t - 1);
    if (NM == 5) { MFMA(NM - 1, t); if (!PF) MAX2(NM - 1, t - 1); }
  }
  if (!PF) {
    asm volatile("s_nop 7\n\ts_nop 7" ::: "memory");
#pragma unroll
    for (int m = 0; m < NM; ++m) MAX2(m, 7);
  }
}

template <int NM>
__device__ __forceinline__ void consumer(const Params& p, char* me, LAS char* lds, int lane, int wave, unsigned call,
                                         int gidx, int row0, int row4, int pn, unsigned long long t0) {
  f32x4 acc[NM][8];
#pragma unroll
  for (int m = 0; m < NM; ++m)
#pragma unroll
    for (int t = 0; t < 8; ++t) { acc[m][t] = f32x4{0.f, 0.f, 0.f, 0.f}; asm volatile("" : "+v"(acc[m][t])); }
  bf16x8 Wf[8], Xa[NM], Xb[NM];
  unsigned slot = STG, fpre, vdone = 1;
  float mx0 = 0.f, mx1 = 0.f;
  unsigned nslow = 0;
  const unsigned va0 = lane * 16, vx0 = lane * 16 + wave * 5120, vsy = SY + 4 * wave;
  unsigned va = va0, vx = vx0;
  while (lds_ld1(SY_FULL) == 0) __builtin_amdgcn_s_sleep(1);
  STAMP(8 + wave);
#pragma unroll
  for (int t = 0; t < 7; ++t) LDR(Wf[t], va, WOF + t * 1024);
#pragma unroll
  for (int t = 0; t < NM; ++t) LDR(Xa[t], vx, t * 1024);
  asm volatile("ds_read_b32 %0, %1 offset:4\n\ts_waitcnt lgkmcnt(0)" : "=v"(fpre) : "v"(SY_FULL) : "memory");
  for (int s = 0; s < NSTEP - 2; s += 2) {
    mstep<NM, 0, 0>(acc, Wf, Xa, Xb, va, vx, va0, vx0, vsy, vdone, slot, fpre, s, mx0, mx1, nslow);
    mstep<NM, 0, 1>(acc, Wf, Xb, Xa, va, vx, va0, vx0, vsy, vdone, slot, fpre, s + 1, mx0, mx1, nslow);
  }
  mstep<NM, 1, 0>(acc, Wf, Xa, Xb, va, vx, va0, vx0, vsy, vdone, slot, fpre, NSTEP - 2, mx0, mx1, nslow);
  mstep<NM, 2, 1>(acc, Wf, Xb, Xa, va, vx, va0, vx0, vsy, vdone, slot, fpre, NSTEP - 1, mx0, mx1, nslow);
  STAMP(12 + wave);
  unsigned wm = wave_umax(umax(__builtin_bit_cast(unsigned, mx0), __builtin_bit_cast(unsigned, mx1)));
  {
    unsigned tk, m4; unsigned long long pp;
    asm volatile("s_mov_b64 exec, 1\n\tds_max_u32 %3, %4\n\tds_add_rtn_u32 %0, %3, %5 offset:8\n\tds_read_b32 %1, %3\n\t"
                 "s_mov_b64 exec, -1\n\tds_read_b64 %2, %6\n\ts_waitcnt lgkmcnt(0)"
                 : "=&v"(tk), "=&v"(m4), "=&v"(pp) : "v"(SY_WMAX), "v"(wm), "v"(1u), "v"(SY_PTR + (lane & 7) * 8) : "memory");
    if (rfl(tk) == 3u) {
      const char* g = (const char*)pp + GR_OFF + (size_t)(((call & 1) * NR + p.rank) * 256 + gidx) * 8;
      asm volatile("s_mov_b64 exec, 0xff\n\ts_nop 3\n\tglobal_store_dwordx2 %0, %1, off sc0 sc1\n\ts_mov_b64 exec, 1\n\t"
                   "ds_write_b32 %2, %3 offset:4\n\ts_mov_b64 exec, -1\n\ts_nop 1"
                   :: "v"(g), "v"(u32x2{rfl(m4), call}), "v"(SY_WMAX), "v"(1u) : "memory");
    }
  }
  char* ob = p.out + pn * 128 + 16 * (lane >> 4) + (FULLN ? (lane & 7) * NLOC + (lane & 8) * 8 : (lane & 15) * NLOC);
  unsigned used = 0xffffffffu;
  for (;;) {
    u32x4 g;
    asm volatile("s_mov_b64 exec, 1\n\tds_read2_b64 %0, %1 offset1:2\n\ts_mov_b64 exec, -1\n\ts_waitcnt lgkmcnt(0)" : "=&v"(g) : "v"(SY_WMAX) : "memory");
    const unsigned st = rfl(g[3]);
    const bool bad = st == 3u, fin = st >= 2u;
    const unsigned gb = bad ? 0xfffffffeu : fin ? rfl(g[2]) : st == 1u && rfl(g[1]) ? umax(rfl(g[2]), rfl(g[0])) : used;
    if (gb != used) {
      used = gb;
      const float gmax = __builtin_bit_cast(float, gb);
      const float inv = bad ? 0.f : gmax > 0.f ? 127.0f / gmax : 0.f, mag = bad ? 128.f : 0.f;
      const f32x2 inv2 = {inv, inv}, mag2 = {mag, mag};
      if (fin) STAMP(16 + wave);
      STAMP(48 + wave);
#pragma unroll
      for (int m = 0; m < NM; ++m) {
        u32x4 o[2];
#pragma unroll
        for (int hf = 0; hf < 2; ++hf) {
          f32x2 q[4][2];
#pragma unroll
          for (int c = 0; c < 4; ++c)
#pragma unroll
            for (int jj = 0; jj < 2; ++jj)
              q[c][jj] = __builtin_elementwise_fma(f32x2{acc[m][4 * hf + c][2 * jj], acc[m][4 * hf + c][2 * jj + 1]}, inv2, mag2);
#pragma unroll
          for (int j = 0; j < 4; ++j) {
            unsigned w = 0;
#pragma unroll
            for (int c = 0; c < 4; ++c) w = __builtin_amdgcn_cvt_pk_u8_f32(q[c][j >> 1][j & 1], c, w);
            o[hf][j] = w;
          }
        }
#pragma unroll
        for (int j = 0; j < 4; ++j) {
          const unsigned lo = o[0][j], hi = o[1][j];
          o[0][j] = __builtin_amdgcn_update_dpp(lo, hi, 0x128, 0xf, 0xc, false);
          o[1][j] = __builtin_amdgcn_update_dpp(hi, lo, 0x128, 0xf, 0x3, false);
        }
        char* dst = ob + (size_t)(m < 4 ? row0 + 64 * wave + 16 * m : row4 + 16 * wave) * NLOC;
        asm volatile("global_store_dwordx4 %0, %1, off" OSC "\n\ts_nop 1" :: "v"(dst), "v"(o[0]) : "memory");
        asm volatile("global_store_dwordx4 %0, %1, off" OSC "\n\ts_nop 1" :: "v"(dst + 8 * NLOC), "v"(o[1]) : "memory");
      }
      STAMP(52 + wave);
    }
    if (fin) break;
    __builtin_amdgcn_s_sleep(1);
  }
  STAMP(20 + wave);
}

__global__ __launch_bounds__(512, 2) void fused(Params p) {
  extern __shared__ char smem[];
  LAS char* lds = (LAS char*)smem;
  const int lane = threadIdx.x & 63;
  const int wave = __builtin_amdgcn_readfirstlane(threadIdx.x >> 6);
  const int b = blockIdx.x;
  const int rank = p.rank;
  char* me = p.ws[rank];
  const char* tmo = me + TMO_OFF;
  if (wave == 0) STAMP(0);
  const unsigned long long ts = rt();

  unsigned* cntp = (unsigned*)(me + CNT_OFF + b * 64);
  unsigned cnt0, cnt1;
  asm volatile("global_load_dword %0, %1, off sc0 sc1" : "=&v"(cnt0) : "v"(cntp) : "memory");
#define GETCNT(N) asm volatile("s_waitcnt vmcnt(" #N ")\n\tv_readfirstlane_b32 %0, %1" : "=s"(cnt1) : "v"(cnt0) : "memory")
  const int wt0 = b - NCOMM + NGEMM * (7 - wave);
  const bool wup = b >= NCOMM && wave >= 8 - UPF;
  f32x4 xv[8], xw[8];
  WRegs w0;
  const int rb = NWG - 1 - b;
  const int ub = rb < NPUSH ? rb : b < NPULL ? b + NPUSH : b < NCOMM ? b + NGEMM : b - NCONV + NPUSH;
  const bool pusher = rb >= NPUSH && rb < NPUSH * NSPL;
  const int pu = NPUSH ? rb % (NPUSH + !NPUSH) : 0, pj = NPUSH ? rb / (NPUSH + !NPUSH) : 0;
  if (XDLY && rb >= NPUSH * NSPL && b >= NPULL) __builtin_amdgcn_s_sleep(XDLY);
  if (wave < 4) {
    const float* xp = p.x + (size_t)(64 * wave + (lane >> 3)) * KD + 4 * (lane & 7);
    if (pusher) {
#pragma unroll
      for (int i = 0; i < 8; ++i) xw[i] = *(const f32x4*)(xp + (size_t)(256 * (pu & 1) + i * 8) * KD + 32 * (pu >> 1));
    }
#pragma unroll
    for (int i = 0; i < 8; ++i) xv[i] = __builtin_nontemporal_load((const f32x4*)(xp + (size_t)(256 * (ub & 1) + i * 8) * KD + 32 * (ub >> 1)));
    if (pusher) GETCNT(16); else GETCNT(8);
  } else if (wup) {
    w_load(w0, p, wt0 & 15, wt0 >> 4, lane);
    GETCNT(16);
  } else {
    if (wave == 4 && lane < 36) lds_st(SY + lane * 4, 0u);
    if (wave == 5 && lane < NR) *(LAS unsigned long long*)(lds + SY_PTR + lane * 8) = (unsigned long long)p.ws[lane];
    GETCNT(0);
  }
  const unsigned call = cnt1 + 1u;
  asm volatile("s_waitcnt lgkmcnt(0)" ::: "memory");
  __builtin_amdgcn_s_barrier();
  cfence();
  if (wave == 7 && lane == 0) __hip_atomic_store(cntp, call, __ATOMIC_RELAXED, __HIP_MEMORY_SCOPE_SYSTEM);
  if (wup) {
    w_store(w0, me, wt0 & 15, wt0 >> 4, lane);
    w_flag(me, wt0 & 15, wt0 >> 4, lane, call);
    if (wave == 7) STAMP(2);
  }
  if (wave < 4) {
    if (pusher) pack<true>(p, xw, me, lds, lane, wave, call, pu, pj);
    pack<false>(p, xv, me, lds, lane, wave, call, ub, 0);
  }
  __builtin_amdgcn_s_waitcnt(0x0f70);
  const unsigned long long t0 = rt();

  if (b < NCOMM) {
    const bool conv = b >= NPULL;
    const char* tkp = me + TK_OFF + (conv ? 640 : (call & 1) * 128);
    if (b == 0 && wave == 7 && lane == 0) st4(me + TK_OFF + ((call + 1) & 1) * 128, 0u);
    if (wave == 7 && !conv) {
      const char* sf = me + SF_OFF + lane * 16;
      for (;;) {
        u32x4 f[8]; unsigned t;
        asm volatile("global_load_dwordx4 %0, %9, off sc0 sc1\n\tglobal_load_dwordx4 %1, %9, off offset:1024 sc0 sc1\n\t"
                     "global_load_dwordx4 %2, %9, off offset:2048 sc0 sc1\n\tglobal_load_dwordx4 %3, %9, off offset:3072 sc0 sc1\n\t"
                     "global_load_dwordx4 %4, %10, off sc0 sc1\n\tglobal_load_dwordx4 %5, %10, off offset:1024 sc0 sc1\n\t"
                     "global_load_dwordx4 %6, %10, off offset:2048 sc0 sc1\n\tglobal_load_dwordx4 %7, %10, off offset:3072 sc0 sc1\n\t"
                     "global_load_dword %8, %11, off sc0 sc1\n\ts_waitcnt vmcnt(0)"
                     : "=&v"(f[0]), "=&v"(f[1]), "=&v"(f[2]), "=&v"(f[3]), "=&v"(f[4]), "=&v"(f[5]), "=&v"(f[6]), "=&v"(f[7]), "=&v"(t)
                     : "v"(sf), "v"(sf + 4096), "v"(tmo) : "memory");
        unsigned all = 256;
#pragma unroll
        for (int j = 0; j < NR; ++j) {
          const unsigned c = f[j][0] < call ? 0u : f[j][1] < call ? 1u : f[j][2] < call ? 2u : f[j][3] < call ? 3u : 4u;
          const unsigned long long part = __builtin_amdgcn_ballot_w64(c < 4u && lane >= NPUSH / 4);
          unsigned r = 256;
          if (part && j != rank) { const unsigned l = __builtin_ctzll(part); r = 4 * l + __builtin_amdgcn_readlane(c, l); }
          lds_st(SY_SRC + 4 * j, r);
          all = umin(all, r);
        }
        if (all == 256) break;
        if (expired(t, call, t0, me)) {
#pragma unroll
          for (int j = 0; j < NR; ++j) lds_st(SY_SRC + 4 * j, ABORT);
          break;
        }
      }
    }
    const unsigned v0 = lane * 16, vc = lane * 12, vh = (lane < 48 ? lane : 47) * 4;
    const char* fl = me + TK_OFF + 512;
    unsigned T = NPUSH * 14 + (conv ? NWP * NPULL + wave * NCONV + (b - NPULL) : wave * NPULL + b);
    const int nw = conv ? CPULL : NWP;
    for (bool first = true; wave < nw; first = false) {
      if (T >= 7u * 512u) break;
      const unsigned hu = T / 7u, src = (rank + 1 + (T - hu * 7u)) & 7u, u = hu >> 1, r = hu & 1, s = u >> 1, h = u & 1, n = u >> 2;
      unsigned av = lds_ld1(SY_SRC + 4 * src);
      if (first) {
        while (rt() - ts < (unsigned long long)(TST0 + TSTD * (hu - 2 * NPUSH))) __builtin_amdgcn_s_sleep(1);
        const char* fp = me + PF_OFF + (size_t)(src * 64 + u) * 128;
        for (;;) {
          unsigned f, t;
          asm volatile("global_load_dword %0, %2, %3 sc0 sc1\n\tglobal_load_dword %1, %2, %4 sc0 sc1\n\ts_waitcnt vmcnt(0)"
                       : "=&v"(f), "=&v"(t) : "v"(0), "s"(fp), "s"(tmo) : "memory");
          av = u + 1;
          if (rfl(f) >= call) break;
          av = ABORT;
          if (expired(t, call, t0, me)) break;
          __builtin_amdgcn_s_sleep(1);
        }
      }
      if (av <= u) {
        waitv<0>();
        flag_st(fl, call);
        fl = me + TK_OFF + 512;
        do { __builtin_amdgcn_s_sleep(1); av = lds_ld1(SY_SRC + 4 * src); } while (av <= u);
      }
      if (av >= ABORT) break;
      if (STAG && first) for (unsigned i = 2 * NPUSH; i < hu; ++i) __builtin_amdgcn_s_sleep(STAG);
      if (n < 8 && !conv) STAMP(8 + n);
      u32x4 A[4]; u32x3 P[2]; unsigned E, tk;
      const char* sws = p.ws[src];
      const char* sb = sws + PK_OFF + (size_t)u * PKU;
      asm volatile("s_nop 4\n\t"
                   "global_load_dword %6, %10, %13 sc0 sc1\n\t"
                   "global_load_dwordx3 %4, %9, %12 sc0 sc1\n\t"
                   "global_load_dwordx3 %5, %9, %12 offset:768 sc0 sc1\n\t"
                   "global_load_dwordx4 %0, %8, %11 sc0 sc1\n\t"
                   "global_load_dwordx4 %1, %8, %11 offset:1024 sc0 sc1\n\t"
                   "global_load_dwordx4 %2, %8, %11 offset:2048 sc0 sc1\n\t"
                   "global_load_dwordx4 %3, %8, %11 offset:3072 sc0 sc1\n\t"
                   "s_mov_b64 exec, 1\n\ts_nop 3\n\t"
                   "global_atomic_add %7, %8, %16, %14 sc0\n\t"
                   "s_waitcnt vmcnt(8)\n\t"
                   "global_store_dword %8, %17, %15 sc0 sc1\n\t"
                   "s_mov_b64 exec, -1\n\t"
                   "s_waitcnt vmcnt(1)"
                   : "=&v"(A[0]), "=&v"(A[1]), "=&v"(A[2]), "=&v"(A[3]), "=&v"(P[0]), "=&v"(P[1]), "=&v"(E), "=&v"(tk)
                   : "v"(v0), "v"(vc), "v"(vh), "s"(sb + r * 4096), "s"(sb + PK_C + r * 1536), "s"(sb + PK_E + r * PK_R),
                     "s"(tkp), "s"(fl), "v"(1u), "v"(call) : "memory");
      const size_t xo = XB_OFF + (size_t)((src * 2 + h) * NSTEP + s) * XSTG + r * 8192;
      decode(A, P, E, lds, wave * XSTG, me + xo, sws + xo, lane);
      fl = me + XF_OFF + (size_t)(((src * 2 + h) * 2 + r) * NSTEP + s) * 4;
      if (n < 8) STAMP(16 + n);
      if ((n & 7) == 7) STAMP(48 + n / 8);
      if (conv) break;
      T = umin(rfl(tk), 7u * 512u) + NWP * NPULL + CPULL * NCONV + NPUSH * 14;
    }
    if (wave < nw) { waitv<0>(); flag_st(fl, call); }
    if (conv) {
      auto next = [&]() {
        const int q = (b - NPULL) + NCONV * (int)lds_ticket(SY_TKT);
        return NGEMM * (UPF + UPC) + q / (9 - WH) * 9 + q % (9 - WH);
      };
      for (;;) {
        const int t = next();
        if (t >= 16 * NSTEP) break;
        if ((t & 255) < NCONV) STAMP(8 + (t >> 8));
        convert_w(p, me, t & 15, t >> 4, lane, call);
      }
      if (wave == 0) STAMP(2);
      return;
    }
    if (wave < NWP) { STAMP(24 + wave); lds_st(SY_TKT, 64u); }
    if (b >= 8) return;
    if (wave >= NWP) {
      while (lds_ld1(SY_TKT) < 64u) __builtin_amdgcn_s_sleep(32);
    }
    const int lg = lane < NGEMM / 4 ? lane : NGEMM / 4 - 1;
    const char* g0 = me + GR_OFF + (size_t)(((call & 1) * NR + wave) * 256) * 8 + lg * 32;
    unsigned seen = 0, told = 0xffffffffu;
    const unsigned long long tp = rt(), real = (1ull << (NGEMM / 4)) - 1;
    const char* gp = me + GM_OFF + b * 128;
    for (;;) {
      u32x4 a0, a1;
      asm volatile("global_load_dwordx4 %0, %2, off sc0 sc1\n\tglobal_load_dwordx4 %1, %2, off offset:16 sc0 sc1\n\ts_waitcnt vmcnt(0)"
                   : "=&v"(a0), "=&v"(a1) : "v"(g0) : "memory");
      const unsigned cnt = __builtin_popcountll(__builtin_amdgcn_ballot_w64(a0[1] == call) & real) + __builtin_popcountll(__builtin_amdgcn_ballot_w64(a0[3] == call) & real)
                         + __builtin_popcountll(__builtin_amdgcn_ballot_w64(a1[1] == call) & real) + __builtin_popcountll(__builtin_amdgcn_ballot_w64(a1[3] == call) & real);
      if (cnt != seen && (SPEC || cnt == (unsigned)NGEMM)) {
        const unsigned part = wave_umax(umax(umax(a0[1] == call ? a0[0] : 0u, a0[3] == call ? a0[2] : 0u),
                                             umax(a1[1] == call ? a1[0] : 0u, a1[3] == call ? a1[2] : 0u)));
        unsigned v0; u32x2 gt;
        asm volatile("s_mov_b64 exec, 1\n\tds_max_u32 %2, %3\n\tds_add_rtn_u32 %0, %2, %4 offset:4\n\tds_read_b64 %1, %2\n\t"
                     "s_mov_b64 exec, -1\n\ts_waitcnt lgkmcnt(0)"
                     : "=&v"(v0), "=&v"(gt) : "v"(SY_GMAX), "v"(part), "v"(cnt - seen) : "memory");
        const unsigned gm = rfl(gt[0]), tot = rfl(v0) + cnt - seen;
        seen = cnt;
        if (tot == (unsigned)(NR * NGEMM)) {
          asm volatile("s_mov_b64 exec, 1\n\ts_nop 3\n\tglobal_store_dwordx2 %0, %1, %2" GSC "\n\t"
                       "global_store_dwordx2 %0, %1, %2 offset:1024 sc0 sc1\n\ts_mov_b64 exec, -1\n\ts_nop 1"
                       :: "v"(0), "v"(u32x2{gm, call}), "s"(gp) : "memory");
          STAMP(4);
        } else if (SPEC && tot >= SPEC && gm != told) {
          told = gm;
          asm volatile("s_mov_b64 exec, 1\n\ts_nop 3\n\tglobal_store_dwordx2 %0, %1, %2 offset:8" GSC "\n\ts_mov_b64 exec, -1\n\ts_nop 1"
                       :: "v"(0), "v"(u32x2{gm, call}), "s"(gp) : "memory");
        }
        if (cnt == (unsigned)NGEMM) break;
        continue;
      }
      if (rt() - tp > 3000u) {
        unsigned t;
        asm volatile("global_load_dword %0, %1, %2 sc0 sc1\n\ts_waitcnt vmcnt(0)" : "=&v"(t) : "v"(0), "s"(tmo) : "memory");
        if (expired(t, call, t0, me)) break;
      }
      if (ASLP) __builtin_amdgcn_s_sleep(ASLP);
    }
    return;
  }

  const int xcd = b & 7, idx = (b >> 3) - NCOMM / 8, xh = xcd & 1;
  const bool extra = idx < 24;
  const int di = extra ? idx >> 2 : 6;
  const int pn = (xcd >> 1) * 4 + (extra ? idx & 3 : 2 * xh + (idx & 1));
  const int src = (rank + 1 + di) & 7, h = extra ? xh : 0;
  const int esrc = di < 4 ? rank : (rank + 7) & 7, eh = di < 4 ? xh : extra ? 1 : 0, eq = di < 4 ? di : 2 * xh + (di & 1);
  if (wave < 4) {
    if (wave < UPC) {
      const int t = NGEMM * (UPF + wave) + b - NCOMM;
      convert_w(p, me, t & 15, t >> 4, lane, call);
      if (wave == UPC - 1) STAMP(7);
    }
    __builtin_amdgcn_s_setprio(PRIOC);
    if (extra) consumer<5>(p, me, lds, lane, wave, call, b - NCOMM, src * 512 + h * 256, esrc * 512 + eh * 256 + eq * 64, pn, t0);
    else consumer<4>(p, me, lds, lane, wave, call, b - NCOMM, src * 512 + h * 256, 0, pn, t0);
    return;
  }
  bool dead = false;
  if (wave < 6) {
    __builtin_amdgcn_s_setprio(PRIOL);
    const char* xs = me + XB_OFF + (size_t)((src * 2 + h) * NSTEP) * XSTG;
    const char* ls = me + XB_OFF + (size_t)((esrc * 2 + eh) * NSTEP) * XSTG + eq * 4096;
    const char* wsr = me + WB_OFF + (size_t)(pn * NSTEP) * WSTG;
    const unsigned voff = lane * 16, pad = lane < 4 ? SY_DONE + 4 * lane : SY_READY;
    unsigned slot = (wave - 4) * STG;
    const int fw = SY_FULL + 4 * (wave - 4);
    const char* tok = me + IO_OFF + (size_t)(b * 2 + (wave - 4)) * 256;
    st4((char*)tok + lane * 4, 2 * lane + (wave - 4) + 1);
    waitv<0>();
    for (;;) {
      u32x4 d;
      asm volatile("ds_read_b128 %0, %1\n\ts_waitcnt lgkmcnt(0)" : "=&v"(d) : "v"(SY_PACK) : "memory");
      if (rfl(d[0] & d[1] & d[2] & d[3])) break;
      __builtin_amdgcn_s_sleep(2);
    }
    for (int s = wave - 4; s < NSTEP; s += 2) {
      unsigned rdy;
      for (;;) {
        unsigned r, dm, d1, d2, d3;
        asm volatile("s_mov_b64 exec, 31\n\tds_read_b32 %0, %6\n\ts_mov_b64 exec, -1\n\ts_waitcnt lgkmcnt(0)\n\t"
                     "v_readlane_b32 %1, %0, 4\n\tv_readlane_b32 %2, %0, 0\n\tv_readlane_b32 %3, %0, 1\n\t"
                     "v_readlane_b32 %4, %0, 2\n\tv_readlane_b32 %5, %0, 3\n\ts_nop 0\n\t"
                     "s_min_u32 %2, %2, %3\n\ts_min_u32 %4, %4, %5\n\ts_min_u32 %2, %2, %4"
                     : "=&v"(r), "=s"(rdy), "=s"(dm), "=s"(d1), "=s"(d2), "=s"(d3) : "v"(pad) : "memory", "scc");
        if (rdy >= ABORT || (rdy > (unsigned)s && dm + NS > (unsigned)s)) break;
        __builtin_amdgcn_s_sleep(SLEEPL);
      }
      if (rdy >= ABORT) { dead = true; break; }
      const int sa = s;
      const char* xa = xs + (size_t)sa * XSTG;
#define PACE_
      glds4(voff, wsr + (size_t)sa * WSTG, slot + WOF); PACE_ glds4(voff, wsr + (size_t)sa * WSTG + 4096, slot + WOF + 4096); PACE_
      glds4(voff, xa, slot); PACE_ glds4(voff, xa + 4096, slot + 5120); PACE_ glds4(voff, xa + 8192, slot + 10240); PACE_ glds4(voff, xa + 12288, slot + 15360); PACE_
      if (extra) glds4s(voff, ls + (size_t)sa * XSTG, slot + 4096);
      asm volatile("s_mov_b32 m0, %2\n\ts_mov_b64 exec, 1\n\ts_nop 1\n\tglobal_load_lds_dword %0, %1 sc0 sc1\n\ts_mov_b64 exec, -1"
                   :: "v"(voff), "s"(tok + (s >> 1) * 4), "s"(fw) : "memory");
      slot += 2 * STG; if (slot >= NS * STG) slot -= NS * STG;
    }
    waitv<0>();
    if (dead) lds_st(fw, ABORT);
    __builtin_amdgcn_s_setprio(0);
  } else if (wave == 6) {
    __builtin_amdgcn_s_setprio(PRIOP);
    const char* pa = lane < 32 ? me + XF_OFF + (size_t)((src * 2 + h) * 2 * NSTEP) * 4 + lane * 16
                               : me + WF_OFF + (size_t)(pn * NSTEP) * 4 + (lane & 31) * 16;
    const char* pb = me + XF_OFF + (size_t)((lane < 32 ? (esrc * 2 + eh) * 2 + (eq >> 1) : (src * 2 + h) * 2 + 1) * NSTEP) * 4 + (lane & 31) * 16;
    unsigned ready = 0;
    while (!dead && ready < (unsigned)NSTEP) {
      if (ready > lds_ld1(SY_DONE) + NS + 2) { __builtin_amdgcn_s_sleep(SLEEPP); continue; }
      u32x4 f, g; unsigned t;
      asm volatile("global_load_dwordx4 %0, %3, off sc0 sc1\n\tglobal_load_dwordx4 %1, %4, off sc0 sc1\n\t"
                   "global_load_dword %2, %5, off sc0 sc1\n\ts_waitcnt vmcnt(0)"
                   : "=&v"(f), "=&v"(g), "=&v"(t) : "v"(pa), "v"(pb), "v"(tmo) : "memory");
#pragma unroll
      for (int i = 0; i < 4; ++i) f[i] = umin(f[i], g[i]);
      unsigned c = f[0] < call ? 0u : f[1] < call ? 1u : f[2] < call ? 2u : f[3] < call ? 3u : 4u;
      unsigned long long part = __builtin_amdgcn_ballot_w64(c < 4u);
      unsigned px = (unsigned)part, pw = (unsigned)(part >> 32);
      unsigned rx = NSTEP, rw = NSTEP;
      if (px) { unsigned l = __builtin_ctz(px); rx = 4 * l + __builtin_amdgcn_readlane(c, l); }
      if (pw) { unsigned l = __builtin_ctz(pw); rw = 4 * l + __builtin_amdgcn_readlane(c, 32 + l); }
      unsigned r = umin(rx, rw);
      if (r > ready) {
        ready = r; lds_st(SY_READY, ready); continue; }
      if (expired(t, call, t0, me)) { dead = true; break; }
      __builtin_amdgcn_s_sleep(1);
    }
    if (dead) lds_st(SY_READY, ABORT);
    STAMP(3);
    while (!dead && lds_ld1(SY_DONE) < (unsigned)(NSTEP - EPOLL)) __builtin_amdgcn_s_sleep(4);
    const char* gp = me + GM_OFF + xcd * 128;
    unsigned told = 0xffffffffu;
    const unsigned long long tp = rt();
    for (;;) {
      u32x4 g;
      asm volatile("global_load_dwordx4 %0, %1, %2" GLC "\n\ts_waitcnt vmcnt(0)" : "=&v"(g) : "v"(0), "s"(gp) : "memory");
      bool bad = false, fin = rfl(g[1]) == call;
      if (!fin && rt() - tp > 2000u) {
        u32x2 h2; unsigned t;
        asm volatile("global_load_dwordx2 %0, %2, %3 offset:1024 sc0 sc1\n\tglobal_load_dword %1, %2, %4 sc0 sc1\n\ts_waitcnt vmcnt(0)"
                     : "=&v"(h2), "=&v"(t) : "v"(0), "s"(gp), "s"(tmo) : "memory");
        bad = expired(t, call, t0, me);
        fin = bad || rfl(h2[1]) == call;
        g[0] = h2[0];
      }
      const unsigned v = fin ? rfl(g[0]) : SPEC && rfl(g[3]) == call ? rfl(g[2]) : told;
      if (fin || v != told) {
        if (fin) STAMP(58); else STAMP(59);
        told = v;
        asm volatile("ds_write_b64 %0, %1" :: "v"(SY_GMAX), "v"(u32x2{v, bad ? 3u : fin ? 2u : 1u}) : "memory");
      }
      if (fin) break;
      __builtin_amdgcn_s_sleep(1);
    }
  } else {
    for (unsigned T = b - NCOMM; T < NPUSH * 14u; T += NGEMM) {
      const unsigned hu = T / 7u, sr = (rank + 1 + (T - hu * 7u)) & 7u, u = hu >> 1, r = hu & 1;
      auto wait_flag = [&](const char* fp) {
        for (;;) {
          unsigned f, t;
          asm volatile("global_load_dword %0, %2, %3 sc0 sc1\n\tglobal_load_dword %1, %2, %4 sc0 sc1\n\ts_waitcnt vmcnt(0)"
                       : "=&v"(f), "=&v"(t) : "v"(0), "s"(fp), "s"(tmo) : "memory");
          if (rfl(f) >= call) break;
          if (expired(t, call, t0, me)) { dead = true; break; }
          __builtin_amdgcn_s_sleep(1);
        }
      };
      wait_flag(me + PF_OFF + (size_t)(sr * 64 + u) * 128);
      if (dead) break;
      STAMP(4);
      u32x4 A[4]; u32x3 P[2]; unsigned E;
      const char* sb = me + LD_OFF + (size_t)(sr * NPUSH + u) * PKU;
      asm volatile("s_nop 4\n\t"
                   "global_load_dword %6, %9, %12 sc0 sc1\n\t"
                   "global_load_dwordx3 %4, %8, %11 sc0 sc1\n\t"
                   "global_load_dwordx3 %5, %8, %11 offset:768 sc0 sc1\n\t"
                   "global_load_dwordx4 %0, %7, %10 sc0 sc1\n\t"
                   "global_load_dwordx4 %1, %7, %10 offset:1024 sc0 sc1\n\t"
                   "global_load_dwordx4 %2, %7, %10 offset:2048 sc0 sc1\n\t"
                   "global_load_dwordx4 %3, %7, %10 offset:3072 sc0 sc1\n\t"
                   "s_waitcnt vmcnt(0)"
                   : "=&v"(A[0]), "=&v"(A[1]), "=&v"(A[2]), "=&v"(A[3]), "=&v"(P[0]), "=&v"(P[1]), "=&v"(E)
                   : "v"(lane * 16), "v"(lane * 12), "v"((lane < 48 ? lane : 47) * 4), "s"(sb + r * 4096), "s"(sb + PK_C + r * 1536),
                     "s"(sb + PK_E + r * PK_R) : "memory");
      const size_t xo = XB_OFF + (size_t)((sr * 2 + (u & 1)) * NSTEP + (u >> 1)) * XSTG + r * 8192;
      const unsigned c0 = (rfl(E) >> 8) & 0xffu, c1 = ((unsigned)__builtin_amdgcn_readlane(E, 47) >> 8) & 0xffu;
      if (c0 + c1 + 2u > EXC) wait_flag(me + SF_OFF + (size_t)(sr * 256 + u) * 4);
      if (dead) break;
      decode(A, P, E, lds, SCR, me + xo, p.ws[sr] + xo, lane);
      waitv<0>();
      flag_st(me + XF_OFF + (size_t)(((sr * 2 + (u & 1)) * 2 + r) * NSTEP + (u >> 1)) * 4, call);
      STAMP(5);
    }
    if (WH) {
      const int g = b - NCOMM;
      while (rt() - ts < (unsigned long long)(WH0 + WHD * g)) __builtin_amdgcn_s_sleep(32);
      for (int t = NGEMM * (UPF + UPC) + 9 * g + 9 - WH; t < NGEMM * (UPF + UPC) + 9 * g + 9 && t < 16 * NSTEP; ++t) convert_w(p, me, t & 15, t >> 4, lane, call);
    }
  }
}

}

extern "C" void kernel_launch(int rank, int nranks, ncclComm_t comm, hipStream_t stream,
                              void* const* d_in, const int* in_sizes, int n_in,
                              void* d_out, int out_size, const int* in_dtypes, int out_dtype,
                              void* const* d_out_all, void* const* d_ws_all, size_t ws_bytes) {
  if (nranks != NR || (unsigned)rank >= (unsigned)NR || n_in < 2 || ws_bytes < WS_NEED ||
      in_sizes[0] != 512 * KD || out_size != 4096 * NLOC) {
    fprintf(stderr, "kernel_launch: unexpected problem shape\n");
    return;
  }
  HIP_CHECK(hipFuncSetAttribute((const void*)fused, hipFuncAttributeMaxDynamicSharedMemorySize, LDS_BYTES));
  Params p;
  p.x = (const float*)d_in[0]; p.w = (const float*)d_in[1]; p.out = (char*)d_out; p.rank = rank;
  for (int i = 0; i < NR; ++i) p.ws[i] = (char*)d_ws_all[i];
  fused<<<NWG, 512, LDS_BYTES, stream>>>(p);
  HIP_CHECK(hipGetLastError());
}
```
